```python
import numpy as np

kernel_name = "gpu_gemm_ar_m4096_k4096_n32768_f32_gelu"


ABSMAX_RTOL = 0.001

COMPUTE_DTYPE = "float32"
EPILOGUE = "gelu"
IN_QUANT = None
OUT_QUANT = None
IN_DTYPES = {"x": "float32", "w_mat": "float32"}
WORLD_SIZE = 8

def setup_inputs():

    x = np.random.randn(4096, 4096).astype(np.float32)
    w_mat = (np.random.randn(4096, 32768) / np.sqrt(4096)).astype(np.float32)
    return {"x": x, "w_mat": w_mat}

def _epilogue(y):
    c = 0.7978845608028654
    return (0.5 * y * (1.0 + np.tanh(c * (y + 0.044715 * y ** 3)))).astype(np.float32)

def reference(x, w_mat):
    y = (x @ w_mat).astype(np.float32)
    return _epilogue(y)
```

```cpp
#include <hip/hip_runtime.h>
#include <rccl/rccl.h>
#include <cstdio>
#include <cstdint>
#include <type_traits>

#define HIP_CHECK(expr) do { hipError_t _e = (expr); if (_e != hipSuccess) { \
    fprintf(stderr, "HIP error %s at %s:%d\n", hipGetErrorString(_e), __FILE__, __LINE__); \
    return; } } while (0)

namespace fk {
typedef _Float16 h16x8 __attribute__((ext_vector_type(8)));
typedef float f32x4 __attribute__((ext_vector_type(4)));
typedef unsigned u32x4 __attribute__((ext_vector_type(4)));
#define LAS __attribute__((address_space(3)))

constexpr int M = 4096, N = 32768, NR = 8, KL = 512;
constexpr int NPM = 16, NPN = 128, NS = 64;
#define NCOMM_ 16
#define NEX_ 2
constexpr int NCOMM = NCOMM_, GRID = 256, WPX = (GRID - NCOMM) / 8;
constexpr int NEX = NEX_;
constexpr int NCOM = NPN - NR * NEX;
constexpr int ENT = 2 * NCOM + 2 * NEX, XPOS = 3 * WPX / 4 * 4;
constexpr int XGRP = XPOS / WPX;
constexpr int NGRP = (ENT + WPX - 1) / WPX;
constexpr unsigned BLK = 32768, SLAB = 16384;
#define TPAD 36864
constexpr size_t TILE_BYTES = (size_t)NS * BLK + TPAD;
constexpr size_t BUF_BYTES = (size_t)(NPM + NPN) * TILE_BYTES;
constexpr size_t CTRL_BYTES = 1u << 20;
constexpr size_t WS_NEED = CTRL_BYTES + 2 * BUF_BYTES;
constexpr unsigned O_TICKET = 0, O_TMO = 128, O_XCNT = 1024  , O_DONE = 4096  ,
                   O_FLAG = 8192  , O_DBG = 65536;
constexpr unsigned long long DEADLINE = 50000000ull;
constexpr int STAGE_BYTES = 4 * 2 * SLAB;
constexpr int L_CALL = STAGE_BYTES, L_P = STAGE_BYTES + 4, L_BAD = STAGE_BYTES + 8, L_FLG = STAGE_BYTES + 256  ;
constexpr int LDS_BYTES = STAGE_BYTES + 1024;
constexpr int AUX_SYS = 17, AUX_SC1 = 16;
#define SCHEDV 1
#define DMAV 1
#define AUX_LD 16

__host__ __device__ __forceinline__ constexpr int colmap(int pn) {
    return pn;
}
struct Params { const float* x; const float* w; float* out[NR]; char* ws[NR]; char* myws; float* myout; int rank; int mode; };
struct Ctx { char* myws; float* myout; int rank, mode; };

__device__ __forceinline__ unsigned ld_sys(const unsigned* p) { return __hip_atomic_load(p, __ATOMIC_RELAXED, __HIP_MEMORY_SCOPE_SYSTEM); }
__device__ __forceinline__ unsigned ld_agent(const unsigned* p) { return __hip_atomic_load(p, __ATOMIC_RELAXED, __HIP_MEMORY_SCOPE_AGENT); }
__device__ __forceinline__ void st_sys(unsigned* p, unsigned v) { __hip_atomic_store(p, v, __ATOMIC_RELAXED, __HIP_MEMORY_SCOPE_SYSTEM); }
__device__ __forceinline__ void st_agent(unsigned* p, unsigned v) { __hip_atomic_store(p, v, __ATOMIC_RELAXED, __HIP_MEMORY_SCOPE_AGENT); }
__device__ __forceinline__ unsigned lds_ld(LAS unsigned char* lds, int off) { return *(volatile LAS unsigned*)(lds + off); }
__device__ __forceinline__ void lds_st(LAS unsigned char* lds, int off, unsigned v) { *(volatile LAS unsigned*)(lds + off) = v; }
template <class T> __device__ __forceinline__ T* uni_ptr(T* q) {
    const unsigned long long v = (unsigned long long)q;
    return (T*)(((unsigned long long)(unsigned)__builtin_amdgcn_readfirstlane((int)(v >> 32)) << 32) | (unsigned)__builtin_amdgcn_readfirstlane((int)v));
}
#define WAIT_V(n) asm volatile("s_waitcnt vmcnt(" #n ")" ::: "memory")
#define WAIT_L(n) asm volatile("s_waitcnt lgkmcnt(" #n ")" ::: "memory")
#define BAR __builtin_amdgcn_s_barrier()
#define SCHED __builtin_amdgcn_sched_barrier(0)
#define ACLOB "a0","a1","a2","a3","a4","a5","a6","a7","a8","a9","a10","a11","a12","a13","a14","a15","a16","a17","a18","a19","a20","a21","a22","a23","a24","a25","a26","a27","a28","a29","a30","a31","a32","a33","a34","a35","a36","a37","a38","a39","a40","a41","a42","a43","a44","a45","a46","a47","a48","a49","a50","a51","a52","a53","a54","a55","a56","a57","a58","a59","a60","a61","a62","a63","a64","a65","a66","a67","a68","a69","a70","a71","a72","a73","a74","a75","a76","a77","a78","a79","a80","a81","a82","a83","a84","a85","a86","a87","a88","a89","a90","a91","a92","a93","a94","a95","a96","a97","a98","a99","a100","a101","a102","a103","a104","a105","a106","a107","a108","a109","a110","a111","a112","a113","a114","a115","a116","a117","a118","a119","a120","a121","a122","a123","a124","a125","a126","a127","a128","a129","a130","a131","a132","a133","a134","a135","a136","a137","a138","a139","a140","a141","a142","a143","a144","a145","a146","a147","a148","a149","a150","a151","a152","a153","a154","a155","a156","a157","a158","a159","a160","a161","a162","a163","a164","a165","a166","a167","a168","a169","a170","a171","a172","a173","a174","a175","a176","a177","a178","a179","a180","a181","a182","a183","a184","a185","a186","a187","a188","a189","a190","a191","a192","a193","a194","a195","a196","a197","a198","a199","a200","a201","a202","a203","a204","a205","a206","a207","a208","a209","a210","a211","a212","a213","a214","a215","a216","a217","a218","a219","a220","a221","a222","a223","a224","a225","a226","a227","a228","a229","a230","a231","a232","a233","a234","a235","a236","a237","a238","a239","a240","a241","a242","a243","a244","a245","a246","a247","a248","a249","a250","a251","a252","a253","a254","a255"
template <int I> __device__ __forceinline__ void mfma4(const h16x8& a, const h16x8& b0, const h16x8& b1, const h16x8& b2, const h16x8& b3) {
    asm volatile("v_mfma_f32_16x16x32_f16 a[%c5:%c6], %0, %1, a[%c5:%c6]\n\tv_mfma_f32_16x16x32_f16 a[%c7:%c8], %0, %2, a[%c7:%c8]\n\t"
                 "v_mfma_f32_16x16x32_f16 a[%c9:%c10], %0, %3, a[%c9:%c10]\n\tv_mfma_f32_16x16x32_f16 a[%c11:%c12], %0, %4, a[%c11:%c12]"
                 :: "v"(a), "v"(b0), "v"(b1), "v"(b2), "v"(b3), "i"(4 * I), "i"(4 * I + 3), "i"(4 * I + 4), "i"(4 * I + 7), "i"(4 * I + 8), "i"(4 * I + 11), "i"(4 * I + 12), "i"(4 * I + 15) : ACLOB);
}
template <int R> __device__ __forceinline__ float acc_take() {
    float v; asm volatile("v_accvgpr_read_b32 %0, a[%c1]\n\tv_accvgpr_write_b32 a[%c1], 0" : "=v"(v) : "i"(R) : ACLOB); return v;
}
template <int R> __device__ __forceinline__ void acc_zero() { asm volatile("v_accvgpr_write_b32 a[%c0], 0" :: "i"(R) : ACLOB); }
template <int... Is> struct Seq {};
template <int N_, int... Is> struct MkSeq : MkSeq<N_ - 1, N_ - 1, Is...> {};
template <int... Is> struct MkSeq<0, Is...> { typedef Seq<Is...> type; };
template <class F, int... Is> __device__ __forceinline__ void static_for_impl(F&& f, Seq<Is...>) { (f(std::integral_constant<int, Is>{}), ...); }
template <int N_, class F> __device__ __forceinline__ void static_for(F&& f) { static_for_impl(f, typename MkSeq<N_>::type{}); }

__device__ __forceinline__ bool give_up(unsigned* tmo, unsigned call, unsigned long long t0) {
    const bool g = (unsigned)__builtin_amdgcn_readfirstlane(ld_agent(tmo)) == call || __builtin_amdgcn_s_memrealtime() - t0 > DEADLINE;
    if (g) st_agent(tmo, call);
    return g;
}

constexpr int grp_start_c(int g) {
    if (g <= 0) return 0;
    int e = g * WPX - 1; if (e > ENT - 1) e = ENT - 1;
    int r = e < XPOS ? e : (e < XPOS + 2 * NEX ? XPOS - 1 : e - 2 * NEX);
    return 2 * (r / 4 + 1);
}
struct GrpTab { int s[NGRP + 1]; constexpr GrpTab() : s{} { for (int g = 0; g <= NGRP; ++g) s[g] = grp_start_c(g); } };
constexpr GrpTab GT{};
constexpr int NBLK = 8 * (NPM + NPN);
struct Blk { int T, j, dest; };
__device__ __forceinline__ Blk decode(int b) {
    Blk r; r.T = 0; r.j = 0; r.dest = -1;
#pragma unroll
    for (int g = 0; g < NGRP; ++g) {
        const int s = GT.s[g], nb = GT.s[g + 1] - s, mem = nb + (g == 0 ? NPM : 0) + (g == XGRP ? NR * NEX : 0);
        if (b >= 0 && b < 8 * mem) {
            const int j = b / mem, m = b - j * mem; r.j = j;
            if (g == 0) r.T = m < 2 * NPM ? ((m & 1) ? NPM + (m >> 1) : (m >> 1)) : m;
            else if (m < nb) r.T = NPM + s + m;
            else { r.dest = (m - nb) / NEX; r.T = NPM + NCOM + (m - nb); }
        }
        b -= 8 * mem;
    }
    return r;
}
static_assert(GT.s[1] >= NPM, "group 0 interleave assumes at least 16 column tiles");
static_assert(GT.s[NGRP] == NCOM, "groups cover every common column tile");
static_assert(XGRP > 0, "exclusive tiles not in group 0");
__device__ __forceinline__ void slab_load(const Params& p, const Blk& k, int kf, int wid, int lane, f32x4 (&v)[8]) {
    const bool isx = k.T < NPM;
    const float* src = isx ? p.x + (size_t)(k.T * 256 + wid * 64 + (lane & 15)) * KL + k.j * 64 + kf * 32 + (lane >> 4) * 8
                           : p.w + (size_t)(k.j * 64 + kf * 32 + (lane >> 4) * 8) * N + (size_t)colmap(k.T - NPM) * 256 + wid * 64 + (lane & 15) * 4;
#pragma unroll
    for (int r = 0; r < 8; ++r) {
        const size_t o = isx ? (size_t)((r >> 1) * 16 * KL + (r & 1) * 4) : (size_t)r * N;
        asm volatile("global_load_dwordx4 %0, %1, off nt" : "=v"(v[r]) : "v"(src + o) : "memory");
    }
}
#define PIN_LOADS() asm volatile("; pin loads" : "+v"(v[0]), "+v"(v[1]), "+v"(v[2]), "+v"(v[3]), "+v"(v[4]), "+v"(v[5]), "+v"(v[6]), "+v"(v[7]) :: "memory")
__device__ __forceinline__ void slab_convert(const Blk& k, const f32x4 (&v)[8], u32x4 (&o)[4]) {
    if (k.T < NPM) {
#pragma unroll
        for (int i = 0; i < 4; ++i) { h16x8 h;
#pragma unroll
            for (int e = 0; e < 4; ++e) { h[e] = (_Float16)v[2 * i][e]; h[4 + e] = (_Float16)v[2 * i + 1][e]; }
            o[i] = __builtin_bit_cast(u32x4, h); }
    } else {
#pragma unroll
        for (int i = 0; i < 4; ++i) { h16x8 h;
#pragma unroll
            for (int r = 0; r < 8; ++r) h[r] = (_Float16)v[r][i];
            o[i] = __builtin_bit_cast(u32x4, h); }
    }
}

__device__ __forceinline__ void comm_role(const Params& p, const Ctx& cx, LAS unsigned char* lds, unsigned call) {
    const int tid = threadIdx.x, wid = __builtin_amdgcn_readfirstlane(tid >> 6), lane = tid & 63, c = blockIdx.x, rank = cx.rank;
    char* myws = cx.myws;
    unsigned* tmo = (unsigned*)(myws + O_TMO);
    const size_t bufoff = CTRL_BYTES + (size_t)(call & 1u) * BUF_BYTES;
    __amdgpu_buffer_rsrc_t rs[NR];
#pragma unroll
    for (int q = 0; q < NR; ++q) rs[q] = __builtin_amdgcn_make_buffer_rsrc((void*)(p.ws[(rank + 1 + q) & 7] + bufoff), 0, (int)BUF_BYTES, 0x00020000);
    unsigned* flag_base = (unsigned*)(p.ws[lane & 7] + O_FLAG);
    const int voff = wid * 4096 + lane * 16;
    auto publish = [&](const Blk& k) {
        if (wid == 0 && lane < NR && (k.dest < 0 || k.dest == lane)) st_sys(flag_base + k.T * NS + k.j * 8 + rank, call);
    };
    if (!(p.mode & 2)) {
        u32x4 o[4];
        Blk prev = decode(c), pprev = prev;
        { f32x4 v[8]; slab_load(p, prev, 0, wid, lane, v); WAIT_V(0); PIN_LOADS(); slab_convert(prev, v, o); }
        const int nit = 2 * ((NBLK - c + NCOMM - 1) / NCOMM);
        for (int it = 0; it < nit; ++it) {
            f32x4 v[8];
            const int nx = it + 1 < nit ? it + 1 : it;
            const Blk cur = (nx & 1) ? prev : decode(c + (nx >> 1) * NCOMM);
            slab_load(p, cur, nx & 1, wid, lane, v);
            const unsigned base = (unsigned)prev.T * (unsigned)TILE_BYTES + (unsigned)(prev.j * 8 + rank) * BLK + (unsigned)(it & 1) * SLAB;
            if (prev.dest < 0) {
#pragma unroll
                for (int q = 0; q < NR; ++q)
#pragma unroll
                    for (int i = 0; i < 4; ++i) __builtin_amdgcn_raw_buffer_store_b128(o[i], rs[q], voff + i * 1024, (int)base, AUX_SYS);
                WAIT_V(32);
            } else {
                const __amdgpu_buffer_rsrc_t r1 = __builtin_amdgcn_make_buffer_rsrc((void*)(p.ws[prev.dest & 7] + bufoff), 0, (int)BUF_BYTES, 0x00020000);
#pragma unroll
                for (int i = 0; i < 4; ++i) __builtin_amdgcn_raw_buffer_store_b128(o[i], r1, voff + i * 1024, (int)base, AUX_SYS);
                WAIT_V(4);
            }
            PIN_LOADS();
            if (it > 0 && !(it & 1)) { BAR; publish(pprev); }
            slab_convert(cur, v, o);
            pprev = prev; prev = cur;
        }
        WAIT_V(0); BAR;
        publish(pprev);
    }
    bool bad = false;
    if (!(p.mode & 6)) {
        for (int xt = c; xt < NPM * NEX; xt += NCOMM) {
            const unsigned* xc = (const unsigned*)(myws + O_XCNT + xt * 64);
            const unsigned long long t0 = __builtin_amdgcn_s_memrealtime();
            while ((unsigned)__builtin_amdgcn_readfirstlane(ld_agent(xc)) < call * 4u) { if (give_up(tmo, call, t0)) { bad = true; break; } __builtin_amdgcn_s_sleep(8); }
            __builtin_amdgcn_fence(__ATOMIC_ACQUIRE, "agent");
            const size_t eoff = (size_t)((xt / NEX) * 256 + wid * 64) * N + (size_t)colmap(NCOM + rank * NEX + xt % NEX) * 256 + lane * 4;
            for (int r0 = 0; r0 < 64; r0 += 8) {
                f32x4 t[8];
#pragma unroll
                for (int r = 0; r < 8; ++r) t[r] = *(const f32x4*)(p.out[rank] + eoff + (size_t)(r0 + r) * N);
                if (bad) { const float qn = __builtin_nanf("");
#pragma unroll
                    for (int r = 0; r < 8; ++r) t[r] = (f32x4){qn, qn, qn, qn}; }
#pragma unroll
                for (int q = 0; q < NR - 1; ++q) { float* d = p.out[(rank + 1 + q) & 7] + eoff;
#pragma unroll
                    for (int r = 0; r < 8; ++r) __builtin_nontemporal_store(t[r], (f32x4*)(d + (size_t)(r0 + r) * N)); }
            }
        }
        WAIT_V(0);
        __syncthreads();
        if (wid == 0) {
            __builtin_amdgcn_fence(__ATOMIC_RELEASE, "");
            WAIT_V(0);
            if (lane < NR && lane != rank) st_sys((unsigned*)(p.ws[lane & 7] + O_DONE) + rank * 32 + c, call);
        }
        if (c == 0) {
            const unsigned* dn = (const unsigned*)(myws + O_DONE) + tid;
            const unsigned long long t0 = __builtin_amdgcn_s_memrealtime();
            for (;;) {
                const bool ok = (tid >> 5) == rank || (tid & 31) >= NCOMM || ld_sys(dn) >= call;
                if (__all(ok)) break;
                if (give_up(tmo, call, t0)) { if (lane == 0) p.out[rank][0] = __builtin_nanf(""); break; }
                __builtin_amdgcn_s_sleep(8);
            }
            __builtin_amdgcn_fence(__ATOMIC_ACQUIRE, "");
        }
    }
}

struct Unit { int pm, pn; };
__device__ __forceinline__ Unit unit_of(int e, int xcd, int rank) {
    Unit u;
    if (e >= XPOS && e < XPOS + 2 * NEX) {
        const int i = e - XPOS;
        if (NEX == 1) { u.pm = 4 * (xcd >> 1) + 2 * (xcd & 1) + i; u.pn = NCOM + rank; }
        else { u.pm = 4 * (xcd >> 1) + i; u.pn = NCOM + rank * NEX + (xcd & 1); }
    } else { const int r = e < XPOS ? e : e - 2 * NEX; u.pm = 4 * (xcd >> 1) + (r & 3); u.pn = 2 * (r >> 2) + (xcd & 1); }
    return u;
}
typedef float f32x2 __attribute__((ext_vector_type(2)));
__device__ __forceinline__ f32x2 gelu2(f32x2 y) {
    constexpr float k1 = -2.0f * 0.7978845608028654f * 1.4426950408889634f, k3 = k1 * 0.044715f;
    const f32x2 t = y * ((y * y) * k3 + k1);
    f32x2 d; d.x = __builtin_amdgcn_exp2f(t.x); d.y = __builtin_amdgcn_exp2f(t.y);
    d = d + 1.0f;
    f32x2 r; r.x = __builtin_amdgcn_rcpf(d.x); r.y = __builtin_amdgcn_rcpf(d.y);
    return y * r;
}

__device__ __forceinline__ void gemm_role(const Ctx& cx, LAS unsigned char* lds, unsigned call) {
    const int tid = threadIdx.x, wid = __builtin_amdgcn_readfirstlane(tid >> 6), lane = tid & 63, wr = wid >> 1, wc = wid & 1;
    const int rank = cx.rank, xcd = blockIdx.x & 7, w = (blockIdx.x - NCOMM) >> 3;
    char* myws = cx.myws;
    unsigned* tmo = (unsigned*)(myws + O_TMO);
    const unsigned* flags = (const unsigned*)(myws + O_FLAG);
    const __amdgpu_buffer_rsrc_t rsrc = __builtin_amdgcn_make_buffer_rsrc((void*)(myws + CTRL_BYTES + (size_t)(call & 1u) * BUF_BYTES), 0, (int)BUF_BYTES, 0x00020000);
    const char* base = myws + CTRL_BYTES + (size_t)(call & 1u) * BUF_BYTES;
    const int ntiles = (ENT - w + WPX - 1) / WPX, total = ntiles * NS;
    const unsigned voff = (unsigned)(wid * 4096 + lane * 16), ldsw = (unsigned)wid * 4096u;
    const int aoff = wr * 8192 + lane * 16, boff = SLAB + wc * 8192 + lane * 16;
    const int mode = cx.mode;
    int P = (mode & 1) ? total : 0;
    int pt = 0;
#define DMA(slot, ga, gb, i) __builtin_amdgcn_raw_ptr_buffer_load_lds(rsrc, (LAS void*)(lds + (slot) * 2 * SLAB + ((i) < 4 ? 0 : SLAB) + ldsw), 16, \
        (int)voff, (int)((i) < 4 ? (ga) : (gb)), ((i) & 3) * 1024, AUX_LD)
#define RDA(slot, m) (*(const LAS h16x8*)(lds + (slot) * 2 * SLAB + aoff + (m) * 1024))
#define RDB(slot, n) (*(const LAS h16x8*)(lds + (slot) * 2 * SLAB + boff + (n) * 1024))
#define RDF(s, nA_, nB_, f) do { if constexpr ((f) < 8) nB_[(f)] = RDB(((s) + 1) & 3, (f)); else nA_[(f) - 8] = RDA(((s) + 1) & 3, (f) - 8); } while (0)
#define STEP(s, cA_, cB_, nA_, nB_, ga, gb) do { \
        __builtin_amdgcn_s_waitcnt(0x4070);   BAR; SCHED; \
        static_for<16>([&](auto G_) { constexpr int g = decltype(G_)::value; \
            mfma4<4 * g>(cA_[g >> 1], cB_[(g & 1) * 4], cB_[(g & 1) * 4 + 1], cB_[(g & 1) * 4 + 2], cB_[(g & 1) * 4 + 3]); \
            SCHED; \
            if constexpr (SCHEDV == 0) { \
                if constexpr (g < 8) { RDF(s, nA_, nB_, 2 * g); RDF(s, nA_, nB_, 2 * g + 1); } else { DMA(s, ga, gb, g - 8); } \
            } else if constexpr (SCHEDV == 1) { \
                if constexpr (g < 2) { RDF(s, nA_, nB_, 2 * g); RDF(s, nA_, nB_, 2 * g + 1); } else if constexpr (g < 14) { RDF(s, nA_, nB_, g + 2); } \
                if constexpr (g & 1) { DMA(s, ga, gb, g >> 1); } \
            } else { \
                if constexpr ((g & 1) == 0 && g < 12) { RDF(s, nA_, nB_, g); RDF(s, nA_, nB_, g + 1); } \
                if constexpr (g == 12) { RDF(s, nA_, nB_, 12); RDF(s, nA_, nB_, 13); RDF(s, nA_, nB_, 14); RDF(s, nA_, nB_, 15); } \
                if constexpr (g & 1) { DMA(s, ga, gb, g >> 1); } \
            } \
            SCHED; \
        }); } while (0)
#define DMA8(s, ga, gb) do { DMA(s, ga, gb, 0); DMA(s, ga, gb, 1); DMA(s, ga, gb, 2); DMA(s, ga, gb, 3); DMA(s, ga, gb, 4); DMA(s, ga, gb, 5); DMA(s, ga, gb, 6); DMA(s, ga, gb, 7); } while (0)
    auto wait_pos = [&](int need, int ui) {
        if (need > total) need = total;
        if (P >= need) return;
        if (wid == 0) {
            const unsigned long long t0 = __builtin_amdgcn_s_memrealtime();
            for (;;) {
                const Unit u = unit_of(w + pt * WPX, xcd, rank);
                const unsigned fa = ld_sys(flags + u.pm * NS + lane), fb = ld_sys(flags + (NPM + u.pn) * NS + lane);
                const unsigned long long mask = __ballot(fa >= call && fb >= call);
                const int cnt = mask == ~0ull ? 64 : __builtin_ctzll(~mask);
                P = pt * NS + cnt;
                if (cnt == 64 && pt + 1 < ntiles && pt <= ui) { ++pt; continue; }
                if (P >= need) break;
                if (give_up(tmo, call, t0)) { lds_st(lds, L_BAD, 1u); P = total; pt = ntiles; break; }
                __builtin_amdgcn_s_sleep(4);
            }
            __builtin_amdgcn_fence(__ATOMIC_ACQUIRE, "");
            WAIT_V(0);
            lds_st(lds, L_P, (unsigned)P);
            WAIT_L(0);
        } else {
            for (;;) { P = __builtin_amdgcn_readfirstlane((int)lds_ld(lds, L_P)); if (P >= need) break; __builtin_amdgcn_s_sleep(2); }
        }
    };
    int ui = 0;
    Unit cur = unit_of(w, xcd, rank), nxt = cur;
    static_for<256>([&](auto R_) { acc_zero<decltype(R_)::value>(); });
    h16x8 A0[8], B0[8], A1[8], B1[8];
    const int lm = (mode & 8) ? 0 : -1;
    unsigned cA = (unsigned)(cur.pm & lm) * (unsigned)TILE_BYTES, cB = (unsigned)(NPM + (cur.pn & lm)) * (unsigned)TILE_BYTES;
    wait_pos(2, 0);
    DMA8(0, cA, cB); DMA8(1, cA + SLAB, cB + SLAB); DMA8(2, cA + 2 * SLAB, cB + 2 * SLAB); DMA8(3, cA + 3 * SLAB, cB + 3 * SLAB);
    WAIT_V(24); BAR;
#pragma unroll
    for (int i = 0; i < 8; ++i) { A0[i] = RDA(0, i); B0[i] = RDB(0, i); }
    for (;;) {
        const bool has_next = ui + 1 < ntiles;
        if (has_next) nxt = unit_of(w + (ui + 1) * WPX, xcd, rank);
        const unsigned nA = (unsigned)(nxt.pm & lm) * (unsigned)TILE_BYTES, nB = (unsigned)(NPM + (nxt.pn & lm)) * (unsigned)TILE_BYTES;
        for (int t = 0; t < NS; t += 2) {
            const bool last = (t == NS - 2);
            const unsigned a2 = last ? nA : cA + (unsigned)(t + 2) * BLK, b2 = last ? nB : cB + (unsigned)(t + 2) * BLK;
            const int gpos = ui * NS + t;
            wait_pos(gpos + 4, ui);
            const bool apoll = (wid == 0) && P < total && P < gpos + 28 && pt < ntiles;
            if (apoll) {
                const Unit u = unit_of(w + pt * WPX, xcd, rank);
                __builtin_amdgcn_global_load_lds(flags + u.pm * NS + lane, (LAS unsigned*)(lds + L_FLG), 4, 0, AUX_SYS);
                __builtin_amdgcn_global_load_lds(flags + (NPM + u.pn) * NS + lane, (LAS unsigned*)(lds + L_FLG + 256), 4, 0, AUX_SYS);
            }
            STEP(0, A0, B0, A1, B1, a2, b2);
            STEP(1, A1, B1, A0, B0, a2 + SLAB, b2 + SLAB);
            STEP(2, A0, B0, A1, B1, a2 + 2 * SLAB, b2 + 2 * SLAB);
            if (apoll) {
                const unsigned fa = lds_ld(lds, L_FLG + lane * 4), fb = lds_ld(lds, L_FLG + 256 + lane * 4);
                const unsigned long long mask = __ballot(fa >= call && fb >= call);
                const int cnt = mask == ~0ull ? 64 : __builtin_ctzll(~mask);
                const int newP = pt * NS + cnt;
                if (cnt == 64 && pt + 1 < ntiles && pt <= ui) ++pt;
                if (newP > P) { P = newP; lds_st(lds, L_P, (unsigned)P); }
            }
            STEP(3, A1, B1, A0, B0, a2 + 3 * SLAB, b2 + 3 * SLAB);
        }
        {
            const float poison = lds_ld(lds, L_BAD) != 0u ? __builtin_nanf("") : -0.0f;
            float* o = cx.myout + (size_t)(cur.pm * 256 + wr * 128 + (lane >> 4) * 4) * N + (size_t)colmap(cur.pn) * 256 + wc * 128 + (lane & 15) * 4;
            asm volatile("s_nop 15\n\ts_nop 15" ::: "memory");
            static_for<64>([&](auto E_) { constexpr int e = decltype(E_)::value, m = e >> 3, r = (e >> 1) & 3, g = e & 1, a0 = 4 * (8 * m + 4 * g) + r;
                const f32x2 lo = gelu2((f32x2){acc_take<a0>(), acc_take<a0 + 4>()}) + poison, hi = gelu2((f32x2){acc_take<a0 + 8>(), acc_take<a0 + 12>()}) + poison;
                const f32x4 v = (f32x4){lo.x, lo.y, hi.x, hi.y};
                *(f32x4*)(o + (size_t)(m * 16 + r) * N + g * 64) = v;
            });
            if (cur.pn >= NCOM) {
                WAIT_V(0);
                __builtin_amdgcn_fence(__ATOMIC_RELEASE, "agent");
                WAIT_V(0);
                if (lane == 0) __hip_atomic_fetch_add((unsigned*)(myws + O_XCNT + (cur.pm * NEX + cur.pn - NCOM - rank * NEX) * 64), 1u, __ATOMIC_RELAXED, __HIP_MEMORY_SCOPE_AGENT);
            }
        }
        if (!has_next) break;
        cur = nxt; cA = nA; cB = nB; ++ui;
    }
    WAIT_V(0);
}

__global__ __launch_bounds__(256, 1) void fused_kernel(Params p) {
    extern __shared__ __attribute__((aligned(16))) unsigned char smem[];
    LAS unsigned char* lds = (LAS unsigned char*)smem;
    Ctx cx; cx.myws = p.myws; cx.myout = p.myout; cx.rank = p.rank; cx.mode = p.mode;
    asm volatile("" : "+s"(cx.myws), "+s"(cx.myout), "+s"(cx.rank), "+s"(cx.mode));
    if (threadIdx.x < 32) {
        unsigned v = 0;
        if (threadIdx.x == 0) v = __hip_atomic_fetch_add((unsigned*)(cx.myws + O_TICKET), 1u, __ATOMIC_RELAXED, __HIP_MEMORY_SCOPE_AGENT) / (unsigned)GRID + 1u;
        lds_st(lds, L_CALL + threadIdx.x * 4, v);
    }
    __syncthreads();
    const unsigned call = __builtin_amdgcn_readfirstlane(lds_ld(lds, L_CALL));
    if (blockIdx.x < NCOMM) comm_role(p, cx, lds, call);
    else if (!(cx.mode & 4)) gemm_role(cx, lds, call);
}
}

#define FK_MODE 0

extern "C" void kernel_launch(int rank, int nranks, ncclComm_t comm, hipStream_t stream,
                              void* const* d_in, const int* in_sizes, int n_in,
                              void* d_out, int out_size, const int* in_dtypes, int out_dtype,
                              void* const* d_out_all, void* const* d_ws_all, size_t ws_bytes) {
    using namespace fk;
    if (nranks != fk::NR || ws_bytes < WS_NEED || (unsigned)rank >= (unsigned)fk::NR) { fprintf(stderr, "unexpected configuration\n"); return; }
    static bool attr_set = false;
    if (!attr_set) { HIP_CHECK(hipFuncSetAttribute((const void*)fused_kernel, hipFuncAttributeMaxDynamicSharedMemorySize, LDS_BYTES)); attr_set = true; }
    Params p;
    p.x = (const float*)d_in[0]; p.w = (const float*)d_in[1];
    for (int q = 0; q < fk::NR; ++q) { p.out[q] = (float*)d_out_all[q]; p.ws[q] = (char*)d_ws_all[q]; }
    p.out[rank] = (float*)d_out;
    p.myws = p.ws[rank]; p.myout = (float*)d_out; p.rank = rank; p.mode = FK_MODE;
    fused_kernel<<<GRID, 256, LDS_BYTES, stream>>>(p);
    HIP_CHECK(hipGetLastError());
}
```

```cpp
#include <hip/hip_runtime.h>
#include <rccl/rccl.h>
#include <cstdio>
#include <cstdint>
#include <type_traits>

#define HIP_CHECK(expr) do { hipError_t _e = (expr); if (_e != hipSuccess) { \
    fprintf(stderr, "HIP error %s at %s:%d\n", hipGetErrorString(_e), __FILE__, __LINE__); \
    return; } } while (0)

namespace fk {
typedef _Float16 h16x8 __attribute__((ext_vector_type(8)));
typedef float f32x4 __attribute__((ext_vector_type(4)));
typedef unsigned u32x4 __attribute__((ext_vector_type(4)));
#define LAS __attribute__((address_space(3)))
#define GAS __attribute__((address_space(1)))

constexpr int M = 4096, N = 32768, NR = 8, KL = 512;
constexpr int NPM = 16, NPN = 128, NS = 64;
#define NCOMM_ 24
#define NEX_ 3
constexpr int NCOMM = NCOMM_, GRID = 256, WPX = (GRID - NCOMM) / 8;
constexpr int NEX = NEX_;
constexpr int NCOM = NPN - NR * NEX;
constexpr int ENT = 2 * NCOM + 2 * NEX;
constexpr int NGRP = (ENT + WPX - 1) / WPX;
constexpr int R0 = ENT - (NGRP - 1) * WPX;
constexpr int XGRP = 3, XPOS = (R0 + 2 * WPX + 3) / 4 * 4 + 4;
constexpr int SPLIT = (R0 & 3) == 2 ? R0 >> 2 : -1;
#define NHELP_ 0
#define SELF_SLEEP 0
#define STORD 1
#define ROT 1
#define HELP_TH 176
constexpr int NHELP = NHELP_;
constexpr int NSELF = (WPX - R0) * 8 - NHELP;
static_assert(NPM * NEX * 64 <= 3072 && NCOMM <= 32, "control block layout");
static_assert(NSELF > 0 && XPOS + 2 * NEX <= R0 + 3 * WPX, "schedule shape");
constexpr unsigned BLK = 32768, SLAB = 16384;
#define TPAD 36864
constexpr size_t TILE_BYTES = (size_t)NS * BLK + TPAD;
constexpr size_t BUF_BYTES = (size_t)(NPM + NPN) * TILE_BYTES;
constexpr size_t CTRL_BYTES = 1u << 20;
constexpr size_t WS_NEED = CTRL_BYTES + 2 * BUF_BYTES;
constexpr unsigned O_TICKET = 0, O_TMO = 128, O_BCNT = 256, O_XCNT = 1024  , O_DONE = 4096  ,
                   O_FLAG = 8192  , O_DBG = 65536;
constexpr unsigned long long DEADLINE = 50000000ull;
constexpr int STAGE_BYTES = 4 * 2 * SLAB;
constexpr int L_CALL = STAGE_BYTES, L_P = STAGE_BYTES + 4, L_BAD = STAGE_BYTES + 8, L_TKT = STAGE_BYTES + 16  , L_FLG = STAGE_BYTES + 256  ;
constexpr int LDS_BYTES = STAGE_BYTES + 1024;
constexpr int AUX_SYS = 17, AUX_SC1 = 16;
#define SCHEDV 2
#define DMAV 1
#define AUX_ST 17
#define OUT_MOD ""
#define AUX_LD 16

__host__ __device__ __forceinline__ constexpr int colmap(int pn) {
    return pn;
}
struct Params { const float* x; const float* w; float* out[NR]; char* ws[NR]; char* myws; float* myout; int rank; int mode; };
struct Ctx { GAS char* myws; GAS float* myout; int rank, mode; };

__device__ __forceinline__ unsigned ld_sys(const GAS unsigned* p) { return __hip_atomic_load(p, __ATOMIC_RELAXED, __HIP_MEMORY_SCOPE_SYSTEM); }
__device__ __forceinline__ unsigned ld_agent(const GAS unsigned* p) { return __hip_atomic_load(p, __ATOMIC_RELAXED, __HIP_MEMORY_SCOPE_AGENT); }
__device__ __forceinline__ void st_sys(GAS unsigned* p, unsigned v) { __hip_atomic_store(p, v, __ATOMIC_RELAXED, __HIP_MEMORY_SCOPE_SYSTEM); }
__device__ __forceinline__ void st_agent(GAS unsigned* p, unsigned v) { __hip_atomic_store(p, v, __ATOMIC_RELAXED, __HIP_MEMORY_SCOPE_AGENT); }
__device__ __forceinline__ unsigned lds_ld(LAS unsigned char* lds, int off) { return *(volatile LAS unsigned*)(lds + off); }
__device__ __forceinline__ void lds_st(LAS unsigned char* lds, int off, unsigned v) { *(volatile LAS unsigned*)(lds + off) = v; }
template <class T> __device__ __forceinline__ T* uni_ptr(T* q) {
    const unsigned long long v = (unsigned long long)q;
    return (T*)(((unsigned long long)(unsigned)__builtin_amdgcn_readfirstlane((int)(v >> 32)) << 32) | (unsigned)__builtin_amdgcn_readfirstlane((int)v));
}
#define WAIT_V_(n) asm volatile("s_nop 1\n\ts_waitcnt vmcnt(" #n ")" ::: "memory")
#define WAIT_V(n) WAIT_V_(n)
#define CW1 63
#define WAIT_L(n) asm volatile("s_waitcnt lgkmcnt(" #n ")" ::: "memory")
#define BAR __builtin_amdgcn_s_barrier()
#define SCHED __builtin_amdgcn_sched_barrier(0)
#define ACLOB "a0","a1","a2","a3","a4","a5","a6","a7","a8","a9","a10","a11","a12","a13","a14","a15","a16","a17","a18","a19","a20","a21","a22","a23","a24","a25","a26","a27","a28","a29","a30","a31","a32","a33","a34","a35","a36","a37","a38","a39","a40","a41","a42","a43","a44","a45","a46","a47","a48","a49","a50","a51","a52","a53","a54","a55","a56","a57","a58","a59","a60","a61","a62","a63","a64","a65","a66","a67","a68","a69","a70","a71","a72","a73","a74","a75","a76","a77","a78","a79","a80","a81","a82","a83","a84","a85","a86","a87","a88","a89","a90","a91","a92","a93","a94","a95","a96","a97","a98","a99","a100","a101","a102","a103","a104","a105","a106","a107","a108","a109","a110","a111","a112","a113","a114","a115","a116","a117","a118","a119","a120","a121","a122","a123","a124","a125","a126","a127","a128","a129","a130","a131","a132","a133","a134","a135","a136","a137","a138","a139","a140","a141","a142","a143","a144","a145","a146","a147","a148","a149","a150","a151","a152","a153","a154","a155","a156","a157","a158","a159","a160","a161","a162","a163","a164","a165","a166","a167","a168","a169","a170","a171","a172","a173","a174","a175","a176","a177","a178","a179","a180","a181","a182","a183","a184","a185","a186","a187","a188","a189","a190","a191","a192","a193","a194","a195","a196","a197","a198","a199","a200","a201","a202","a203","a204","a205","a206","a207","a208","a209","a210","a211","a212","a213","a214","a215","a216","a217","a218","a219","a220","a221","a222","a223","a224","a225","a226","a227","a228","a229","a230","a231","a232","a233","a234","a235","a236","a237","a238","a239","a240","a241","a242","a243","a244","a245","a246","a247","a248","a249","a250","a251","a252","a253","a254","a255"
template <int I, bool ZERO> __device__ __forceinline__ void mfma4(const h16x8& a, const h16x8& b0, const h16x8& b1, const h16x8& b2, const h16x8& b3) {
    if constexpr (ZERO)
    asm volatile("v_mfma_f32_16x16x32_f16 a[%c5:%c6], %0, %1, 0\n\tv_mfma_f32_16x16x32_f16 a[%c7:%c8], %0, %2, 0\n\t"
                 "v_mfma_f32_16x16x32_f16 a[%c9:%c10], %0, %3, 0\n\tv_mfma_f32_16x16x32_f16 a[%c11:%c12], %0, %4, 0"
                 :: "v"(a), "v"(b0), "v"(b1), "v"(b2), "v"(b3), "i"(4 * I), "i"(4 * I + 3), "i"(4 * I + 4), "i"(4 * I + 7), "i"(4 * I + 8), "i"(4 * I + 11), "i"(4 * I + 12), "i"(4 * I + 15) : ACLOB);
    else
    asm volatile("v_mfma_f32_16x16x32_f16 a[%c5:%c6], %0, %1, a[%c5:%c6]\n\tv_mfma_f32_16x16x32_f16 a[%c7:%c8], %0, %2, a[%c7:%c8]\n\t"
                 "v_mfma_f32_16x16x32_f16 a[%c9:%c10], %0, %3, a[%c9:%c10]\n\tv_mfma_f32_16x16x32_f16 a[%c11:%c12], %0, %4, a[%c11:%c12]"
                 :: "v"(a), "v"(b0), "v"(b1), "v"(b2), "v"(b3), "i"(4 * I), "i"(4 * I + 3), "i"(4 * I + 4), "i"(4 * I + 7), "i"(4 * I + 8), "i"(4 * I + 11), "i"(4 * I + 12), "i"(4 * I + 15) : ACLOB);
}
template <int R> __device__ __forceinline__ void acc_read16(float (&v)[16]) {
    asm volatile("v_accvgpr_read_b32 %0, a[%c16+0]\n\tv_accvgpr_read_b32 %1, a[%c16+1]\n\tv_accvgpr_read_b32 %2, a[%c16+2]\n\tv_accvgpr_read_b32 %3, a[%c16+3]\n\tv_accvgpr_read_b32 %4, a[%c16+4]\n\tv_accvgpr_read_b32 %5, a[%c16+5]\n\tv_accvgpr_read_b32 %6, a[%c16+6]\n\tv_accvgpr_read_b32 %7, a[%c16+7]\n\tv_accvgpr_read_b32 %8, a[%c16+8]\n\tv_accvgpr_read_b32 %9, a[%c16+9]\n\tv_accvgpr_read_b32 %10, a[%c16+10]\n\tv_accvgpr_read_b32 %11, a[%c16+11]\n\tv_accvgpr_read_b32 %12, a[%c16+12]\n\tv_accvgpr_read_b32 %13, a[%c16+13]\n\tv_accvgpr_read_b32 %14, a[%c16+14]\n\tv_accvgpr_read_b32 %15, a[%c16+15]" : "=v"(v[0]), "=v"(v[1]), "=v"(v[2]), "=v"(v[3]), "=v"(v[4]), "=v"(v[5]), "=v"(v[6]), "=v"(v[7]), "=v"(v[8]), "=v"(v[9]), "=v"(v[10]), "=v"(v[11]), "=v"(v[12]), "=v"(v[13]), "=v"(v[14]), "=v"(v[15]) : "i"(R) : ACLOB);
}
template <int... Is> struct Seq {};
template <int N_, int... Is> struct MkSeq : MkSeq<N_ - 1, N_ - 1, Is...> {};
template <int... Is> struct MkSeq<0, Is...> { typedef Seq<Is...> type; };
template <class F, int... Is> __device__ __forceinline__ void static_for_impl(F&& f, Seq<Is...>) { (f(std::integral_constant<int, Is>{}), ...); }
template <int N_, class F> __device__ __forceinline__ void static_for(F&& f) { static_for_impl(f, typename MkSeq<N_>::type{}); }

__device__ __forceinline__ bool give_up(GAS unsigned* tmo, unsigned call, unsigned long long t0) {
    const bool g = (unsigned)__builtin_amdgcn_readfirstlane(ld_agent(tmo)) == call || __builtin_amdgcn_s_memrealtime() - t0 > DEADLINE;
    if (g) st_agent(tmo, call);
    return g;
}

constexpr int grp_start_c(int g) {
    if (g <= 0) return 0;
    if (g == 1 && SPLIT >= 0) return 2 * SPLIT + 1;
    int e = R0 + (g - 1) * WPX - 1;
    int r = e < XPOS ? e : (e < XPOS + 2 * NEX ? XPOS - 1 : e - 2 * NEX);
    return 2 * (r / 4 + 1);
}
struct GrpTab { int s[NGRP + 1]; constexpr GrpTab() : s{} { for (int g = 0; g <= NGRP; ++g) s[g] = grp_start_c(g); } };
constexpr GrpTab GT{};
constexpr int NBLK = 8 * (NPM + NPN);
struct Blk { int T, j, dest; };
__device__ __forceinline__ Blk decode(int b) {
    Blk r; r.T = 0; r.j = 0; r.dest = -1;
#pragma unroll
    for (int g = 0; g < NGRP; ++g) {
        const int s = GT.s[g], nb = GT.s[g + 1] - s, mem = nb + (g == 0 ? NPM : 0) + (g == XGRP ? NR * NEX : 0);
        if (b >= 0 && b < 8 * mem) {
            const int j = b / mem, m = b - j * mem; r.j = j;
            if (g == 0) r.T = m < 2 * nb ? ((m & 1) ? NPM + (m >> 1) : (m >> 1)) : m - nb;
            else if (m < nb) r.T = NPM + s + m;
            else { r.dest = (m - nb) / NEX; r.T = NPM + NCOM + (m - nb); }
        }
        b -= 8 * mem;
    }
    return r;
}
static_assert(GT.s[1] <= NPM, "group 0 interleave assumes at most 16 column tiles");
static_assert(GT.s[NGRP] == NCOM, "groups cover every common column tile");
static_assert(XGRP > 0, "exclusive tiles not in group 0");
template <int SET> __device__ __forceinline__ void slab_load(const Params& p, const Blk& k, int kf, int wid, int lane) {
    const bool isx = k.T < NPM;
    const float* src = isx ? p.x + (size_t)(k.T * 256 + wid * 64 + (lane & 15)) * KL + k.j * 64 + kf * 32 + (lane >> 4) * 8
                           : p.w + (size_t)(k.j * 64 + kf * 32 + (lane >> 4) * 8) * N + (size_t)colmap(k.T - NPM) * 256 + wid * 64 + (lane & 15) * 4;
    static_for<8>([&](auto R_) { constexpr int r = decltype(R_)::value;
        const size_t o = isx ? (size_t)((r >> 1) * 16 * KL + (r & 1) * 4) : (size_t)r * N;
        const float* q = src + o;
        asm volatile("global_load_dwordx4 a[%c1:%c2], %0, off nt" :: "v"(q), "i"(SET * 32 + r * 4), "i"(SET * 32 + r * 4 + 3) : "memory", ACLOB);
    });
}
template <int SET> __device__ __forceinline__ void slab_fetch(f32x4 (&v)[8]) {
    static_for<8>([&](auto R_) { constexpr int r = decltype(R_)::value; float x0, x1, x2, x3;
        asm volatile("v_accvgpr_read_b32 %0, a[%c4]\n\tv_accvgpr_read_b32 %1, a[%c5]\n\tv_accvgpr_read_b32 %2, a[%c6]\n\tv_accvgpr_read_b32 %3, a[%c7]"
                     : "=v"(x0), "=v"(x1), "=v"(x2), "=v"(x3) : "i"(SET * 32 + r * 4), "i"(SET * 32 + r * 4 + 1), "i"(SET * 32 + r * 4 + 2), "i"(SET * 32 + r * 4 + 3) : "memory", ACLOB);
        v[r] = (f32x4){x0, x1, x2, x3};
    });
}
__device__ __forceinline__ void slab_convert(const Blk& k, const f32x4 (&v)[8], u32x4 (&o)[4]) {
    if (k.T < NPM) {
#pragma unroll
        for (int i = 0; i < 4; ++i) { h16x8 h;
#pragma unroll
            for (int e = 0; e < 4; ++e) { h[e] = (_Float16)v[2 * i][e]; h[4 + e] = (_Float16)v[2 * i + 1][e]; }
            o[i] = __builtin_bit_cast(u32x4, h); }
    } else {
#pragma unroll
        for (int i = 0; i < 4; ++i) { h16x8 h;
#pragma unroll
            for (int r = 0; r < 8; ++r) h[r] = (_Float16)v[r][i];
            o[i] = __builtin_bit_cast(u32x4, h); }
    }
}

__device__ __forceinline__ void stream_role(const Params& p, const Ctx& cx, LAS unsigned char* lds, unsigned call, int c, int th) {
    const int tid = threadIdx.x, wid = __builtin_amdgcn_readfirstlane(tid >> 6), lane = tid & 63, rank = cx.rank;
    GAS char* myws = cx.myws;
    const size_t bufoff = CTRL_BYTES + (size_t)(call & 1u) * BUF_BYTES;
    __amdgpu_buffer_rsrc_t rs[NR - 1];
#pragma unroll
    for (int q = 0; q < NR - 1; ++q) rs[q] = __builtin_amdgcn_make_buffer_rsrc((void*)(p.ws[(rank + 1 + (q + ROT * (c * 4 + wid)) % 7) & 7] + bufoff), 0, (int)BUF_BYTES, 0x00020000);
    GAS unsigned* flag_base = (GAS unsigned*)(p.ws[lane & 7] + O_FLAG);
    const int voff = wid * 4096 + lane * 16;
    auto publish = [&](const Blk& k) {
        if (wid == 0 && lane < NR && (k.dest < 0 ? lane != rank : k.dest == lane)) st_sys(flag_base + k.T * NS + k.j * 8 + rank, call);
    };
    {
        auto store_slab = [&](const Blk& k, int kf, const u32x4 (&o)[4]) {
            const unsigned base = (unsigned)k.T * (unsigned)TILE_BYTES + (unsigned)(k.j * 8 + rank) * BLK + (unsigned)kf * SLAB;
            if (k.dest < 0) {
#pragma unroll
                for (int i = 0; i < 4; ++i)
#pragma unroll
                    for (int q = 0; q < NR - 1; ++q) __builtin_amdgcn_raw_buffer_store_b128(o[i], rs[q], voff + i * 1024, (int)base, AUX_ST);
            } else {
                const __amdgpu_buffer_rsrc_t r1 = __builtin_amdgcn_make_buffer_rsrc((void*)(p.ws[k.dest & 7] + bufoff), 0, (int)BUF_BYTES, 0x00020000);
#pragma unroll
                for (int i = 0; i < 4; ++i) __builtin_amdgcn_raw_buffer_store_b128(o[i], r1, voff + i * 1024, (int)base, AUX_ST);
            }
        };
        u32x4 o[4]; f32x4 v[8];
        const GAS unsigned* bcnt = (const GAS unsigned*)(myws + O_BCNT + (call & 1u) * 128u);
        if (c == 0 && wid == 0 && lane == 0) st_agent((GAS unsigned*)(myws + O_BCNT + (~call & 1u) * 128u), 0u);
        asm volatile("v_accvgpr_write_b32 a[65], 1" ::: ACLOB);
        int bc = c, bn = c + NCOMM + NHELP;
        Blk cur = decode(bc), prev = cur;
        slab_load<0>(p, cur, 0, wid, lane); slab_load<1>(p, cur, 1, wid, lane);
        WAIT_V(8); slab_fetch<0>(v); slab_convert(cur, v, o);
        for (int i = 0; bc < NBLK; ++i) {
            const Blk nxt = decode(bn < NBLK ? bn : bc);
            if (wid == 0 && lane == 0 && bn < th) asm volatile("global_atomic_add a[64], %0, a[65], off sc0" :: "v"(bcnt) : "memory", ACLOB);
            slab_load<0>(p, nxt, 0, wid, lane);
            store_slab(cur, 0, o);
            if (cur.dest < 0 && prev.dest < 0 && i > 0) WAIT_V(CW1); else WAIT_V(12);
            slab_fetch<1>(v);
            slab_convert(cur, v, o);
            slab_load<1>(p, nxt, 1, wid, lane);
            store_slab(cur, 1, o);
            if (cur.dest < 0) WAIT_V(CW1); else WAIT_V(16);
            if (wid == 0 && lane == 0) { unsigned tk; asm volatile("v_accvgpr_read_b32 %0, a[64]" : "=v"(tk) :: "memory", ACLOB); lds_st(lds, L_TKT + (i & 1) * 4, bn < th ? tk + 2u * (unsigned)(NCOMM + NHELP) : (unsigned)NBLK); }
            WAIT_L(0); BAR;
            if (i > 0) publish(prev);
            slab_fetch<0>(v);
            slab_convert(nxt, v, o);
            prev = cur; cur = nxt; bc = bn; bn = __builtin_amdgcn_readfirstlane((int)lds_ld(lds, L_TKT + (i & 1) * 4));
        }
        WAIT_V(0); BAR;
        publish(prev);
    }
}

__device__ __forceinline__ void push_role(const Params& p, const Ctx& cx, unsigned call) {
    const int tid = threadIdx.x, wid = __builtin_amdgcn_readfirstlane(tid >> 6), lane = tid & 63, c = blockIdx.x, rank = cx.rank;
    GAS char* myws = cx.myws;
    GAS unsigned* tmo = (GAS unsigned*)(myws + O_TMO);
    bool bad = false;
    __amdgpu_buffer_rsrc_t ro[NR - 1];
#pragma unroll
    for (int q = 0; q < NR - 1; ++q) ro[q] = __builtin_amdgcn_make_buffer_rsrc((void*)p.out[(rank + 1 + (q + ROT * (c * 4 + wid)) % 7) & 7], 0, (int)((size_t)M * N * 4), 0x00020000);
    if (!(p.mode & 6)) {
        for (int xu = c; xu < NPM * NEX * 8; xu += NCOMM) {
            const int xt = xu >> 3;
            const GAS unsigned* xc = (const GAS unsigned*)(myws + O_XCNT + xt * 64);
            const unsigned long long t0 = __builtin_amdgcn_s_memrealtime();
            while ((unsigned)__builtin_amdgcn_readfirstlane(ld_agent(xc)) < call * 4u) { if (give_up(tmo, call, t0)) { bad = true; break; } __builtin_amdgcn_s_sleep(8); }
            const size_t eoff = (size_t)((xt / NEX) * 256 + (xu & 7) * 32 + wid * 8) * N + (size_t)colmap(NCOM + rank * NEX + xt % NEX) * 256 + lane * 4;
            f32x4 t[8];
#pragma unroll
            for (int r = 0; r < 8; ++r) t[r] = __builtin_nontemporal_load((const GAS f32x4*)(cx.myout + eoff + (size_t)r * N));
            if (bad) { const float qn = __builtin_nanf("");
#pragma unroll
                for (int r = 0; r < 8; ++r) t[r] = (f32x4){qn, qn, qn, qn}; }
#pragma unroll
            for (int r = 0; r < 8; ++r)
#pragma unroll
                for (int q = 0; q < NR - 1; ++q) __builtin_amdgcn_raw_buffer_store_b128(__builtin_bit_cast(u32x4, t[r]), ro[q], (int)(eoff * 4), r * N * 4, AUX_ST);
            asm volatile("s_nop 1" ::: "memory");
        }
        WAIT_V(0);
        __syncthreads();
        if (wid == 0) {
            __builtin_amdgcn_fence(__ATOMIC_RELEASE, "");
            WAIT_V(0);
            if (lane < NR && lane != rank) st_sys((GAS unsigned*)(p.ws[lane & 7] + O_DONE) + rank * 32 + c, call);
        }
        if (c == 0) {
            const GAS unsigned* dn = (const GAS unsigned*)(myws + O_DONE) + tid;
            const unsigned long long t0 = __builtin_amdgcn_s_memrealtime();
            for (;;) {
                const bool ok = (tid >> 5) == rank || (tid & 31) >= NCOMM || ld_sys(dn) >= call;
                if (__all(ok)) break;
                if (give_up(tmo, call, t0)) { if (lane == 0) cx.myout[0] = __builtin_nanf(""); break; }
                __builtin_amdgcn_s_sleep(8);
            }
            __builtin_amdgcn_fence(__ATOMIC_ACQUIRE, "");
        }
    }
}

struct Unit { int pm, pn; };
__device__ __forceinline__ Unit unit_of(int e, int xcd, int rank) {
    Unit u;
    if (e >= XPOS && e < XPOS + 2 * NEX) {
        const int i = e - XPOS;
        if ((NEX & 1) && i >= 2 * NEX - 2) { u.pm = 2 * xcd + (i & 1); u.pn = NCOM + rank * NEX + NEX - 1; }
        else { u.pm = 4 * (xcd >> 1) + (i & 3); u.pn = NCOM + rank * NEX + 2 * (i >> 2) + (xcd & 1); }
    } else if ((e >> 2) == SPLIT) { u.pm = 2 * xcd + (e & 1); u.pn = 2 * SPLIT + ((e >> 1) & 1); }
    else { const int r = e < XPOS ? e : e - 2 * NEX; u.pm = 4 * (xcd >> 1) + (r & 3); u.pn = 2 * (r >> 2) + (xcd & 1); }
    return u;
}
typedef float f32x2 __attribute__((ext_vector_type(2)));
__device__ __forceinline__ f32x2 gelu2(f32x2 y) {
    constexpr float k1 = -2.0f * 0.7978845608028654f * 1.4426950408889634f, k3 = k1 * 0.044715f;
    const f32x2 t = y * ((y * y) * k3 + k1);
    f32x2 d; d.x = __builtin_amdgcn_exp2f(t.x); d.y = __builtin_amdgcn_exp2f(t.y);
    d = d + 1.0f;
    f32x2 r; r.x = __builtin_amdgcn_rcpf(d.x); r.y = __builtin_amdgcn_rcpf(d.y);
    return y * r;
}

template <int OFF> __device__ __forceinline__ void out_store(const f32x4& v, int voff, const __amdgpu_buffer_rsrc_t& rs, unsigned soff) {
    asm volatile("buffer_store_dwordx4 %0, %1, %2, %3 offen offset:%c4" OUT_MOD "\n\ts_nop 1" :: "v"(v), "v"(voff), "s"(rs), "s"(soff), "i"(OFF) : "memory");
}

__device__ __forceinline__ void self_role(const Params& p, const Ctx& cx, unsigned call, int ib) {
    const int tid = threadIdx.x, wid = __builtin_amdgcn_readfirstlane(tid >> 6), lane = tid & 63, rank = cx.rank;
    const __amdgpu_buffer_rsrc_t rs = __builtin_amdgcn_make_buffer_rsrc((void*)(cx.myws + CTRL_BYTES + (size_t)(call & 1u) * BUF_BYTES), 0, (int)BUF_BYTES, 0x00020000);
    GAS unsigned* flags = (GAS unsigned*)(cx.myws + O_FLAG);
    const int voff = wid * 4096 + lane * 16;
    Blk cur = decode(ib), prev = cur; prev.dest = 0;
    auto publish = [&]() { if (wid == 0 && lane == 0 && prev.dest < 0) st_sys(flags + prev.T * NS + prev.j * 8 + rank, call); };
    auto trip = [&](auto PH, int b) {
        constexpr int ph = decltype(PH)::value;
        const Blk nxt = decode(b + NSELF < NBLK ? b + NSELF : b);
        slab_load<2 * (ph ^ 1)>(p, nxt, 0, wid, lane); slab_load<2 * (ph ^ 1) + 1>(p, nxt, 1, wid, lane);
        WAIT_V(16); BAR;
        publish();
        if (cur.dest < 0) {
            const unsigned base = (unsigned)cur.T * (unsigned)TILE_BYTES + (unsigned)(cur.j * 8 + rank) * BLK;
            u32x4 o[4], o2[4]; f32x4 v[8];
            slab_fetch<2 * ph>(v); slab_convert(cur, v, o);
            slab_fetch<2 * ph + 1>(v); slab_convert(cur, v, o2);
#pragma unroll
            for (int i = 0; i < 4; ++i) __builtin_amdgcn_raw_buffer_store_b128(o[i], rs, voff + i * 1024, (int)base, AUX_SYS);
#pragma unroll
            for (int i = 0; i < 4; ++i) __builtin_amdgcn_raw_buffer_store_b128(o2[i], rs, voff + i * 1024, (int)(base + SLAB), AUX_SYS);
            asm volatile("s_nop 1" ::: "memory"); SCHED;
        }
        prev = cur; cur = nxt;
#pragma unroll
        for (int z = 0; z < SELF_SLEEP; ++z) __builtin_amdgcn_s_sleep(64);
    };
    slab_load<0>(p, cur, 0, wid, lane); slab_load<1>(p, cur, 1, wid, lane);
    for (int b = ib; b < NBLK; b += 2 * NSELF) {
        trip(std::integral_constant<int, 0>{}, b);
        if (b + NSELF < NBLK) trip(std::integral_constant<int, 1>{}, b + NSELF);
    }
    WAIT_V(0); BAR;
    publish();
}

__device__ __forceinline__ void gemm_role(const Ctx& cx, LAS unsigned char* lds, unsigned call) {
    const int tid = threadIdx.x, wid = __builtin_amdgcn_readfirstlane(tid >> 6), lane = tid & 63, wr = wid >> 1, wc = wid & 1;
    const int rank = cx.rank, xcd = blockIdx.x & 7, w = (blockIdx.x - NCOMM) >> 3;
    GAS char* myws = cx.myws;
    GAS unsigned* tmo = (GAS unsigned*)(myws + O_TMO);
    const GAS unsigned* flags = (const GAS unsigned*)(myws + O_FLAG);
    const __amdgpu_buffer_rsrc_t rsrc = __builtin_amdgcn_make_buffer_rsrc((void*)(myws + CTRL_BYTES + (size_t)(call & 1u) * BUF_BYTES), 0, (int)BUF_BYTES, 0x00020000);
    const __amdgpu_buffer_rsrc_t orsrc = __builtin_amdgcn_make_buffer_rsrc((void*)cx.myout, 0, (int)((size_t)M * N * 4), 0x00020000);
    const int ovoff = (lane >> 4) * 4 * N * 4 + (lane & 15) * 16;
    const int ntiles = w < R0 ? NGRP : NGRP - 1, total = ntiles * NS;
    const int e1 = w < R0 ? R0 - WPX + w : R0 + w;
    auto tile_of = [&](int i) { const int e = e1 + i * WPX; return unit_of(e < R0 ? w : e, xcd, rank); };
    const unsigned voff = (unsigned)(wid * 4096 + lane * 16), ldsw = (unsigned)wid * 4096u;
    const int aoff = wr * 8192 + lane * 16, boff = SLAB + wc * 8192 + lane * 16;
    const int mode = cx.mode;
    int P = (mode & 1) ? total : 0;
    int pt = 0;
#define DMA(slot, ga, gb, i) __builtin_amdgcn_raw_ptr_buffer_load_lds(rsrc, (LAS void*)(lds + (slot) * 2 * SLAB + ((i) < 4 ? 0 : SLAB) + ldsw), 16, \
        (int)voff, (int)((i) < 4 ? (ga) : (gb)), ((i) & 3) * 1024, AUX_LD)
#define RDA(slot, m) (*(const LAS h16x8*)(lds + (slot) * 2 * SLAB + aoff + (m) * 1024))
#define RDB(slot, n) (*(const LAS h16x8*)(lds + (slot) * 2 * SLAB + boff + (n) * 1024))
#define RDF(s, nA_, nB_, f) do { if constexpr ((f) < 8) nB_[(f)] = RDB(((s) + 1) & 3, (f)); else nA_[(f) - 8] = RDA(((s) + 1) & 3, (f) - 8); } while (0)
#define STEP(s, Z, cA_, cB_, nA_, nB_, ga, gb) do { \
        __builtin_amdgcn_s_waitcnt(0x4070);   BAR; SCHED; \
        static_for<16>([&](auto G_) { constexpr int g = decltype(G_)::value; \
            mfma4<4 * g, Z>(cA_[g >> 1], cB_[(g & 1) * 4], cB_[(g & 1) * 4 + 1], cB_[(g & 1) * 4 + 2], cB_[(g & 1) * 4 + 3]); \
            SCHED; \
            if constexpr (SCHEDV == 0) { \
                if constexpr (g < 8) { RDF(s, nA_, nB_, 2 * g); RDF(s, nA_, nB_, 2 * g + 1); } else { DMA(s, ga, gb, g - 8); } \
            } else if constexpr (SCHEDV == 1) { \
                if constexpr (g < 2) { RDF(s, nA_, nB_, 2 * g); RDF(s, nA_, nB_, 2 * g + 1); } else if constexpr (g < 14) { RDF(s, nA_, nB_, g + 2); } \
                if constexpr (g & 1) { DMA(s, ga, gb, g >> 1); } \
            } else if constexpr (SCHEDV == 3) { \
                if constexpr (g < 8) { DMA(s, ga, gb, (g + 4) & 7); RDF(s, nA_, nB_, g); } \
                else if constexpr (g < 10) { RDF(s, nA_, nB_, 2 * g - 8); RDF(s, nA_, nB_, 2 * g - 7); } \
                else if constexpr (g < 14) { RDF(s, nA_, nB_, g + 2); } \
            } else if constexpr (SCHEDV == 4) { \
                if constexpr (g < 4) { DMA(s, ga, gb, (2 * g + 4) & 7); DMA(s, ga, gb, (2 * g + 5) & 7); } \
                else if constexpr (g < 10) { RDF(s, nA_, nB_, 2 * g - 8); RDF(s, nA_, nB_, 2 * g - 7); } \
                else if constexpr (g < 14) { RDF(s, nA_, nB_, g + 2); } \
            } else { \
                if constexpr ((g & 1) == 0 && g < 12) { RDF(s, nA_, nB_, g); RDF(s, nA_, nB_, g + 1); } \
                if constexpr (g == 12) { RDF(s, nA_, nB_, 12); RDF(s, nA_, nB_, 13); RDF(s, nA_, nB_, 14); RDF(s, nA_, nB_, 15); } \
                if constexpr (g & 1) { DMA(s, ga, gb, g >> 1); } \
            } \
            SCHED; \
        }); } while (0)
#define DMA8(s, ga, gb) do { DMA(s, ga, gb, 0); DMA(s, ga, gb, 1); DMA(s, ga, gb, 2); DMA(s, ga, gb, 3); DMA(s, ga, gb, 4); DMA(s, ga, gb, 5); DMA(s, ga, gb, 6); DMA(s, ga, gb, 7); } while (0)
    auto wait_pos = [&](int need, int ui) {
        if (need > total) need = total;
        if (P >= need) return;
        if (wid == 0) {
            const unsigned long long t0 = __builtin_amdgcn_s_memrealtime();
            for (;;) {
                const Unit u = tile_of(pt);
                const unsigned fa = ld_sys(flags + u.pm * NS + lane), fb = ld_sys(flags + (NPM + u.pn) * NS + lane);
                const unsigned long long mask = __ballot(fa >= call && fb >= call);
                const int cnt = mask == ~0ull ? 64 : __builtin_ctzll(~mask);
                P = pt * NS + cnt;
                if (cnt == 64 && pt + 1 < ntiles && pt <= ui) { ++pt; continue; }
                if (P >= need) break;
                if (give_up(tmo, call, t0)) { lds_st(lds, L_BAD, 1u); P = total; pt = ntiles; break; }
                __builtin_amdgcn_s_sleep(4);
            }
            lds_st(lds, L_P, (unsigned)P);
            WAIT_L(0);
        } else {
            for (;;) { P = __builtin_amdgcn_readfirstlane((int)lds_ld(lds, L_P)); if (P >= need) break; __builtin_amdgcn_s_sleep(2); }
        }
    };
    int ui = 0;
    Unit cur = tile_of(0), nxt = cur;
    h16x8 A0[8], B0[8], A1[8], B1[8];
    const int lm = (mode & 8) ? 0 : -1;
    unsigned cA = (unsigned)(cur.pm & lm) * (unsigned)TILE_BYTES, cB = (unsigned)(NPM + (cur.pn & lm)) * (unsigned)TILE_BYTES;
    wait_pos(2, 0);
    DMA8(0, cA, cB); DMA8(1, cA + SLAB, cB + SLAB); DMA8(2, cA + 2 * SLAB, cB + 2 * SLAB); DMA8(3, cA + 3 * SLAB, cB + 3 * SLAB);
    WAIT_V(24); BAR;
#pragma unroll
    for (int i = 0; i < 8; ++i) { A0[i] = RDA(0, i); B0[i] = RDB(0, i); }
    for (;;) {
        const bool has_next = ui + 1 < ntiles;
        if (has_next) nxt = tile_of(ui + 1);
        const unsigned nA = (unsigned)(nxt.pm & lm) * (unsigned)TILE_BYTES, nB = (unsigned)(NPM + (nxt.pn & lm)) * (unsigned)TILE_BYTES;
        for (int t = 0; t < NS; t += 2) {
            const bool last = (t == NS - 2);
            const unsigned a2 = last ? nA : cA + (unsigned)(t + 2) * BLK, b2 = last ? nB : cB + (unsigned)(t + 2) * BLK;
            const int gpos = ui * NS + t;
            wait_pos(gpos + 4, ui);
            const bool apoll = (wid == 0) && P < total && P < gpos + 28 && pt < ntiles;
            if (apoll) {
                const Unit u = tile_of(pt);
                __builtin_amdgcn_global_load_lds(flags + u.pm * NS + lane, (LAS unsigned*)(lds + L_FLG), 4, 0, AUX_SYS);
                __builtin_amdgcn_global_load_lds(flags + (NPM + u.pn) * NS + lane, (LAS unsigned*)(lds + L_FLG + 256), 4, 0, AUX_SYS);
            }
            if (t == 0) STEP(0, true, A0, B0, A1, B1, a2, b2); else STEP(0, false, A0, B0, A1, B1, a2, b2);
            STEP(1, false, A1, B1, A0, B0, a2 + SLAB, b2 + SLAB);
            STEP(2, false, A0, B0, A1, B1, a2 + 2 * SLAB, b2 + 2 * SLAB);
            if (apoll) {
                const unsigned fa = lds_ld(lds, L_FLG + lane * 4), fb = lds_ld(lds, L_FLG + 256 + lane * 4);
                const unsigned long long mask = __ballot(fa >= call && fb >= call);
                const int cnt = mask == ~0ull ? 64 : __builtin_ctzll(~mask);
                const int newP = pt * NS + cnt;
                if (cnt == 64 && pt + 1 < ntiles && pt <= ui) ++pt;
                if (newP > P) { P = newP; lds_st(lds, L_P, (unsigned)P); }
            }
            STEP(3, false, A1, B1, A0, B0, a2 + 3 * SLAB, b2 + 3 * SLAB);
        }
        {
            const float poison = lds_ld(lds, L_BAD) != 0u ? __builtin_nanf("") : -0.0f;
            const unsigned obase = ((unsigned)(cur.pm * 256 + wr * 128) * (unsigned)N + (unsigned)colmap(cur.pn) * 256u + (unsigned)wc * 128u) * 4u;
            asm volatile("s_nop 15\n\ts_nop 15" ::: "memory");
            static_for<16>([&](auto E_) { constexpr int e = decltype(E_)::value, m = e >> 1, g = e & 1;
                float a[16]; acc_read16<16 * e>(a);
#pragma unroll
                for (int r = 0; r < 4; ++r) {
                    const f32x2 lo = gelu2((f32x2){a[r], a[4 + r]}) + poison, hi = gelu2((f32x2){a[8 + r], a[12 + r]}) + poison;
                    const f32x4 v = (f32x4){lo.x, lo.y, hi.x, hi.y};
                    out_store<g * 256>(v, ovoff, orsrc, obase + (unsigned)(m * 16 + r) * (unsigned)(N * 4));
                }
            });
            if (cur.pn >= NCOM) {
                WAIT_V(0);
                __builtin_amdgcn_fence(__ATOMIC_RELEASE, "agent");
                WAIT_V(0);
                if (lane == 0) __hip_atomic_fetch_add((GAS unsigned*)(myws + O_XCNT + (cur.pm * NEX + cur.pn - NCOM - rank * NEX) * 64), 1u, __ATOMIC_RELAXED, __HIP_MEMORY_SCOPE_AGENT);
            }
        }
        if (!has_next) break;
        cur = nxt; cA = nA; cB = nB; ++ui;
    }
    WAIT_V(0);
}

__global__ __launch_bounds__(256, 1) void fused_kernel(Params p) {
    extern __shared__ __attribute__((aligned(16))) unsigned char smem[];
    LAS unsigned char* lds = (LAS unsigned char*)smem;
    Ctx cx; cx.myws = (GAS char*)p.myws; cx.myout = (GAS float*)p.myout; cx.rank = p.rank; cx.mode = p.mode;
    asm volatile("" : "+s"(cx.myws), "+s"(cx.myout), "+s"(cx.rank), "+s"(cx.mode));
    if (threadIdx.x < 32) {
        unsigned v = 0;
        if (threadIdx.x == 0) v = __hip_atomic_fetch_add((GAS unsigned*)(cx.myws + O_TICKET), 1u, __ATOMIC_RELAXED, __HIP_MEMORY_SCOPE_AGENT) / (unsigned)GRID + 1u;
        lds_st(lds, L_CALL + threadIdx.x * 4, v);
    }
    __builtin_amdgcn_fence(__ATOMIC_ACQUIRE, "");
    __syncthreads();
    const unsigned call = __builtin_amdgcn_readfirstlane(lds_ld(lds, L_CALL));
    const int bid = blockIdx.x, idle = bid - NCOMM - R0 * 8;
    if (!(cx.mode & 2)) {
        if (bid < NCOMM || (idle >= 0 && idle < NHELP)) stream_role(p, cx, lds, call, bid < NCOMM ? bid : NCOMM + idle, bid < NCOMM ? NBLK : HELP_TH);
        else if (idle >= NHELP) self_role(p, cx, call, idle - NHELP);
    }
    if (bid < NCOMM) push_role(p, cx, call);
    else if (!(cx.mode & 4)) gemm_role(cx, lds, call);
}
}

#define FK_MODE 0

extern "C" void kernel_launch(int rank, int nranks, ncclComm_t comm, hipStream_t stream,
                              void* const* d_in, const int* in_sizes, int n_in,
                              void* d_out, int out_size, const int* in_dtypes, int out_dtype,
                              void* const* d_out_all, void* const* d_ws_all, size_t ws_bytes) {
    using namespace fk;
    if (nranks != fk::NR || ws_bytes < WS_NEED || (unsigned)rank >= (unsigned)fk::NR) { fprintf(stderr, "unexpected configuration\n"); return; }
    static bool attr_set = false;
    if (!attr_set) { HIP_CHECK(hipFuncSetAttribute((const void*)fused_kernel, hipFuncAttributeMaxDynamicSharedMemorySize, LDS_BYTES)); attr_set = true; }
    Params p;
    p.x = (const float*)d_in[0]; p.w = (const float*)d_in[1];
    for (int q = 0; q < fk::NR; ++q) { p.out[q] = (float*)d_out_all[q]; p.ws[q] = (char*)d_ws_all[q]; }
    p.out[rank] = (float*)d_out;
    p.myws = p.ws[rank]; p.myout = (float*)d_out; p.rank = rank; p.mode = FK_MODE;
    fused_kernel<<<GRID, 256, LDS_BYTES, stream>>>(p);
    HIP_CHECK(hipGetLastError());
}
```

```cpp
#include <hip/hip_runtime.h>
#include <rccl/rccl.h>
#include <cstdio>
#include <cstdint>
#include <type_traits>

#define HIP_CHECK(expr) do { hipError_t _e = (expr); if (_e != hipSuccess) { \
    fprintf(stderr, "HIP error %s at %s:%d\n", hipGetErrorString(_e), __FILE__, __LINE__); \
    return; } } while (0)

namespace fk {
typedef _Float16 h16x8 __attribute__((ext_vector_type(8)));
typedef float f32x4 __attribute__((ext_vector_type(4)));
typedef unsigned u32x4 __attribute__((ext_vector_type(4)));
#define LAS __attribute__((address_space(3)))
#define GAS __attribute__((address_space(1)))

constexpr int M = 4096, N = 32768, NR = 8, KL = 512;
constexpr int NPM = 16, NPN = 128, NS = 64;
#define NCOMM_ 24
#define NEX_ 3
constexpr int NCOMM = NCOMM_, GRID = 256, WPX = (GRID - NCOMM) / 8;
constexpr int NEX = NEX_;
constexpr int NCOM = NPN - NR * NEX;
constexpr int ENT = 2 * NCOM + 2 * NEX;
constexpr int NGRP = (ENT + WPX - 1) / WPX;
constexpr int R0 = ENT - (NGRP - 1) * WPX;
constexpr int RC0 = R0 - 2 * NEX;
constexpr int SPLIT = ((RC0 & 3) == 1 || (RC0 & 3) == 2) ? RC0 >> 2 : -1;
#define KROT 0
#define PP 7
#define PG0 3
constexpr int NPUSH = NPM * NEX * 8;
#define NHELP_ 0
#define SELF_SLEEP 0
#define STORD 1
#define ROT 1
#define HELP_TH 176
constexpr int NHELP = NHELP_;
constexpr int NSELF = (WPX - R0) * 8 - NHELP;
static_assert(NPM * NEX * 64 <= 3072 && NCOMM <= 32, "control block layout");
static_assert(NSELF > 0 && RC0 >= 0 && 8 * PP * (NGRP - PG0) <= NPUSH, "schedule shape");
constexpr unsigned BLK = 32768, SLAB = 16384;
#define TPAD 36864
constexpr size_t TILE_BYTES = (size_t)NS * BLK + TPAD;
constexpr size_t BUF_BYTES = (size_t)(NPM + NPN) * TILE_BYTES;
constexpr size_t CTRL_BYTES = 1u << 20;
constexpr size_t WS_NEED = CTRL_BYTES + 2 * BUF_BYTES;
constexpr unsigned O_TICKET = 0, O_TMO = 128, O_BCNT = 256, O_XCNT = 1024  , O_DONE = 4096  ,
                   O_FLAG = 8192  , O_DBG = 65536;
constexpr unsigned long long DEADLINE = 50000000ull;
constexpr int STAGE_BYTES = 4 * 2 * SLAB;
constexpr int L_CALL = STAGE_BYTES, L_P = STAGE_BYTES + 4, L_BAD = STAGE_BYTES + 8, L_TKT = STAGE_BYTES + 16  , L_FLG = STAGE_BYTES + 256  ;
constexpr int LDS_BYTES = STAGE_BYTES + 1024;
constexpr int AUX_SYS = 17, AUX_SC1 = 16;
#define SCHEDV 2
#define DMAV 1
#define AUX_ST 17
#define WTEPI 1
#define AUX_PUSH 19
#define OUT_MOD ""
#define AUX_LD 16

__host__ __device__ __forceinline__ constexpr int colmap(int pn) {
    return pn;
}
struct Params { const float* x; const float* w; float* out[NR]; char* ws[NR]; char* myws; float* myout; int rank; int mode; };
struct Ctx { GAS char* myws; GAS float* myout; int rank, mode; };

__device__ __forceinline__ unsigned ld_sys(const GAS unsigned* p) { return __hip_atomic_load(p, __ATOMIC_RELAXED, __HIP_MEMORY_SCOPE_SYSTEM); }
__device__ __forceinline__ unsigned ld_agent(const GAS unsigned* p) { return __hip_atomic_load(p, __ATOMIC_RELAXED, __HIP_MEMORY_SCOPE_AGENT); }
__device__ __forceinline__ void st_sys(GAS unsigned* p, unsigned v) { __hip_atomic_store(p, v, __ATOMIC_RELAXED, __HIP_MEMORY_SCOPE_SYSTEM); }
__device__ __forceinline__ void st_agent(GAS unsigned* p, unsigned v) { __hip_atomic_store(p, v, __ATOMIC_RELAXED, __HIP_MEMORY_SCOPE_AGENT); }
__device__ __forceinline__ unsigned lds_ld(LAS unsigned char* lds, int off) { return *(volatile LAS unsigned*)(lds + off); }
__device__ __forceinline__ void lds_st(LAS unsigned char* lds, int off, unsigned v) { *(volatile LAS unsigned*)(lds + off) = v; }
template <class T> __device__ __forceinline__ T* uni_ptr(T* q) {
    const unsigned long long v = (unsigned long long)q;
    return (T*)(((unsigned long long)(unsigned)__builtin_amdgcn_readfirstlane((int)(v >> 32)) << 32) | (unsigned)__builtin_amdgcn_readfirstlane((int)v));
}
#define WAIT_V_(n) asm volatile("s_nop 1\n\ts_waitcnt vmcnt(" #n ")" ::: "memory")
#define WAIT_V(n) WAIT_V_(n)
#define CW1 63
#define WAIT_L(n) asm volatile("s_waitcnt lgkmcnt(" #n ")" ::: "memory")
#define BAR __builtin_amdgcn_s_barrier()
#define SCHED __builtin_amdgcn_sched_barrier(0)
#define ACLOB "a0","a1","a2","a3","a4","a5","a6","a7","a8","a9","a10","a11","a12","a13","a14","a15","a16","a17","a18","a19","a20","a21","a22","a23","a24","a25","a26","a27","a28","a29","a30","a31","a32","a33","a34","a35","a36","a37","a38","a39","a40","a41","a42","a43","a44","a45","a46","a47","a48","a49","a50","a51","a52","a53","a54","a55","a56","a57","a58","a59","a60","a61","a62","a63","a64","a65","a66","a67","a68","a69","a70","a71","a72","a73","a74","a75","a76","a77","a78","a79","a80","a81","a82","a83","a84","a85","a86","a87","a88","a89","a90","a91","a92","a93","a94","a95","a96","a97","a98","a99","a100","a101","a102","a103","a104","a105","a106","a107","a108","a109","a110","a111","a112","a113","a114","a115","a116","a117","a118","a119","a120","a121","a122","a123","a124","a125","a126","a127","a128","a129","a130","a131","a132","a133","a134","a135","a136","a137","a138","a139","a140","a141","a142","a143","a144","a145","a146","a147","a148","a149","a150","a151","a152","a153","a154","a155","a156","a157","a158","a159","a160","a161","a162","a163","a164","a165","a166","a167","a168","a169","a170","a171","a172","a173","a174","a175","a176","a177","a178","a179","a180","a181","a182","a183","a184","a185","a186","a187","a188","a189","a190","a191","a192","a193","a194","a195","a196","a197","a198","a199","a200","a201","a202","a203","a204","a205","a206","a207","a208","a209","a210","a211","a212","a213","a214","a215","a216","a217","a218","a219","a220","a221","a222","a223","a224","a225","a226","a227","a228","a229","a230","a231","a232","a233","a234","a235","a236","a237","a238","a239","a240","a241","a242","a243","a244","a245","a246","a247","a248","a249","a250","a251","a252","a253","a254","a255"
template <int I, bool ZERO> __device__ __forceinline__ void mfma4(const h16x8& a, const h16x8& b0, const h16x8& b1, const h16x8& b2, const h16x8& b3) {
    if constexpr (ZERO)
    asm volatile("v_mfma_f32_16x16x32_f16 a[%c5:%c6], %0, %1, 0\n\tv_mfma_f32_16x16x32_f16 a[%c7:%c8], %0, %2, 0\n\t"
                 "v_mfma_f32_16x16x32_f16 a[%c9:%c10], %0, %3, 0\n\tv_mfma_f32_16x16x32_f16 a[%c11:%c12], %0, %4, 0"
                 :: "v"(a), "v"(b0), "v"(b1), "v"(b2), "v"(b3), "i"(4 * I), "i"(4 * I + 3), "i"(4 * I + 4), "i"(4 * I + 7), "i"(4 * I + 8), "i"(4 * I + 11), "i"(4 * I + 12), "i"(4 * I + 15) : ACLOB);
    else
    asm volatile("v_mfma_f32_16x16x32_f16 a[%c5:%c6], %0, %1, a[%c5:%c6]\n\tv_mfma_f32_16x16x32_f16 a[%c7:%c8], %0, %2, a[%c7:%c8]\n\t"
                 "v_mfma_f32_16x16x32_f16 a[%c9:%c10], %0, %3, a[%c9:%c10]\n\tv_mfma_f32_16x16x32_f16 a[%c11:%c12], %0, %4, a[%c11:%c12]"
                 :: "v"(a), "v"(b0), "v"(b1), "v"(b2), "v"(b3), "i"(4 * I), "i"(4 * I + 3), "i"(4 * I + 4), "i"(4 * I + 7), "i"(4 * I + 8), "i"(4 * I + 11), "i"(4 * I + 12), "i"(4 * I + 15) : ACLOB);
}
template <int R> __device__ __forceinline__ void acc_read16(float (&v)[16]) {
    asm volatile("v_accvgpr_read_b32 %0, a[%c16+0]\n\tv_accvgpr_read_b32 %1, a[%c16+1]\n\tv_accvgpr_read_b32 %2, a[%c16+2]\n\tv_accvgpr_read_b32 %3, a[%c16+3]\n\tv_accvgpr_read_b32 %4, a[%c16+4]\n\tv_accvgpr_read_b32 %5, a[%c16+5]\n\tv_accvgpr_read_b32 %6, a[%c16+6]\n\tv_accvgpr_read_b32 %7, a[%c16+7]\n\tv_accvgpr_read_b32 %8, a[%c16+8]\n\tv_accvgpr_read_b32 %9, a[%c16+9]\n\tv_accvgpr_read_b32 %10, a[%c16+10]\n\tv_accvgpr_read_b32 %11, a[%c16+11]\n\tv_accvgpr_read_b32 %12, a[%c16+12]\n\tv_accvgpr_read_b32 %13, a[%c16+13]\n\tv_accvgpr_read_b32 %14, a[%c16+14]\n\tv_accvgpr_read_b32 %15, a[%c16+15]" : "=v"(v[0]), "=v"(v[1]), "=v"(v[2]), "=v"(v[3]), "=v"(v[4]), "=v"(v[5]), "=v"(v[6]), "=v"(v[7]), "=v"(v[8]), "=v"(v[9]), "=v"(v[10]), "=v"(v[11]), "=v"(v[12]), "=v"(v[13]), "=v"(v[14]), "=v"(v[15]) : "i"(R) : ACLOB);
}
template <int... Is> struct Seq {};
template <int N_, int... Is> struct MkSeq : MkSeq<N_ - 1, N_ - 1, Is...> {};
template <int... Is> struct MkSeq<0, Is...> { typedef Seq<Is...> type; };
template <class F, int... Is> __device__ __forceinline__ void static_for_impl(F&& f, Seq<Is...>) { (f(std::integral_constant<int, Is>{}), ...); }
template <int N_, class F> __device__ __forceinline__ void static_for(F&& f) { static_for_impl(f, typename MkSeq<N_>::type{}); }

__device__ __forceinline__ bool give_up(GAS unsigned* tmo, unsigned call, unsigned long long t0) {
    const bool g = (unsigned)__builtin_amdgcn_readfirstlane(ld_agent(tmo)) == call || __builtin_amdgcn_s_memrealtime() - t0 > DEADLINE;
    if (g) st_agent(tmo, call);
    return g;
}

constexpr int grp_start_c(int g) {
    if (g <= 0) return 0;
    if (g == 1 && SPLIT >= 0) return 2 * SPLIT + 1;
    const int r = RC0 + (g - 1) * WPX - 1;
    return r < 0 ? 0 : 2 * (r / 4 + 1);
}
struct GrpTab { int s[NGRP + 1]; constexpr GrpTab() : s{} { for (int g = 0; g <= NGRP; ++g) s[g] = grp_start_c(g); } };
constexpr GrpTab GT{};
constexpr int NBLK = 8 * (NPM + NPN);
constexpr int NITEM = 8 * (NPM + NCOM) + NPUSH;
constexpr int TPUSH = NPM + NPN, G0END = 8 * (NPM + GT.s[1]);
#define DRAIN 0
struct Blk { int T, j, dest; };
template <bool SELF> __device__ __forceinline__ Blk decode(int b, int rank) {
    Blk r; r.T = 0; r.j = 0; r.dest = -1;
#pragma unroll
    for (int g = 0; g < NGRP; ++g) {
        const int s = GT.s[g], nb = GT.s[g + 1] - s, nc = nb + (g == 0 ? NPM : 0), mem = nc + (SELF ? (g == 0 ? NR * NEX : 0) : (g >= PG0 ? PP : 0));
        if (b >= 0 && b < 8 * mem) {
            const int j = b / mem, m = b - j * mem; r.j = j;
            if (m >= nc) { if (SELF) { r.dest = (m - nc) & 7; r.T = NPM + NCOM + r.dest * NEX + ((m - nc) >> 3); } else r.T = TPUSH + ((g - PG0) * 8 + j) * PP + m - nc; }
            else if (g == 0) r.T = m < 2 * nb ? ((m & 1) ? NPM + (m >> 1) : (m >> 1)) : m - nb;
            else r.T = NPM + s + m;
        }
        b -= 8 * mem;
    }
    if (!SELF && b >= 0) r.T = TPUSH + (NGRP - PG0) * 8 * PP + b;
    return r;
}
static_assert(GT.s[1] <= NPM, "group 0 interleave assumes at most 16 column tiles");
static_assert(GT.s[NGRP] == NCOM, "groups cover every common column tile");
__device__ __forceinline__ unsigned push_off(int u, int kf, int wid, int rank) {
    const int xt = u >> 3;
    return (unsigned)((xt / NEX) * 256 + (u & 7) * 32 + wid * 8 + kf * 4) * (unsigned)N + (unsigned)colmap(NCOM + rank * NEX + xt % NEX) * 256u;
}
template <int SET> __device__ __forceinline__ void slab_load(const Params& p, const Blk& k, int kf, int wid, int lane) {
    const bool isx = k.T < NPM, isp = k.T >= TPUSH;
    const float* src = isx ? p.x + (size_t)(k.T * 256 + wid * 64 + (lane & 15)) * KL + k.j * 64 + kf * 32 + (lane >> 4) * 8
                     : isp ? p.myout + push_off(k.T - TPUSH, kf, wid, p.rank) + lane * 4
                           : p.w + (size_t)(k.j * 64 + kf * 32 + (lane >> 4) * 8) * N + (size_t)colmap(k.T - NPM) * 256 + wid * 64 + (lane & 15) * 4;
    static_for<8>([&](auto R_) { constexpr int r = decltype(R_)::value;
        const size_t o = isx ? (size_t)((r >> 1) * 16 * KL + (r & 1) * 4) : (size_t)(isp ? r & 3 : r) * N;
        const float* q = src + o;
        asm volatile("global_load_dwordx4 a[%c1:%c2], %0, off nt" :: "v"(q), "i"(SET * 32 + r * 4), "i"(SET * 32 + r * 4 + 3) : "memory", ACLOB);
    });
}
template <int SET> __device__ __forceinline__ void slab_fetch(f32x4 (&v)[8]) {
    static_for<8>([&](auto R_) { constexpr int r = decltype(R_)::value; float x0, x1, x2, x3;
        asm volatile("v_accvgpr_read_b32 %0, a[%c4]\n\tv_accvgpr_read_b32 %1, a[%c5]\n\tv_accvgpr_read_b32 %2, a[%c6]\n\tv_accvgpr_read_b32 %3, a[%c7]"
                     : "=v"(x0), "=v"(x1), "=v"(x2), "=v"(x3) : "i"(SET * 32 + r * 4), "i"(SET * 32 + r * 4 + 1), "i"(SET * 32 + r * 4 + 2), "i"(SET * 32 + r * 4 + 3) : "memory", ACLOB);
        v[r] = (f32x4){x0, x1, x2, x3};
    });
}
__device__ __forceinline__ void slab_convert(const Blk& k, const f32x4 (&v)[8], u32x4 (&o)[4]) {
    if (k.T >= TPUSH) {
#pragma unroll
        for (int i = 0; i < 4; ++i) o[i] = __builtin_bit_cast(u32x4, v[i]);
    } else if (k.T < NPM) {
#pragma unroll
        for (int i = 0; i < 4; ++i) { h16x8 h;
#pragma unroll
            for (int e = 0; e < 4; ++e) { h[e] = (_Float16)v[2 * i][e]; h[4 + e] = (_Float16)v[2 * i + 1][e]; }
            o[i] = __builtin_bit_cast(u32x4, h); }
    } else {
#pragma unroll
        for (int i = 0; i < 4; ++i) { h16x8 h;
#pragma unroll
            for (int r = 0; r < 8; ++r) h[r] = (_Float16)v[r][i];
            o[i] = __builtin_bit_cast(u32x4, h); }
    }
}

__device__ __forceinline__ void stream_role(const Params& p, const Ctx& cx, LAS unsigned char* lds, unsigned call, int c, int th) {
    const int tid = threadIdx.x, wid = __builtin_amdgcn_readfirstlane(tid >> 6), lane = tid & 63, rank = cx.rank;
    GAS char* myws = cx.myws;
    const size_t bufoff = CTRL_BYTES + (size_t)(call & 1u) * BUF_BYTES;
    __amdgpu_buffer_rsrc_t rs[NR - 1];
#pragma unroll
    for (int q = 0; q < NR - 1; ++q) rs[q] = __builtin_amdgcn_make_buffer_rsrc((void*)(p.ws[(rank + 1 + (q + ROT * (c * 4 + wid)) % 7) & 7] + bufoff), 0, (int)BUF_BYTES, 0x00020000);
    __amdgpu_buffer_rsrc_t ro[NR - 1];
#pragma unroll
    for (int q = 0; q < NR - 1; ++q) ro[q] = __builtin_amdgcn_make_buffer_rsrc((void*)p.out[(rank + 1 + (q + ROT * (c * 4 + wid)) % 7) & 7], 0, (int)((size_t)M * N * 4), 0x00020000);
    GAS unsigned* flag_base = (GAS unsigned*)(p.ws[lane & 7] + O_FLAG);
    GAS unsigned* tmo = (GAS unsigned*)(myws + O_TMO);
    const int voff = wid * 4096 + lane * 16;
    bool xok = (cx.mode & 4) != 0, bad = false;
    auto publish = [&](const Blk& k) {
        if (wid == 0 && k.T < TPUSH && lane < NR && lane != rank) st_sys(flag_base + k.T * NS + k.j * 8 + rank, call);
    };
    auto store_slab = [&](const Blk& k, int kf, u32x4 (&o)[4]) {
        if (k.T < TPUSH) {
            const unsigned base = (unsigned)k.T * (unsigned)TILE_BYTES + (unsigned)(k.j * 8 + rank) * BLK + (unsigned)kf * SLAB;
#pragma unroll
            for (int i = 0; i < 4; ++i)
#pragma unroll
                for (int q = 0; q < NR - 1; ++q) __builtin_amdgcn_raw_buffer_store_b128(o[i], rs[q], voff + i * 1024, (int)base, AUX_ST);
        } else {
            const unsigned base = push_off(k.T - TPUSH, kf, wid, rank) * 4u;
            if (bad) {
#pragma unroll
                for (int i = 0; i < 4; ++i) o[i] = (u32x4){0x7fc00000u, 0x7fc00000u, 0x7fc00000u, 0x7fc00000u}; }
#pragma unroll
            for (int i = 0; i < 4; ++i)
#pragma unroll
                for (int q = 0; q < NR - 1; ++q) __builtin_amdgcn_raw_buffer_store_b128(o[i], ro[q], lane * 16, (int)(base + (unsigned)i * (unsigned)(N * 4)), AUX_PUSH);
        }
    };
    u32x4 o[4]; f32x4 v[8];
    const GAS unsigned* bcnt = (const GAS unsigned*)(myws + O_BCNT + (call & 1u) * 128u);
    if (c == 0 && wid == 0 && lane == 0) st_agent((GAS unsigned*)(myws + O_BCNT + (~call & 1u) * 128u), 0u);
    asm volatile("v_accvgpr_write_b32 a[65], 1" ::: ACLOB);
    int bc = c, bn = c + NCOMM + NHELP;
    Blk cur = decode<false>(bc, rank), prev = cur;
    WAIT_V(0); slab_fetch<0>(v); slab_convert(cur, v, o);
    for (int i = 0; bc < NITEM; ++i) {
        const Blk nxt = decode<false>(bn < NITEM ? bn : bc, rank);
        if (nxt.T >= TPUSH && !xok) {
            const GAS unsigned* xc = (const GAS unsigned*)(myws + O_XCNT + (lane < NPM * NEX ? lane : 0) * 64);
            const unsigned long long t0 = __builtin_amdgcn_s_memrealtime();
            while (!__all(ld_agent(xc) >= call * 4u)) { if (give_up(tmo, call, t0)) { bad = true; break; } __builtin_amdgcn_s_sleep(8); }
            xok = true;
        }
        if (wid == 0 && lane == 0 && bn < th) asm volatile("global_atomic_add a[64], %0, a[65], off sc0" :: "v"(bcnt) : "memory", ACLOB);
        slab_load<0>(p, nxt, 0, wid, lane);
        store_slab(cur, 0, o);
        WAIT_V(36);
        BAR;
        if (i > 0) publish(prev);
        slab_fetch<1>(v);
        slab_convert(cur, v, o);
        slab_load<1>(p, nxt, 1, wid, lane);
        store_slab(cur, 1, o);
        const bool drain = bc >= G0END - DRAIN && bc < G0END;
        if (drain) WAIT_V(0); else WAIT_V(CW1);
        if (wid == 0 && lane == 0) { unsigned tk; asm volatile("v_accvgpr_read_b32 %0, a[64]" : "=v"(tk) :: "memory", ACLOB); lds_st(lds, L_TKT + (i & 1) * 4, bn < th ? tk + 2u * (unsigned)(NCOMM + NHELP) : (unsigned)NITEM); }
        WAIT_L(0); BAR;
        if (drain) publish(cur);
        slab_fetch<0>(v);
        slab_convert(nxt, v, o);
        prev = cur; cur = nxt; bc = bn; bn = __builtin_amdgcn_readfirstlane((int)lds_ld(lds, L_TKT + (i & 1) * 4));
    }
    WAIT_V(0); BAR;
    publish(prev);
}

__device__ __forceinline__ void push_role(const Params& p, const Ctx& cx, unsigned call) {
    const int tid = threadIdx.x, wid = __builtin_amdgcn_readfirstlane(tid >> 6), lane = tid & 63, c = blockIdx.x, rank = cx.rank;
    GAS char* myws = cx.myws;
    GAS unsigned* tmo = (GAS unsigned*)(myws + O_TMO);
    if (!(p.mode & 2)) {
        if (wid == 0 && lane < NR && lane != rank) st_sys((GAS unsigned*)(p.ws[lane & 7] + O_DONE) + rank * 32 + c, call);
        if (c == 0) {
            const GAS unsigned* dn = (const GAS unsigned*)(myws + O_DONE) + tid;
            const unsigned long long t0 = __builtin_amdgcn_s_memrealtime();
            for (;;) {
                const bool ok = (tid >> 5) == rank || (tid & 31) >= NCOMM || ld_sys(dn) >= call;
                if (__all(ok)) break;
                if (give_up(tmo, call, t0)) { if (lane == 0) cx.myout[0] = __builtin_nanf(""); break; }
                __builtin_amdgcn_s_sleep(8);
            }
            __builtin_amdgcn_fence(__ATOMIC_ACQUIRE, "");
        }
    }
}

struct Unit { int pm, pn; };
__device__ __forceinline__ Unit unit_of(int e, int xcd, int rank) {
    Unit u;
    const int r = e - 2 * NEX;
    if (r < 0) {
        if ((NEX & 1) && e >= 2 * NEX - 2) { u.pm = 2 * xcd + (e & 1); u.pn = NCOM + rank * NEX + NEX - 1; }
        else { u.pm = 4 * (xcd >> 1) + (e & 3); u.pn = NCOM + rank * NEX + 2 * (e >> 2) + (xcd & 1); }
    } else if ((r >> 2) == SPLIT) { u.pm = 2 * xcd + (r & 1); u.pn = 2 * SPLIT + ((r >> 1) & 1); }
    else { u.pm = 4 * (xcd >> 1) + (r & 3); u.pn = 2 * (r >> 2) + (xcd & 1); }
    return u;
}
typedef float f32x2 __attribute__((ext_vector_type(2)));
__device__ __forceinline__ f32x2 gelu2(f32x2 y) {
    constexpr float k1 = -2.0f * 0.7978845608028654f * 1.4426950408889634f, k3 = k1 * 0.044715f;
    const f32x2 t = y * ((y * y) * k3 + k1);
    f32x2 d; d.x = __builtin_amdgcn_exp2f(t.x); d.y = __builtin_amdgcn_exp2f(t.y);
    d = d + 1.0f;
    f32x2 r; r.x = __builtin_amdgcn_rcpf(d.x); r.y = __builtin_amdgcn_rcpf(d.y);
    return y * r;
}

template <int OFF, bool WT> __device__ __forceinline__ void out_store(const f32x4& v, int voff, const __amdgpu_buffer_rsrc_t& rs, unsigned soff) {
    if constexpr (WT) asm volatile("buffer_store_dwordx4 %0, %1, %2, %3 offen offset:%c4 sc0 sc1\n\ts_nop 1" :: "v"(v), "v"(voff), "s"(rs), "s"(soff), "i"(OFF) : "memory");
    else asm volatile("buffer_store_dwordx4 %0, %1, %2, %3 offen offset:%c4" OUT_MOD "\n\ts_nop 1" :: "v"(v), "v"(voff), "s"(rs), "s"(soff), "i"(OFF) : "memory");
}

__device__ __forceinline__ void self_role(const Params& p, const Ctx& cx, unsigned call, int ib) {
    const int tid = threadIdx.x, wid = __builtin_amdgcn_readfirstlane(tid >> 6), lane = tid & 63, rank = cx.rank;
    const size_t bufoff = CTRL_BYTES + (size_t)(call & 1u) * BUF_BYTES;
    GAS unsigned* flag_base = (GAS unsigned*)(p.ws[lane & 7] + O_FLAG);
    const int voff = wid * 4096 + lane * 16;
    Blk cur = decode<true>(ib, rank), prev = cur; prev.dest = NR;
    auto publish = [&]() { if (wid == 0 && lane < NR && lane == (prev.dest < 0 ? rank : prev.dest)) st_sys(flag_base + prev.T * NS + prev.j * 8 + rank, call); };
    auto trip = [&](auto PH, int b) {
        constexpr int ph = decltype(PH)::value;
        const Blk nxt = decode<true>(b + NSELF < NBLK ? b + NSELF : b, rank);
        slab_load<2 * (ph ^ 1)>(p, nxt, 0, wid, lane); slab_load<2 * (ph ^ 1) + 1>(p, nxt, 1, wid, lane);
        WAIT_V(16); BAR;
        publish();
        const __amdgpu_buffer_rsrc_t rs = __builtin_amdgcn_make_buffer_rsrc((void*)(p.ws[(cur.dest < 0 ? rank : cur.dest) & 7] + bufoff), 0, (int)BUF_BYTES, 0x00020000);
        const unsigned base = (unsigned)cur.T * (unsigned)TILE_BYTES + (unsigned)(cur.j * 8 + rank) * BLK;
        u32x4 o[4], o2[4]; f32x4 v[8];
        slab_fetch<2 * ph>(v); slab_convert(cur, v, o);
        slab_fetch<2 * ph + 1>(v); slab_convert(cur, v, o2);
#pragma unroll
        for (int i = 0; i < 4; ++i) __builtin_amdgcn_raw_buffer_store_b128(o[i], rs, voff + i * 1024, (int)base, AUX_SYS);
#pragma unroll
        for (int i = 0; i < 4; ++i) __builtin_amdgcn_raw_buffer_store_b128(o2[i], rs, voff + i * 1024, (int)(base + SLAB), AUX_SYS);
        asm volatile("s_nop 1" ::: "memory"); SCHED;
        prev = cur; cur = nxt;
    };
    for (int b = ib; b < NBLK; b += 2 * NSELF) {
        trip(std::integral_constant<int, 0>{}, b);
        if (b + NSELF < NBLK) trip(std::integral_constant<int, 1>{}, b + NSELF);
    }
    WAIT_V(0); BAR;
    publish();
}

__device__ __forceinline__ void gemm_role(const Ctx& cx, LAS unsigned char* lds, unsigned call) {
    const int tid = threadIdx.x, wid = __builtin_amdgcn_readfirstlane(tid >> 6), lane = tid & 63, wr = wid >> 1, wc = wid & 1;
    const int rank = cx.rank, xcd = blockIdx.x & 7, w = (blockIdx.x - NCOMM) >> 3;
    GAS char* myws = cx.myws;
    GAS unsigned* tmo = (GAS unsigned*)(myws + O_TMO);
    const int rot2 = KROT ? (xcd >> 1) * 2 : 0;
    const GAS unsigned* flags = (const GAS unsigned*)(myws + O_FLAG) + ((lane & ~7) | ((lane + rot2) & 7));
    const __amdgpu_buffer_rsrc_t rsrc = __builtin_amdgcn_make_buffer_rsrc((void*)(myws + CTRL_BYTES + (size_t)(call & 1u) * BUF_BYTES), 0, (int)BUF_BYTES, 0x00020000);
    const __amdgpu_buffer_rsrc_t orsrc = __builtin_amdgcn_make_buffer_rsrc((void*)cx.myout, 0, (int)((size_t)M * N * 4), 0x00020000);
    const int ovoff = (lane >> 4) * 4 * N * 4 + (lane & 15) * 16;
    const int ntiles = w < R0 ? NGRP : NGRP - 1, total = ntiles * NS;
    const int e1 = w < R0 ? R0 - WPX + w : R0 + w;
    auto tile_of = [&](int i) { const int e = e1 + i * WPX; return unit_of(e < R0 ? w : e, xcd, rank); };
    const unsigned voff = (unsigned)(wid * 4096 + lane * 16), ldsw = (unsigned)wid * 4096u;
    const int aoff = wr * 8192 + lane * 16, boff = SLAB + wc * 8192 + lane * 16;
    const int mode = cx.mode;
    int P = (mode & 1) ? total : 0;
    int pt = 0;
#define DMA(slot, ga, gb, i) __builtin_amdgcn_raw_ptr_buffer_load_lds(rsrc, (LAS void*)(lds + (slot) * 2 * SLAB + ((i) < 4 ? 0 : SLAB) + ldsw), 16, \
        (int)voff, (int)((i) < 4 ? (ga) : (gb)), ((i) & 3) * 1024, AUX_LD)
#define RDA(slot, m) (*(const LAS h16x8*)(lds + (slot) * 2 * SLAB + aoff + (m) * 1024))
#define RDB(slot, n) (*(const LAS h16x8*)(lds + (slot) * 2 * SLAB + boff + (n) * 1024))
#define RDF(s, nA_, nB_, f) do { if constexpr ((f) < 8) nB_[(f)] = RDB(((s) + 1) & 3, (f)); else nA_[(f) - 8] = RDA(((s) + 1) & 3, (f) - 8); } while (0)
#define STEP(s, Z, cA_, cB_, nA_, nB_, ga, gb) do { \
        __builtin_amdgcn_s_waitcnt(0x4070);   BAR; SCHED; \
        static_for<16>([&](auto G_) { constexpr int g = decltype(G_)::value; \
            mfma4<4 * g, Z>(cA_[g >> 1], cB_[(g & 1) * 4], cB_[(g & 1) * 4 + 1], cB_[(g & 1) * 4 + 2], cB_[(g & 1) * 4 + 3]); \
            SCHED; \
            if constexpr (SCHEDV == 0) { \
                if constexpr (g < 8) { RDF(s, nA_, nB_, 2 * g); RDF(s, nA_, nB_, 2 * g + 1); } else { DMA(s, ga, gb, g - 8); } \
            } else if constexpr (SCHEDV == 1) { \
                if constexpr (g < 2) { RDF(s, nA_, nB_, 2 * g); RDF(s, nA_, nB_, 2 * g + 1); } else if constexpr (g < 14) { RDF(s, nA_, nB_, g + 2); } \
                if constexpr (g & 1) { DMA(s, ga, gb, g >> 1); } \
            } else if constexpr (SCHEDV == 3) { \
                if constexpr (g < 8) { DMA(s, ga, gb, (g + 4) & 7); RDF(s, nA_, nB_, g); } \
                else if constexpr (g < 10) { RDF(s, nA_, nB_, 2 * g - 8); RDF(s, nA_, nB_, 2 * g - 7); } \
                else if constexpr (g < 14) { RDF(s, nA_, nB_, g + 2); } \
            } else if constexpr (SCHEDV == 4) { \
                if constexpr (g < 4) { DMA(s, ga, gb, (2 * g + 4) & 7); DMA(s, ga, gb, (2 * g + 5) & 7); } \
                else if constexpr (g < 10) { RDF(s, nA_, nB_, 2 * g - 8); RDF(s, nA_, nB_, 2 * g - 7); } \
                else if constexpr (g < 14) { RDF(s, nA_, nB_, g + 2); } \
            } else { \
                if constexpr ((g & 1) == 0 && g < 12) { RDF(s, nA_, nB_, g); RDF(s, nA_, nB_, g + 1); } \
                if constexpr (g == 12) { RDF(s, nA_, nB_, 12); RDF(s, nA_, nB_, 13); RDF(s, nA_, nB_, 14); RDF(s, nA_, nB_, 15); } \
                if constexpr (g & 1) { DMA(s, ga, gb, g >> 1); } \
            } \
            SCHED; \
        }); } while (0)
#define DMA8(s, ga, gb) do { DMA(s, ga, gb, 0); DMA(s, ga, gb, 1); DMA(s, ga, gb, 2); DMA(s, ga, gb, 3); DMA(s, ga, gb, 4); DMA(s, ga, gb, 5); DMA(s, ga, gb, 6); DMA(s, ga, gb, 7); } while (0)
    auto wait_pos = [&](int need, int ui) {
        if (need > total) need = total;
        if (P >= need) return;
        if (wid == 0) {
            const unsigned long long t0 = __builtin_amdgcn_s_memrealtime();
            for (;;) {
                const Unit u = tile_of(pt);
                const unsigned fa = ld_sys(flags + u.pm * NS), fb = ld_sys(flags + (NPM + u.pn) * NS);
                const unsigned long long mask = __ballot(fa >= call && fb >= call);
                const int cnt = mask == ~0ull ? 64 : __builtin_ctzll(~mask);
                P = pt * NS + cnt;
                if (cnt == 64 && pt + 1 < ntiles && pt <= ui) { ++pt; continue; }
                if (P >= need) break;
                if (give_up(tmo, call, t0)) { lds_st(lds, L_BAD, 1u); P = total; pt = ntiles; break; }
                __builtin_amdgcn_s_sleep(4);
            }
            lds_st(lds, L_P, (unsigned)P);
            WAIT_L(0);
        } else {
            for (;;) { P = __builtin_amdgcn_readfirstlane((int)lds_ld(lds, L_P)); if (P >= need) break; __builtin_amdgcn_s_sleep(2); }
        }
    };
    int ui = 0;
    Unit cur = tile_of(0), nxt = cur;
    h16x8 A0[8], B0[8], A1[8], B1[8];
    const int lm = (mode & 8) ? 0 : -1;
    unsigned cA = (unsigned)(cur.pm & lm) * (unsigned)TILE_BYTES, cB = (unsigned)(NPM + (cur.pn & lm)) * (unsigned)TILE_BYTES;
    wait_pos(2, 0);
    { const unsigned a0 = cA + (unsigned)rot2 * BLK, b0 = cB + (unsigned)rot2 * BLK;
      DMA8(0, a0, b0); DMA8(1, a0 + SLAB, b0 + SLAB); DMA8(2, a0 + 2 * SLAB, b0 + 2 * SLAB); DMA8(3, a0 + 3 * SLAB, b0 + 3 * SLAB); }
    WAIT_V(24); BAR;
#pragma unroll
    for (int i = 0; i < 8; ++i) { A0[i] = RDA(0, i); B0[i] = RDB(0, i); }
    for (;;) {
        const bool has_next = ui + 1 < ntiles;
        if (has_next) nxt = tile_of(ui + 1);
        const unsigned nA = (unsigned)(nxt.pm & lm) * (unsigned)TILE_BYTES, nB = (unsigned)(NPM + (nxt.pn & lm)) * (unsigned)TILE_BYTES;
        for (int t = 0; t < NS; t += 2) {
            const bool last = (t == NS - 2);
            const unsigned so = (unsigned)(((t + 2) & (NS - 8)) | ((t + 2 + rot2) & 7)) * BLK;
            const unsigned a2 = (last ? nA : cA) + so, b2 = (last ? nB : cB) + so;
            const int gpos = ui * NS + t;
            wait_pos(gpos + 4, ui);
            const bool apoll = (wid == 0) && P < total && P < gpos + 28 && pt < ntiles;
            if (apoll) {
                const Unit u = tile_of(pt);
                __builtin_amdgcn_global_load_lds(flags + u.pm * NS, (LAS unsigned*)(lds + L_FLG), 4, 0, AUX_SYS);
                __builtin_amdgcn_global_load_lds(flags + (NPM + u.pn) * NS, (LAS unsigned*)(lds + L_FLG + 256), 4, 0, AUX_SYS);
            }
            if (t == 0) STEP(0, true, A0, B0, A1, B1, a2, b2); else STEP(0, false, A0, B0, A1, B1, a2, b2);
            STEP(1, false, A1, B1, A0, B0, a2 + SLAB, b2 + SLAB);
            STEP(2, false, A0, B0, A1, B1, a2 + 2 * SLAB, b2 + 2 * SLAB);
            if (apoll) {
                const unsigned fa = lds_ld(lds, L_FLG + lane * 4), fb = lds_ld(lds, L_FLG + 256 + lane * 4);
                const unsigned long long mask = __ballot(fa >= call && fb >= call);
                const int cnt = mask == ~0ull ? 64 : __builtin_ctzll(~mask);
                const int newP = pt * NS + cnt;
                if (cnt == 64 && pt + 1 < ntiles && pt <= ui) ++pt;
                if (newP > P) { P = newP; lds_st(lds, L_P, (unsigned)P); }
            }
            STEP(3, false, A1, B1, A0, B0, a2 + 3 * SLAB, b2 + 3 * SLAB);
        }
        {
            const float poison = lds_ld(lds, L_BAD) != 0u ? __builtin_nanf("") : -0.0f;
            const unsigned obase = ((unsigned)(cur.pm * 256 + wr * 128) * (unsigned)N + (unsigned)colmap(cur.pn) * 256u + (unsigned)wc * 128u) * 4u;
            asm volatile("s_nop 15\n\ts_nop 15" ::: "memory");
            auto epi = [&](auto WT_) { static_for<16>([&](auto E_) { constexpr int e = decltype(E_)::value, m = e >> 1, g = e & 1;
                float a[16]; acc_read16<16 * e>(a);
#pragma unroll
                for (int r = 0; r < 4; ++r) {
                    const f32x2 lo = gelu2((f32x2){a[r], a[4 + r]}) + poison, hi = gelu2((f32x2){a[8 + r], a[12 + r]}) + poison;
                    const f32x4 v = (f32x4){lo.x, lo.y, hi.x, hi.y};
                    out_store<g * 256, decltype(WT_)::value>(v, ovoff, orsrc, obase + (unsigned)(m * 16 + r) * (unsigned)(N * 4));
                }
            }); };
            if (cur.pn >= NCOM || (WTEPI == 2 && !has_next)) epi(std::true_type{}); else epi(std::false_type{});
            if (cur.pn >= NCOM) {
                WAIT_V(0);
                if (lane == 0) __hip_atomic_fetch_add((GAS unsigned*)(myws + O_XCNT + (cur.pm * NEX + cur.pn - NCOM - rank * NEX) * 64), 1u, __ATOMIC_RELAXED, __HIP_MEMORY_SCOPE_AGENT);
            }
        }
        if (!has_next) break;
        cur = nxt; cA = nA; cB = nB; ++ui;
    }
    WAIT_V(0);
}

__global__ __launch_bounds__(256, 1) void fused_kernel(Params p) {
    extern __shared__ __attribute__((aligned(16))) unsigned char smem[];
    LAS unsigned char* lds = (LAS unsigned char*)smem;
    Ctx cx; cx.myws = (GAS char*)p.myws; cx.myout = (GAS float*)p.myout; cx.rank = p.rank; cx.mode = p.mode;
    asm volatile("" : "+s"(cx.myws), "+s"(cx.myout), "+s"(cx.rank), "+s"(cx.mode));
    const int bid = blockIdx.x, idle = bid - NCOMM - R0 * 8;
    const bool streams = bid < NCOMM || (idle >= 0 && idle < NHELP);
    if (streams || idle >= NHELP) {
        const Blk k = streams ? decode<false>(bid < NCOMM ? bid : NCOMM + idle, cx.rank) : decode<true>(idle - NHELP, cx.rank);
        const int wid = __builtin_amdgcn_readfirstlane(threadIdx.x >> 6);
        slab_load<0>(p, k, 0, wid, threadIdx.x & 63); slab_load<1>(p, k, 1, wid, threadIdx.x & 63);
    }
    if (threadIdx.x < 32) {
        unsigned v = 0;
        if (threadIdx.x == 0) v = __hip_atomic_fetch_add((GAS unsigned*)(cx.myws + O_TICKET), 1u, __ATOMIC_RELAXED, __HIP_MEMORY_SCOPE_AGENT) / (unsigned)GRID + 1u;
        lds_st(lds, L_CALL + threadIdx.x * 4, v);
    }
    __builtin_amdgcn_fence(__ATOMIC_ACQUIRE, "");
    __syncthreads();
    const unsigned call = __builtin_amdgcn_readfirstlane(lds_ld(lds, L_CALL));
    if (cx.mode & 2) WAIT_V(0);
    else {
        if (streams) stream_role(p, cx, lds, call, bid < NCOMM ? bid : NCOMM + idle, bid < NCOMM ? NITEM : HELP_TH);
        else if (idle >= NHELP) self_role(p, cx, call, idle - NHELP);
    }
    if (bid < NCOMM) push_role(p, cx, call);
    else if (!(cx.mode & 4)) gemm_role(cx, lds, call);
}
}

#define FK_MODE 0

extern "C" void kernel_launch(int rank, int nranks, ncclComm_t comm, hipStream_t stream,
                              void* const* d_in, const int* in_sizes, int n_in,
                              void* d_out, int out_size, const int* in_dtypes, int out_dtype,
                              void* const* d_out_all, void* const* d_ws_all, size_t ws_bytes) {
    using namespace fk;
    if (nranks != fk::NR || ws_bytes < WS_NEED || (unsigned)rank >= (unsigned)fk::NR) { fprintf(stderr, "unexpected configuration\n"); return; }
    static bool attr_set = false;
    if (!attr_set) { HIP_CHECK(hipFuncSetAttribute((const void*)fused_kernel, hipFuncAttributeMaxDynamicSharedMemorySize, LDS_BYTES)); attr_set = true; }
    Params p;
    p.x = (const float*)d_in[0]; p.w = (const float*)d_in[1];
    for (int q = 0; q < fk::NR; ++q) { p.out[q] = (float*)d_out_all[q]; p.ws[q] = (char*)d_ws_all[q]; }
    p.out[rank] = (float*)d_out;
    p.myws = p.ws[rank]; p.myout = (float*)d_out; p.rank = rank; p.mode = FK_MODE;
    fused_kernel<<<GRID, 256, LDS_BYTES, stream>>>(p);
    HIP_CHECK(hipGetLastError());
}
```

```cpp
#include <hip/hip_runtime.h>
#include <rccl/rccl.h>
#include <cstdio>
#include <cstdint>
#include <type_traits>

#define HIP_CHECK(expr) do { hipError_t _e = (expr); if (_e != hipSuccess) { \
    fprintf(stderr, "HIP error %s at %s:%d\n", hipGetErrorString(_e), __FILE__, __LINE__); \
    return; } } while (0)

namespace fk {
typedef _Float16 h16x8 __attribute__((ext_vector_type(8)));
typedef float f32x4 __attribute__((ext_vector_type(4)));
typedef unsigned u32x4 __attribute__((ext_vector_type(4)));
#define LAS __attribute__((address_space(3)))
#define GAS __attribute__((address_space(1)))

constexpr int M = 4096, N = 32768, NR = 8, KL = 512;
constexpr int NPM = 16, NPN = 128, NS = 64;
constexpr int NCOMM = 24, GRID = 256, WPX = (GRID - NCOMM) / 8;
constexpr int NEX = 3;
constexpr int NCOM = NPN - NR * NEX;
constexpr int ENT = 2 * NCOM + 2 * NEX;
constexpr int NGRP = (ENT + WPX - 1) / WPX;
constexpr int R0 = ENT - (NGRP - 1) * WPX;
constexpr int EX0 = 2 * R0, ES0 = EX0 + 2 * NEX, EP0 = ES0 + 2 * (R0 & 1), C1 = R0 + (R0 & 1);
constexpr int PP = 8, PG0 = 4, NPUSH = NPM * NEX * 8;
constexpr int NSELF = (WPX - R0) * 8;
static_assert(NPM * NEX * 64 <= 3072 && NCOMM <= 32, "control block layout");
static_assert(NSELF > 0 && EP0 <= R0 + WPX && 8 * PP * (NGRP - PG0) <= NPUSH, "schedule shape");
constexpr unsigned BLK = 32768, SLAB = 16384;
constexpr size_t TILE_BYTES = (size_t)NS * BLK + 36864;
constexpr size_t BUF_BYTES = (size_t)(NPM + NPN) * TILE_BYTES;
constexpr size_t CTRL_BYTES = 1u << 20;
constexpr size_t WS_NEED = CTRL_BYTES + 2 * BUF_BYTES;
constexpr unsigned O_TICKET = 0, O_TMO = 128, O_BCNT = 256, O_XCNT = 1024  , O_DONE = 4096  ,
                   O_FLAG = 8192  , O_DBG = 65536;
constexpr unsigned long long DEADLINE = 150000000ull;
constexpr int STAGE_BYTES = 4 * 2 * SLAB;
constexpr int L_CALL = STAGE_BYTES, L_P = STAGE_BYTES + 4, L_BAD = STAGE_BYTES + 8, L_TKT = STAGE_BYTES + 16  , L_FLG = STAGE_BYTES + 256  ;
constexpr int LDS_BYTES = STAGE_BYTES + 1024;
constexpr int AUX_SYS = 17, AUX_PUSH = 19, AUX_LD = 16;

struct Params { const float* x; const float* w; float* out[NR]; char* ws[NR]; char* myws; float* myout; int rank; int mode; };
struct Ctx { GAS char* myws; GAS float* myout; int rank, mode; };

__device__ __forceinline__ unsigned ld_sys(const GAS unsigned* p) { return __hip_atomic_load(p, __ATOMIC_RELAXED, __HIP_MEMORY_SCOPE_SYSTEM); }
__device__ __forceinline__ unsigned ld_agent(const GAS unsigned* p) { return __hip_atomic_load(p, __ATOMIC_RELAXED, __HIP_MEMORY_SCOPE_AGENT); }
__device__ __forceinline__ void st_sys(GAS unsigned* p, unsigned v) { __hip_atomic_store(p, v, __ATOMIC_RELAXED, __HIP_MEMORY_SCOPE_SYSTEM); }
__device__ __forceinline__ void st_agent(GAS unsigned* p, unsigned v) { __hip_atomic_store(p, v, __ATOMIC_RELAXED, __HIP_MEMORY_SCOPE_AGENT); }
__device__ __forceinline__ unsigned lds_ld(LAS unsigned char* lds, int off) { return *(volatile LAS unsigned*)(lds + off); }
__device__ __forceinline__ void lds_st(LAS unsigned char* lds, int off, unsigned v) { *(volatile LAS unsigned*)(lds + off) = v; }
#define WAIT_V_(n) asm volatile("s_nop 1\n\ts_waitcnt vmcnt(" #n ")" ::: "memory")
#define WAIT_V(n) WAIT_V_(n)
#define WAIT_L(n) asm volatile("s_waitcnt lgkmcnt(" #n ")" ::: "memory")
#define BAR __builtin_amdgcn_s_barrier()
#define SCHED __builtin_amdgcn_sched_barrier(0)
#define ACLOB "a0","a1","a2","a3","a4","a5","a6","a7","a8","a9","a10","a11","a12","a13","a14","a15","a16","a17","a18","a19","a20","a21","a22","a23","a24","a25","a26","a27","a28","a29","a30","a31","a32","a33","a34","a35","a36","a37","a38","a39","a40","a41","a42","a43","a44","a45","a46","a47","a48","a49","a50","a51","a52","a53","a54","a55","a56","a57","a58","a59","a60","a61","a62","a63","a64","a65","a66","a67","a68","a69","a70","a71","a72","a73","a74","a75","a76","a77","a78","a79","a80","a81","a82","a83","a84","a85","a86","a87","a88","a89","a90","a91","a92","a93","a94","a95","a96","a97","a98","a99","a100","a101","a102","a103","a104","a105","a106","a107","a108","a109","a110","a111","a112","a113","a114","a115","a116","a117","a118","a119","a120","a121","a122","a123","a124","a125","a126","a127","a128","a129","a130","a131","a132","a133","a134","a135","a136","a137","a138","a139","a140","a141","a142","a143","a144","a145","a146","a147","a148","a149","a150","a151","a152","a153","a154","a155","a156","a157","a158","a159","a160","a161","a162","a163","a164","a165","a166","a167","a168","a169","a170","a171","a172","a173","a174","a175","a176","a177","a178","a179","a180","a181","a182","a183","a184","a185","a186","a187","a188","a189","a190","a191","a192","a193","a194","a195","a196","a197","a198","a199","a200","a201","a202","a203","a204","a205","a206","a207","a208","a209","a210","a211","a212","a213","a214","a215","a216","a217","a218","a219","a220","a221","a222","a223","a224","a225","a226","a227","a228","a229","a230","a231","a232","a233","a234","a235","a236","a237","a238","a239","a240","a241","a242","a243","a244","a245","a246","a247","a248","a249","a250","a251","a252","a253","a254","a255"
template <int I, bool ZERO> __device__ __forceinline__ void mfma4(const h16x8& a, const h16x8& b0, const h16x8& b1, const h16x8& b2, const h16x8& b3) {
    if constexpr (ZERO)
    asm volatile("v_mfma_f32_16x16x32_f16 a[%c5:%c6], %0, %1, 0\n\tv_mfma_f32_16x16x32_f16 a[%c7:%c8], %0, %2, 0\n\t"
                 "v_mfma_f32_16x16x32_f16 a[%c9:%c10], %0, %3, 0\n\tv_mfma_f32_16x16x32_f16 a[%c11:%c12], %0, %4, 0"
                 :: "v"(a), "v"(b0), "v"(b1), "v"(b2), "v"(b3), "i"(4 * I), "i"(4 * I + 3), "i"(4 * I + 4), "i"(4 * I + 7), "i"(4 * I + 8), "i"(4 * I + 11), "i"(4 * I + 12), "i"(4 * I + 15) : ACLOB);
    else
    asm volatile("v_mfma_f32_16x16x32_f16 a[%c5:%c6], %0, %1, a[%c5:%c6]\n\tv_mfma_f32_16x16x32_f16 a[%c7:%c8], %0, %2, a[%c7:%c8]\n\t"
                 "v_mfma_f32_16x16x32_f16 a[%c9:%c10], %0, %3, a[%c9:%c10]\n\tv_mfma_f32_16x16x32_f16 a[%c11:%c12], %0, %4, a[%c11:%c12]"
                 :: "v"(a), "v"(b0), "v"(b1), "v"(b2), "v"(b3), "i"(4 * I), "i"(4 * I + 3), "i"(4 * I + 4), "i"(4 * I + 7), "i"(4 * I + 8), "i"(4 * I + 11), "i"(4 * I + 12), "i"(4 * I + 15) : ACLOB);
}
template <int R> __device__ __forceinline__ void acc_read16(float (&v)[16]) {
    asm volatile("v_accvgpr_read_b32 %0, a[%c16+0]\n\tv_accvgpr_read_b32 %1, a[%c16+1]\n\tv_accvgpr_read_b32 %2, a[%c16+2]\n\tv_accvgpr_read_b32 %3, a[%c16+3]\n\tv_accvgpr_read_b32 %4, a[%c16+4]\n\tv_accvgpr_read_b32 %5, a[%c16+5]\n\tv_accvgpr_read_b32 %6, a[%c16+6]\n\tv_accvgpr_read_b32 %7, a[%c16+7]\n\tv_accvgpr_read_b32 %8, a[%c16+8]\n\tv_accvgpr_read_b32 %9, a[%c16+9]\n\tv_accvgpr_read_b32 %10, a[%c16+10]\n\tv_accvgpr_read_b32 %11, a[%c16+11]\n\tv_accvgpr_read_b32 %12, a[%c16+12]\n\tv_accvgpr_read_b32 %13, a[%c16+13]\n\tv_accvgpr_read_b32 %14, a[%c16+14]\n\tv_accvgpr_read_b32 %15, a[%c16+15]" : "=v"(v[0]), "=v"(v[1]), "=v"(v[2]), "=v"(v[3]), "=v"(v[4]), "=v"(v[5]), "=v"(v[6]), "=v"(v[7]), "=v"(v[8]), "=v"(v[9]), "=v"(v[10]), "=v"(v[11]), "=v"(v[12]), "=v"(v[13]), "=v"(v[14]), "=v"(v[15]) : "i"(R) : ACLOB);
}
template <int... Is> struct Seq {};
template <int N_, int... Is> struct MkSeq : MkSeq<N_ - 1, N_ - 1, Is...> {};
template <int... Is> struct MkSeq<0, Is...> { typedef Seq<Is...> type; };
template <class F, int... Is> __device__ __forceinline__ void static_for_impl(F&& f, Seq<Is...>) { (f(std::integral_constant<int, Is>{}), ...); }
template <int N_, class F> __device__ __forceinline__ void static_for(F&& f) { static_for_impl(f, typename MkSeq<N_>::type{}); }

__device__ __forceinline__ bool give_up(GAS unsigned* tmo, unsigned call, unsigned long long t0) {
    const bool g = (unsigned)__builtin_amdgcn_readfirstlane(ld_agent(tmo)) == call || __builtin_amdgcn_s_memrealtime() - t0 > DEADLINE;
    if (g) st_agent(tmo, call);
    return g;
}

constexpr int grp_start_c(int g) {
    if (g <= 0) return 0;
    if (g == 1) return R0;
    const int r = R0 + (g - 1) * WPX - 1 - EP0;
    return r < 0 ? C1 : C1 + 2 * (r / 4 + 1);
}
struct GrpTab { int s[NGRP + 1]; constexpr GrpTab() : s{} { for (int g = 0; g <= NGRP; ++g) s[g] = grp_start_c(g); } };
constexpr GrpTab GT{};
constexpr int NBLK = 8 * (NPM + NPN);
constexpr int NITEM = 8 * (NPM + NCOM) + NPUSH;
constexpr int TPUSH = NPM + NPN, G0END = 8 * (NPM / 2 + GT.s[1]), DRAIN = 24;
struct Blk { int T, j, dest; };
template <bool SELF> __device__ __forceinline__ Blk decode(int b) {
    Blk r; r.T = 0; r.j = 0; r.dest = -1;
#pragma unroll
    for (int g = 0; g < NGRP; ++g) {
        const int s = GT.s[g], nb = GT.s[g + 1] - s, nc = nb + (g < 2 ? NPM / 2 : 0), mem = nc + (!SELF && g >= PG0 ? PP : 0);
        if (b >= 0 && b < 8 * mem) {
            const int j = b / mem, m = b - j * mem; r.j = j;
            if (m >= nc) r.T = TPUSH + ((g - PG0) * 8 + j) * PP + m - nc;
            else if (g < 2 && m < NPM / 2) r.T = (m >> 1) * 4 + (m & 1) + 2 * g;
            else r.T = NPM + s + m - (g < 2 ? NPM / 2 : 0);
        }
        b -= 8 * mem;
    }
    if (b >= 0) {
        if (SELF) { r.j = b / (NR * NEX); const int m = b - r.j * (NR * NEX); r.dest = m & 7; r.T = NPM + NCOM + r.dest * NEX + (m >> 3); }
        else r.T = TPUSH + (NGRP - PG0) * 8 * PP + b;
    }
    return r;
}
static_assert(GT.s[NGRP] == NCOM, "groups cover every common column tile");
__device__ __forceinline__ unsigned push_off(int u, int kf, int wid, int rank) {
    const int xt = u >> 3;
    return (unsigned)((xt / NEX) * 256 + (u & 7) * 32 + wid * 8 + kf * 4) * (unsigned)N + (unsigned)(NCOM + rank * NEX + xt % NEX) * 256u;
}
template <int SET> __device__ __forceinline__ void slab_load(const Params& p, const Blk& k, int kf, int wid, int lane) {
    const bool isx = k.T < NPM, isp = k.T >= TPUSH;
    const float* src = isx ? p.x + (size_t)(k.T * 256 + wid * 64 + (lane & 15)) * KL + k.j * 64 + kf * 32 + (lane >> 4) * 8
                     : isp ? p.myout + push_off(k.T - TPUSH, kf, wid, p.rank) + lane * 4
                           : p.w + (size_t)(k.j * 64 + kf * 32 + (lane >> 4) * 8) * N + (size_t)(k.T - NPM) * 256 + wid * 64 + (lane & 15) * 4;
    static_for<8>([&](auto R_) { constexpr int r = decltype(R_)::value;
        const size_t o = isx ? (size_t)((r >> 1) * 16 * KL + (r & 1) * 4) : (size_t)(isp ? r & 3 : r) * N;
        const float* q = src + o;
        asm volatile("global_load_dwordx4 a[%c1:%c2], %0, off nt" :: "v"(q), "i"(SET * 32 + r * 4), "i"(SET * 32 + r * 4 + 3) : "memory", ACLOB);
    });
}
template <int SET> __device__ __forceinline__ void slab_fetch(f32x4 (&v)[8]) {
    static_for<8>([&](auto R_) { constexpr int r = decltype(R_)::value; float x0, x1, x2, x3;
        asm volatile("v_accvgpr_read_b32 %0, a[%c4]\n\tv_accvgpr_read_b32 %1, a[%c5]\n\tv_accvgpr_read_b32 %2, a[%c6]\n\tv_accvgpr_read_b32 %3, a[%c7]"
                     : "=v"(x0), "=v"(x1), "=v"(x2), "=v"(x3) : "i"(SET * 32 + r * 4), "i"(SET * 32 + r * 4 + 1), "i"(SET * 32 + r * 4 + 2), "i"(SET * 32 + r * 4 + 3) : "memory", ACLOB);
        v[r] = (f32x4){x0, x1, x2, x3};
    });
}
__device__ __forceinline__ void slab_convert(const Blk& k, const f32x4 (&v)[8], u32x4 (&o)[4]) {
    if (k.T >= TPUSH) {
#pragma unroll
        for (int i = 0; i < 4; ++i) o[i] = __builtin_bit_cast(u32x4, v[i]);
    } else if (k.T < NPM) {
#pragma unroll
        for (int i = 0; i < 4; ++i) { h16x8 h;
#pragma unroll
            for (int e = 0; e < 4; ++e) { h[e] = (_Float16)v[2 * i][e]; h[4 + e] = (_Float16)v[2 * i + 1][e]; }
            o[i] = __builtin_bit_cast(u32x4, h); }
    } else {
#pragma unroll
        for (int i = 0; i < 4; ++i) { h16x8 h;
#pragma unroll
            for (int r = 0; r < 8; ++r) h[r] = (_Float16)v[r][i];
            o[i] = __builtin_bit_cast(u32x4, h); }
    }
}

__device__ __forceinline__ void stream_role(const Params& p, const Ctx& cx, LAS unsigned char* lds, unsigned call, int c) {
    const int tid = threadIdx.x, wid = __builtin_amdgcn_readfirstlane(tid >> 6), lane = tid & 63, rank = cx.rank;
    GAS char* myws = cx.myws;
    const size_t bufoff = CTRL_BYTES + (size_t)(call & 1u) * BUF_BYTES;
    __amdgpu_buffer_rsrc_t rs[NR - 1];
#pragma unroll
    for (int q = 0; q < NR - 1; ++q) rs[q] = __builtin_amdgcn_make_buffer_rsrc((void*)(p.ws[(rank + 1 + (q + c * 4 + wid) % 7) & 7] + bufoff), 0, (int)BUF_BYTES, 0x00020000);
    __amdgpu_buffer_rsrc_t ro[NR - 1];
#pragma unroll
    for (int q = 0; q < NR - 1; ++q) ro[q] = __builtin_amdgcn_make_buffer_rsrc((void*)p.out[(rank + 1 + (q + c * 4 + wid) % 7) & 7], 0, (int)((size_t)M * N * 4), 0x00020000);
    GAS unsigned* flag_base = (GAS unsigned*)(p.ws[lane & 7] + O_FLAG);
    GAS unsigned* tmo = (GAS unsigned*)(myws + O_TMO);
    const int voff = wid * 4096 + lane * 16;
    bool xok = (cx.mode & 4) != 0, bad = false;
    auto publish = [&](const Blk& k) {
        if (wid == 0 && k.T < TPUSH && lane < NR && lane != rank) st_sys(flag_base + k.T * NS + k.j * 8 + rank, call);
    };
    auto store_slab = [&](const Blk& k, int kf, u32x4 (&o)[4]) {
        if (k.T < TPUSH) {
            const unsigned base = (unsigned)k.T * (unsigned)TILE_BYTES + (unsigned)(k.j * 8 + rank) * BLK + (unsigned)kf * SLAB;
#pragma unroll
            for (int i = 0; i < 4; ++i)
#pragma unroll
                for (int q = 0; q < NR - 1; ++q) __builtin_amdgcn_raw_buffer_store_b128(o[i], rs[q], voff + i * 1024, (int)base, AUX_SYS);
        } else {
            const unsigned base = push_off(k.T - TPUSH, kf, wid, rank) * 4u;
            if (bad) {
#pragma unroll
                for (int i = 0; i < 4; ++i) o[i] = (u32x4){0x7fc00000u, 0x7fc00000u, 0x7fc00000u, 0x7fc00000u}; }
#pragma unroll
            for (int i = 0; i < 4; ++i)
#pragma unroll
                for (int q = 0; q < NR - 1; ++q) __builtin_amdgcn_raw_buffer_store_b128(o[i], ro[q], lane * 16, (int)(base + (unsigned)i * (unsigned)(N * 4)), AUX_PUSH);
        }
    };
    u32x4 o[4]; f32x4 v[8];
    const GAS unsigned* bcnt = (const GAS unsigned*)(myws + O_BCNT + (call & 1u) * 128u);
    if (c == 0 && wid == 0 && lane < 2) st_agent((GAS unsigned*)(myws + O_BCNT + (~call & 1u) * 128u) + lane * 16, 0u);
    asm volatile("v_accvgpr_write_b32 a[65], 1" ::: ACLOB);
    int bc = c, bn = c + NCOMM;
    Blk cur = decode<false>(bc), prev = cur;
    WAIT_V(0); slab_fetch<0>(v); slab_convert(cur, v, o);
    for (int i = 0; bc < NITEM; ++i) {
        const Blk nxt = decode<false>(bn < NITEM ? bn : bc);
        if (nxt.T >= TPUSH && !xok) {
            const GAS unsigned* xc = (const GAS unsigned*)(myws + O_XCNT + (lane < NPM * NEX ? lane : 0) * 64);
            const unsigned long long t0 = __builtin_amdgcn_s_memrealtime();
            while (!__all(ld_agent(xc) >= call * 4u)) { if (give_up(tmo, call, t0)) { bad = true; break; } __builtin_amdgcn_s_sleep(8); }
            xok = true;
        }
        if (wid == 0 && lane == 0 && bn < NITEM) asm volatile("global_atomic_add a[64], %0, a[65], off sc0" :: "v"(bcnt) : "memory", ACLOB);
        slab_load<0>(p, nxt, 0, wid, lane);
        store_slab(cur, 0, o);
        WAIT_V(36);
        BAR;
        if (i > 0) publish(prev);
        slab_fetch<1>(v);
        slab_convert(cur, v, o);
        slab_load<1>(p, nxt, 1, wid, lane);
        store_slab(cur, 1, o);
        const bool drain = bc >= G0END - DRAIN && bc < G0END;
        if (drain) WAIT_V(0); else WAIT_V(63);
        if (wid == 0 && lane == 0) { unsigned tk; asm volatile("v_accvgpr_read_b32 %0, a[64]" : "=v"(tk) :: "memory", ACLOB); lds_st(lds, L_TKT + (i & 1) * 4, bn < NITEM && tk < (unsigned)NITEM ? tk + 2u * (unsigned)NCOMM : (unsigned)NITEM); }
        WAIT_L(0); BAR;
        if (drain) { publish(cur); if (wid == 0 && lane == 0) __hip_atomic_fetch_add((GAS unsigned*)bcnt + 16, 1u, __ATOMIC_RELAXED, __HIP_MEMORY_SCOPE_AGENT); }
        slab_fetch<0>(v);
        slab_convert(nxt, v, o);
        prev = cur; cur = nxt; bc = bn; bn = __builtin_amdgcn_readfirstlane((int)lds_ld(lds, L_TKT + (i & 1) * 4));
    }
    WAIT_V(0); BAR;
    publish(prev);
}

__device__ __forceinline__ void done_role(const Params& p, const Ctx& cx, unsigned call) {
    const int tid = threadIdx.x, wid = __builtin_amdgcn_readfirstlane(tid >> 6), lane = tid & 63, c = blockIdx.x, rank = cx.rank;
    GAS char* myws = cx.myws;
    GAS unsigned* tmo = (GAS unsigned*)(myws + O_TMO);
    if (!(p.mode & 2)) {
        if (wid == 0 && lane < NR && lane != rank) st_sys((GAS unsigned*)(p.ws[lane & 7] + O_DONE) + rank * 32 + c, call);
        if (c == 0) {
            const GAS unsigned* dn = (const GAS unsigned*)(myws + O_DONE) + tid;
            const unsigned long long t0 = __builtin_amdgcn_s_memrealtime();
            for (;;) {
                const bool ok = (tid >> 5) == rank || (tid & 31) >= NCOMM || ld_sys(dn) >= call;
                if (__all(ok)) break;
                if (give_up(tmo, call, t0)) { if (lane == 0) cx.myout[0] = __builtin_nanf(""); break; }
                __builtin_amdgcn_s_sleep(8);
            }
            __builtin_amdgcn_fence(__ATOMIC_ACQUIRE, "");
        }
    }
}

struct Unit { int pm, pn; };
__device__ __forceinline__ Unit unit_of(int e, int xcd, int rank) {
    Unit u;
    const int rb = 4 * (xcd >> 1), par = xcd & 1;
    if (e < EX0) {
        const int h = e >= R0 ? 1 : 0, i = e - h * R0;
        if ((R0 & 1) && i == R0 - 1) { u.pm = rb + 2 * h + par; u.pn = R0 - 1; }
        else { u.pm = rb + 2 * h + (i & 1); u.pn = 2 * (i >> 1) + par; }
    } else if (e < ES0) {
        const int i = e - EX0;
        if ((NEX & 1) && i >= 2 * NEX - 2) { u.pm = 2 * xcd + (i & 1); u.pn = NCOM + rank * NEX + NEX - 1; }
        else { u.pm = rb + (i & 3); u.pn = NCOM + rank * NEX + 2 * (i >> 2) + par; }
    } else if (e < EP0) { u.pm = 2 * xcd + ((e - ES0) & 1); u.pn = R0; }
    else { const int r = e - EP0; u.pm = rb + (r & 3); u.pn = C1 + 2 * (r >> 2) + par; }
    return u;
}
typedef float f32x2 __attribute__((ext_vector_type(2)));
__device__ __forceinline__ f32x2 gelu2(f32x2 y) {
    constexpr float k1 = -2.0f * 0.7978845608028654f * 1.4426950408889634f, k3 = k1 * 0.044715f;
    const f32x2 t = y * ((y * y) * k3 + k1);
    f32x2 d; d.x = __builtin_amdgcn_exp2f(t.x); d.y = __builtin_amdgcn_exp2f(t.y);
    d = d + 1.0f;
    f32x2 r; r.x = __builtin_amdgcn_rcpf(d.x); r.y = __builtin_amdgcn_rcpf(d.y);
    return y * r;
}

template <int OFF, bool WT> __device__ __forceinline__ void out_store(const f32x4& v, int voff, const __amdgpu_buffer_rsrc_t& rs, unsigned soff) {
    if constexpr (WT) asm volatile("buffer_store_dwordx4 %0, %1, %2, %3 offen offset:%c4 sc0 sc1\n\ts_nop 1" :: "v"(v), "v"(voff), "s"(rs), "s"(soff), "i"(OFF) : "memory");
    else asm volatile("buffer_store_dwordx4 %0, %1, %2, %3 offen offset:%c4" "\n\ts_nop 1" :: "v"(v), "v"(voff), "s"(rs), "s"(soff), "i"(OFF) : "memory");
}

__device__ __forceinline__ void self_role(const Params& p, const Ctx& cx, unsigned call, int ib) {
    const int tid = threadIdx.x, wid = __builtin_amdgcn_readfirstlane(tid >> 6), lane = tid & 63, rank = cx.rank;
    const size_t bufoff = CTRL_BYTES + (size_t)(call & 1u) * BUF_BYTES;
    GAS unsigned* flag_base = (GAS unsigned*)(p.ws[lane & 7] + O_FLAG);
    const int voff = wid * 4096 + lane * 16;
    Blk cur = decode<true>(ib), prev = cur; prev.dest = NR;
    bool go = false;
    auto publish = [&]() { if (wid == 0 && lane < NR && lane == (prev.dest < 0 ? rank : prev.dest)) st_sys(flag_base + prev.T * NS + prev.j * 8 + rank, call); };
    auto trip = [&](auto PH, int b) {
        constexpr int ph = decltype(PH)::value;
        const Blk nxt = decode<true>(b + NSELF < NBLK ? b + NSELF : b);
        slab_load<2 * (ph ^ 1)>(p, nxt, 0, wid, lane); slab_load<2 * (ph ^ 1) + 1>(p, nxt, 1, wid, lane);
        WAIT_V(16); BAR;
        publish();
        if (cur.dest >= 0 && !go) {
            const GAS unsigned* bcnt = (const GAS unsigned*)(cx.myws + O_BCNT + (call & 1u) * 128u);
            const unsigned long long t0 = __builtin_amdgcn_s_memrealtime();
            while ((int)__builtin_amdgcn_readfirstlane(ld_agent(bcnt + 16)) < DRAIN && __builtin_amdgcn_s_memrealtime() - t0 < 20000ull) __builtin_amdgcn_s_sleep(16);
            go = true;
        }
        const __amdgpu_buffer_rsrc_t rs = __builtin_amdgcn_make_buffer_rsrc((void*)(p.ws[(cur.dest < 0 ? rank : cur.dest) & 7] + bufoff), 0, (int)BUF_BYTES, 0x00020000);
        const unsigned base = (unsigned)cur.T * (unsigned)TILE_BYTES + (unsigned)(cur.j * 8 + rank) * BLK;
        u32x4 o[4], o2[4]; f32x4 v[8];
        slab_fetch<2 * ph>(v); slab_convert(cur, v, o);
        slab_fetch<2 * ph + 1>(v); slab_convert(cur, v, o2);
#pragma unroll
        for (int i = 0; i < 4; ++i) __builtin_amdgcn_raw_buffer_store_b128(o[i], rs, voff + i * 1024, (int)base, AUX_SYS);
#pragma unroll
        for (int i = 0; i < 4; ++i) __builtin_amdgcn_raw_buffer_store_b128(o2[i], rs, voff + i * 1024, (int)(base + SLAB), AUX_SYS);
        asm volatile("s_nop 1" ::: "memory"); SCHED;
        prev = cur; cur = nxt;
    };
    for (int b = ib; b < NBLK; b += 2 * NSELF) {
        trip(std::integral_constant<int, 0>{}, b);
        if (b + NSELF < NBLK) trip(std::integral_constant<int, 1>{}, b + NSELF);
    }
    WAIT_V(0); BAR;
    publish();
}

__device__ __forceinline__ void gemm_role(const Ctx& cx, LAS unsigned char* lds, unsigned call) {
    const int tid = threadIdx.x, wid = __builtin_amdgcn_readfirstlane(tid >> 6), lane = tid & 63, wr = wid >> 1, wc = wid & 1;
    const int rank = cx.rank, xcd = blockIdx.x & 7, w = (blockIdx.x - NCOMM) >> 3;
    GAS char* myws = cx.myws;
    GAS unsigned* tmo = (GAS unsigned*)(myws + O_TMO);
    const GAS unsigned* flags = (const GAS unsigned*)(myws + O_FLAG) + lane;
    const __amdgpu_buffer_rsrc_t rsrc = __builtin_amdgcn_make_buffer_rsrc((void*)(myws + CTRL_BYTES + (size_t)(call & 1u) * BUF_BYTES), 0, (int)BUF_BYTES, 0x00020000);
    const __amdgpu_buffer_rsrc_t orsrc = __builtin_amdgcn_make_buffer_rsrc((void*)cx.myout, 0, (int)((size_t)M * N * 4), 0x00020000);
    const int ovoff = (lane >> 4) * 4 * N * 4 + (lane & 15) * 16;
    const int ntiles = w < R0 ? NGRP : NGRP - 1, total = ntiles * NS;
    const int e1 = w < R0 ? R0 - WPX + w : R0 + w;
    auto tile_of = [&](int i) { const int e = e1 + i * WPX; return unit_of(e < R0 ? w : e, xcd, rank); };
    const unsigned voff = (unsigned)(wid * 4096 + lane * 16), ldsw = (unsigned)wid * 4096u;
    const int aoff = wr * 8192 + lane * 16, boff = SLAB + wc * 8192 + lane * 16;
    const int mode = cx.mode;
    int P = (mode & 1) ? total : 0;
    int pt = 0;
#define DMA(slot, ga, gb, i) __builtin_amdgcn_raw_ptr_buffer_load_lds(rsrc, (LAS void*)(lds + (slot) * 2 * SLAB + ((i) < 4 ? 0 : SLAB) + ldsw), 16, \
        (int)voff, (int)((i) < 4 ? (ga) : (gb)), ((i) & 3) * 1024, AUX_LD)
#define RDA(slot, m) (*(const LAS h16x8*)(lds + (slot) * 2 * SLAB + aoff + (m) * 1024))
#define RDB(slot, n) (*(const LAS h16x8*)(lds + (slot) * 2 * SLAB + boff + (n) * 1024))
#define RDF(s, nA_, nB_, f) do { if constexpr ((f) < 8) nB_[(f)] = RDB(((s) + 1) & 3, (f)); else nA_[(f) - 8] = RDA(((s) + 1) & 3, (f) - 8); } while (0)
#define STEP(s, Z, WC, cA_, cB_, nA_, nB_, ga, gb) do { \
        __builtin_amdgcn_s_waitcnt(WC); BAR; SCHED; \
        static_for<16>([&](auto G_) { constexpr int g = decltype(G_)::value; \
            mfma4<4 * g, Z>(cA_[g >> 1], cB_[(g & 1) * 4], cB_[(g & 1) * 4 + 1], cB_[(g & 1) * 4 + 2], cB_[(g & 1) * 4 + 3]); \
            SCHED; \
                if constexpr ((g & 1) == 0 && g < 12) { RDF(s, nA_, nB_, g); RDF(s, nA_, nB_, g + 1); } \
                if constexpr (g == 12) { RDF(s, nA_, nB_, 12); RDF(s, nA_, nB_, 13); RDF(s, nA_, nB_, 14); RDF(s, nA_, nB_, 15); } \
                if constexpr (g & 1) { DMA(s, ga, gb, g >> 1); } \
            SCHED; \
        }); } while (0)
#define DMA8(s, ga, gb) do { DMA(s, ga, gb, 0); DMA(s, ga, gb, 1); DMA(s, ga, gb, 2); DMA(s, ga, gb, 3); DMA(s, ga, gb, 4); DMA(s, ga, gb, 5); DMA(s, ga, gb, 6); DMA(s, ga, gb, 7); } while (0)
#define ITER(Z, WC, gp, ga, gb) do { \
        if (__builtin_expect((gp) >= thr, 0)) arrive(gp, 4); \
        STEP(0, Z, WC, A0, B0, A1, B1, ga, gb); STEP(1, false, WC, A1, B1, A0, B0, (ga) + SLAB, (gb) + SLAB); \
        STEP(2, false, WC, A0, B0, A1, B1, (ga) + 2 * SLAB, (gb) + 2 * SLAB); STEP(3, false, 0x4070, A1, B1, A0, B0, (ga) + 3 * SLAB, (gb) + 3 * SLAB); } while (0)
    int ui = 0, thr = 0;
    bool pend = false;
    auto absorb = [&](unsigned fa, unsigned fb) {
        const unsigned long long mask = __ballot(fa >= call && fb >= call);
        const int cnt = mask == ~0ull ? 64 : __builtin_ctzll(~mask);
        if (pt * NS + cnt > P) P = pt * NS + cnt;
        const bool adv = cnt == 64 && pt + 1 < ntiles && pt <= ui;
        if (adv) ++pt;
        return adv;
    };
    auto arrive = [&](int gpos, int ahead) {
        const int need = gpos + ahead < total ? gpos + ahead : total;
        if (wid == 0) {
            const int P0 = P;
            if (pend) { absorb(lds_ld(lds, L_FLG + lane * 4), lds_ld(lds, L_FLG + 256 + lane * 4)); pend = false; }
            if (P < need) {
                const unsigned long long t0 = __builtin_amdgcn_s_memrealtime();
                for (;;) {
                    const Unit u = tile_of(pt);
                    if (absorb(ld_sys(flags + u.pm * NS), ld_sys(flags + (NPM + u.pn) * NS))) continue;
                    if (P >= need) break;
                    if (give_up(tmo, call, t0)) { lds_st(lds, L_BAD, 1u); P = total; break; }
                    __builtin_amdgcn_s_sleep(4);
                }
            }
            if (P != P0) { lds_st(lds, L_P, (unsigned)P); WAIT_L(0); }
            if (P < total && P < gpos + 28 && ahead == 4) {
                const Unit u = tile_of(pt);
                __builtin_amdgcn_global_load_lds(flags + u.pm * NS, (LAS unsigned*)(lds + L_FLG), 4, 0, AUX_SYS);
                __builtin_amdgcn_global_load_lds(flags + (NPM + u.pn) * NS, (LAS unsigned*)(lds + L_FLG + 256), 4, 0, AUX_SYS);
                pend = true;
            }
            thr = P >= total ? 0x7fffffff : P - 27;
        } else {
            while (P < need) { P = __builtin_amdgcn_readfirstlane((int)lds_ld(lds, L_P)); if (P < need) __builtin_amdgcn_s_sleep(2); }
            thr = P >= total ? 0x7fffffff : P - 3;
        }
    };
    Unit cur = tile_of(0), nxt = cur;
    h16x8 A0[8], B0[8], A1[8], B1[8];
    const int lm = (mode & 8) ? 0 : -1;
    unsigned cA = (unsigned)(cur.pm & lm) * (unsigned)TILE_BYTES, cB = (unsigned)(NPM + (cur.pn & lm)) * (unsigned)TILE_BYTES;
    arrive(0, 2);
    DMA8(0, cA, cB); DMA8(1, cA + SLAB, cB + SLAB); DMA8(2, cA + 2 * SLAB, cB + 2 * SLAB); DMA8(3, cA + 3 * SLAB, cB + 3 * SLAB);
    WAIT_V(0); BAR;
#pragma unroll
    for (int i = 0; i < 8; ++i) { A0[i] = RDA(0, i); B0[i] = RDB(0, i); }
    for (;;) {
        const bool has_next = ui + 1 < ntiles;
        if (has_next) nxt = tile_of(ui + 1);
        const unsigned nA = (unsigned)(nxt.pm & lm) * (unsigned)TILE_BYTES, nB = (unsigned)(NPM + (nxt.pn & lm)) * (unsigned)TILE_BYTES;
        const int g0 = ui * NS;
        ITER(true, 0xc07c, g0, cA + 2 * BLK, cB + 2 * BLK);
        for (int t = 2; t < NS - 2; t += 4) {
            const unsigned a2 = cA + (unsigned)(t + 2) * BLK, b2 = cB + (unsigned)(t + 2) * BLK;
            ITER(false, 0x4070, g0 + t, a2, b2);
            ITER(false, 0x4070, g0 + t + 2, a2 + 2 * BLK, b2 + 2 * BLK);
        }
        ITER(false, 0x4070, g0 + NS - 2, nA, nB);
        {
            const float poison = lds_ld(lds, L_BAD) != 0u ? __builtin_nanf("") : -0.0f;
            const unsigned obase = ((unsigned)(cur.pm * 256 + wr * 128) * (unsigned)N + (unsigned)(cur.pn) * 256u + (unsigned)wc * 128u) * 4u;
            asm volatile("s_nop 15\n\ts_nop 15" ::: "memory");
            auto epi = [&](auto WT_) { static_for<16>([&](auto E_) { constexpr int e = decltype(E_)::value, m = e >> 1, g = e & 1;
                float a[16]; acc_read16<16 * e>(a);
#pragma unroll
                for (int r = 0; r < 4; ++r) {
                    const f32x2 lo = gelu2((f32x2){a[r], a[4 + r]}) + poison, hi = gelu2((f32x2){a[8 + r], a[12 + r]}) + poison;
                    const f32x4 v = (f32x4){lo.x, lo.y, hi.x, hi.y};
                    out_store<g * 256, decltype(WT_)::value>(v, ovoff, orsrc, obase + (unsigned)(m * 16 + r) * (unsigned)(N * 4));
                }
            }); };
            if (cur.pn >= NCOM) epi(std::true_type{}); else epi(std::false_type{});
            if (cur.pn >= NCOM) {
                WAIT_V(0);
                if (lane == 0) __hip_atomic_fetch_add((GAS unsigned*)(myws + O_XCNT + (cur.pm * NEX + cur.pn - NCOM - rank * NEX) * 64), 1u, __ATOMIC_RELAXED, __HIP_MEMORY_SCOPE_AGENT);
            }
        }
        if (!has_next) break;
        cur = nxt; cA = nA; cB = nB; ++ui;
    }
    WAIT_V(0);
}

__global__ __launch_bounds__(256, 1) void fused_kernel(Params p) {
    extern __shared__ __attribute__((aligned(16))) unsigned char smem[];
    LAS unsigned char* lds = (LAS unsigned char*)smem;
    Ctx cx; cx.myws = (GAS char*)p.myws; cx.myout = (GAS float*)p.myout; cx.rank = p.rank; cx.mode = p.mode;
    asm volatile("" : "+s"(cx.myws), "+s"(cx.myout), "+s"(cx.rank), "+s"(cx.mode));
    const int bid = blockIdx.x, idle = bid - NCOMM - R0 * 8;
    const bool streams = bid < NCOMM;
    if (streams || idle >= 0) {
        const Blk k = streams ? decode<false>(bid) : decode<true>(idle);
        const int wid = __builtin_amdgcn_readfirstlane(threadIdx.x >> 6);
        slab_load<0>(p, k, 0, wid, threadIdx.x & 63); slab_load<1>(p, k, 1, wid, threadIdx.x & 63);
    }
    if (threadIdx.x < 32) {
        unsigned v = 0;
        if (threadIdx.x == 0) v = __hip_atomic_fetch_add((GAS unsigned*)(cx.myws + O_TICKET), 1u, __ATOMIC_RELAXED, __HIP_MEMORY_SCOPE_AGENT) / (unsigned)GRID + 1u;
        lds_st(lds, L_CALL + threadIdx.x * 4, v);
    }
    __builtin_amdgcn_fence(__ATOMIC_ACQUIRE, "");
    __syncthreads();
    const unsigned call = __builtin_amdgcn_readfirstlane(lds_ld(lds, L_CALL));
    if (cx.mode & 2) WAIT_V(0);
    else {
        if (streams) stream_role(p, cx, lds, call, bid);
        else if (idle >= 0) self_role(p, cx, call, idle);
    }
    if (bid < NCOMM) done_role(p, cx, call);
    else if (!(cx.mode & 4)) gemm_role(cx, lds, call);
}
}

#define FK_MODE 0

extern "C" void kernel_launch(int rank, int nranks, ncclComm_t comm, hipStream_t stream,
                              void* const* d_in, const int* in_sizes, int n_in,
                              void* d_out, int out_size, const int* in_dtypes, int out_dtype,
                              void* const* d_out_all, void* const* d_ws_all, size_t ws_bytes) {
    using namespace fk;
    if (nranks != fk::NR || ws_bytes < WS_NEED || (unsigned)rank >= (unsigned)fk::NR) { fprintf(stderr, "unexpected configuration\n"); return; }
    static bool attr_set = false;
    if (!attr_set) { HIP_CHECK(hipFuncSetAttribute((const void*)fused_kernel, hipFuncAttributeMaxDynamicSharedMemorySize, LDS_BYTES)); attr_set = true; }
    Params p;
    p.x = (const float*)d_in[0]; p.w = (const float*)d_in[1];
    for (int q = 0; q < fk::NR; ++q) { p.out[q] = (float*)d_out_all[q]; p.ws[q] = (char*)d_ws_all[q]; }
    p.out[rank] = (float*)d_out;
    p.myws = p.ws[rank]; p.myout = (float*)d_out; p.rank = rank; p.mode = FK_MODE;
    fused_kernel<<<GRID, 256, LDS_BYTES, stream>>>(p);
    HIP_CHECK(hipGetLastError());
}
```

```cpp
#include <hip/hip_runtime.h>
#include <rccl/rccl.h>
#include <cstdio>
#include <cstdint>
#include <type_traits>

#define HIP_CHECK(expr) do { hipError_t _e = (expr); if (_e != hipSuccess) { \
    fprintf(stderr, "HIP error %s at %s:%d\n", hipGetErrorString(_e), __FILE__, __LINE__); \
    return; } } while (0)

namespace fk {
typedef _Float16 h16x8 __attribute__((ext_vector_type(8)));
typedef float f32x4 __attribute__((ext_vector_type(4)));
typedef unsigned u32x4 __attribute__((ext_vector_type(4)));
#define LAS __attribute__((address_space(3)))
#define GAS __attribute__((address_space(1)))

constexpr int M = 4096, N = 32768, NR = 8, KL = 512;
constexpr int NPM = 16, NPN = 128, NS = 64;
constexpr int NCOMM = 24, GRID = 256, WPX = (GRID - NCOMM) / 8;
constexpr int NEX = 3;
constexpr int NCOM = NPN - NR * NEX;
constexpr int ENT = 2 * NCOM + 2 * NEX;
constexpr int NGRP = (ENT + WPX - 1) / WPX;
constexpr int R0 = ENT - (NGRP - 1) * WPX;
constexpr int EX0 = 2 * R0, ES0 = EX0 + 2 * NEX, EP0 = ES0 + 2 * (R0 & 1), C1 = R0 + (R0 & 1);
constexpr int PP = 8, PG0 = 4, NPUSH = NPM * NEX * 8;
constexpr int NSELF = (WPX - R0) * 8;
static_assert(NPM * NEX * 64 <= 3072 && NCOMM <= 32, "control block layout");
static_assert(NSELF > 0 && EP0 <= R0 + WPX && 8 * PP * (NGRP - PG0) <= NPUSH, "schedule shape");
constexpr unsigned BLK = 32768, SLAB = 16384;
constexpr size_t TILE_BYTES = (size_t)NS * BLK + 36864;
constexpr size_t BUF_BYTES = (size_t)(NPM + NPN) * TILE_BYTES;
constexpr size_t CTRL_BYTES = 1u << 20;
constexpr size_t WS_NEED = CTRL_BYTES + 2 * BUF_BYTES;
constexpr unsigned O_TICKET = 0, O_TMO = 128, O_BCNT = 256, O_XCNT = 1024  , O_DONE = 4096  ,
                   O_FLAG = 8192  , O_DBG = 65536;
constexpr unsigned long long DEADLINE = 150000000ull;
constexpr int STAGE_BYTES = 4 * 2 * SLAB;
constexpr int L_CALL = STAGE_BYTES, L_P = STAGE_BYTES + 4, L_BAD = STAGE_BYTES + 8, L_TKT = STAGE_BYTES + 16  , L_FLG = STAGE_BYTES + 256  ;
constexpr int LDS_BYTES = STAGE_BYTES + 1024;
constexpr int AUX_SYS = 17, AUX_PUSH = 19, AUX_LD = 16;

struct Params { const float* x; const float* w; float* out[NR]; char* ws[NR]; char* myws; float* myout; int rank; int mode; };
struct Ctx { GAS char* myws; GAS float* myout; int rank, mode; };

__device__ __forceinline__ unsigned ld_sys(const GAS unsigned* p) { return __hip_atomic_load(p, __ATOMIC_RELAXED, __HIP_MEMORY_SCOPE_SYSTEM); }
__device__ __forceinline__ unsigned ld_agent(const GAS unsigned* p) { return __hip_atomic_load(p, __ATOMIC_RELAXED, __HIP_MEMORY_SCOPE_AGENT); }
__device__ __forceinline__ void st_sys(GAS unsigned* p, unsigned v) { __hip_atomic_store(p, v, __ATOMIC_RELAXED, __HIP_MEMORY_SCOPE_SYSTEM); }
__device__ __forceinline__ void st_agent(GAS unsigned* p, unsigned v) { __hip_atomic_store(p, v, __ATOMIC_RELAXED, __HIP_MEMORY_SCOPE_AGENT); }
__device__ __forceinline__ unsigned lds_ld(LAS unsigned char* lds, int off) { return *(volatile LAS unsigned*)(lds + off); }
__device__ __forceinline__ void lds_st(LAS unsigned char* lds, int off, unsigned v) { *(volatile LAS unsigned*)(lds + off) = v; }
#define WAIT_V_(n) asm volatile("s_nop 1\n\ts_waitcnt vmcnt(" #n ")" ::: "memory")
#define WAIT_V(n) WAIT_V_(n)
#define WAIT_L(n) asm volatile("s_waitcnt lgkmcnt(" #n ")" ::: "memory")
#define BAR __builtin_amdgcn_s_barrier()
#define SCHED __builtin_amdgcn_sched_barrier(0)
#define ACLOB "a0","a1","a2","a3","a4","a5","a6","a7","a8","a9","a10","a11","a12","a13","a14","a15","a16","a17","a18","a19","a20","a21","a22","a23","a24","a25","a26","a27","a28","a29","a30","a31","a32","a33","a34","a35","a36","a37","a38","a39","a40","a41","a42","a43","a44","a45","a46","a47","a48","a49","a50","a51","a52","a53","a54","a55","a56","a57","a58","a59","a60","a61","a62","a63","a64","a65","a66","a67","a68","a69","a70","a71","a72","a73","a74","a75","a76","a77","a78","a79","a80","a81","a82","a83","a84","a85","a86","a87","a88","a89","a90","a91","a92","a93","a94","a95","a96","a97","a98","a99","a100","a101","a102","a103","a104","a105","a106","a107","a108","a109","a110","a111","a112","a113","a114","a115","a116","a117","a118","a119","a120","a121","a122","a123","a124","a125","a126","a127","a128","a129","a130","a131","a132","a133","a134","a135","a136","a137","a138","a139","a140","a141","a142","a143","a144","a145","a146","a147","a148","a149","a150","a151","a152","a153","a154","a155","a156","a157","a158","a159","a160","a161","a162","a163","a164","a165","a166","a167","a168","a169","a170","a171","a172","a173","a174","a175","a176","a177","a178","a179","a180","a181","a182","a183","a184","a185","a186","a187","a188","a189","a190","a191","a192","a193","a194","a195","a196","a197","a198","a199","a200","a201","a202","a203","a204","a205","a206","a207","a208","a209","a210","a211","a212","a213","a214","a215","a216","a217","a218","a219","a220","a221","a222","a223","a224","a225","a226","a227","a228","a229","a230","a231","a232","a233","a234","a235","a236","a237","a238","a239","a240","a241","a242","a243","a244","a245","a246","a247","a248","a249","a250","a251","a252","a253","a254","a255"
template <int I, bool ZERO> __device__ __forceinline__ void mfma2(const h16x8& a, const h16x8& b0, const h16x8& b1) {
    if constexpr (ZERO)
    asm volatile("v_mfma_f32_16x16x32_f16 a[%c3:%c4], %0, %1, 0\n\tv_mfma_f32_16x16x32_f16 a[%c5:%c6], %0, %2, 0"
                 :: "v"(a), "v"(b0), "v"(b1), "i"(4 * I), "i"(4 * I + 3), "i"(4 * I + 4), "i"(4 * I + 7) : ACLOB);
    else
    asm volatile("v_mfma_f32_16x16x32_f16 a[%c3:%c4], %0, %1, a[%c3:%c4]\n\tv_mfma_f32_16x16x32_f16 a[%c5:%c6], %0, %2, a[%c5:%c6]"
                 :: "v"(a), "v"(b0), "v"(b1), "i"(4 * I), "i"(4 * I + 3), "i"(4 * I + 4), "i"(4 * I + 7) : ACLOB);
}
template <int R> __device__ __forceinline__ void acc_read16(float (&v)[16]) {
    asm volatile("v_accvgpr_read_b32 %0, a[%c16+0]\n\tv_accvgpr_read_b32 %1, a[%c16+1]\n\tv_accvgpr_read_b32 %2, a[%c16+2]\n\tv_accvgpr_read_b32 %3, a[%c16+3]\n\tv_accvgpr_read_b32 %4, a[%c16+4]\n\tv_accvgpr_read_b32 %5, a[%c16+5]\n\tv_accvgpr_read_b32 %6, a[%c16+6]\n\tv_accvgpr_read_b32 %7, a[%c16+7]\n\tv_accvgpr_read_b32 %8, a[%c16+8]\n\tv_accvgpr_read_b32 %9, a[%c16+9]\n\tv_accvgpr_read_b32 %10, a[%c16+10]\n\tv_accvgpr_read_b32 %11, a[%c16+11]\n\tv_accvgpr_read_b32 %12, a[%c16+12]\n\tv_accvgpr_read_b32 %13, a[%c16+13]\n\tv_accvgpr_read_b32 %14, a[%c16+14]\n\tv_accvgpr_read_b32 %15, a[%c16+15]" : "=v"(v[0]), "=v"(v[1]), "=v"(v[2]), "=v"(v[3]), "=v"(v[4]), "=v"(v[5]), "=v"(v[6]), "=v"(v[7]), "=v"(v[8]), "=v"(v[9]), "=v"(v[10]), "=v"(v[11]), "=v"(v[12]), "=v"(v[13]), "=v"(v[14]), "=v"(v[15]) : "i"(R) : ACLOB);
}
template <int... Is> struct Seq {};
template <int N_, int... Is> struct MkSeq : MkSeq<N_ - 1, N_ - 1, Is...> {};
template <int... Is> struct MkSeq<0, Is...> { typedef Seq<Is...> type; };
template <class F, int... Is> __device__ __forceinline__ void static_for_impl(F&& f, Seq<Is...>) { (f(std::integral_constant<int, Is>{}), ...); }
template <int N_, class F> __device__ __forceinline__ void static_for(F&& f) { static_for_impl(f, typename MkSeq<N_>::type{}); }

__device__ __forceinline__ bool give_up(GAS unsigned* tmo, unsigned call, unsigned long long t0) {
    const bool g = (unsigned)__builtin_amdgcn_readfirstlane(ld_agent(tmo)) == call || __builtin_amdgcn_s_memrealtime() - t0 > DEADLINE;
    if (g) st_agent(tmo, call);
    return g;
}

constexpr int grp_start_c(int g) {
    if (g <= 0) return 0;
    if (g == 1) return R0;
    const int r = R0 + (g - 1) * WPX - 1 - EP0;
    return r < 0 ? C1 : C1 + 2 * (r / 4 + 1);
}
struct GrpTab { int s[NGRP + 1]; constexpr GrpTab() : s{} { for (int g = 0; g <= NGRP; ++g) s[g] = grp_start_c(g); } };
constexpr GrpTab GT{};
constexpr int NBLK = 8 * (NPM + NPN);
constexpr int NITEM = 8 * (NPM + NCOM) + NPUSH;
constexpr int TPUSH = NPM + NPN, G0END = 8 * (NPM / 2 + GT.s[1]), DRAIN = 24;
struct Blk { int T, j, dest; };
template <bool SELF> __device__ __forceinline__ Blk decode(int b) {
    Blk r; r.T = 0; r.j = 0; r.dest = -1;
#pragma unroll
    for (int g = 0; g < NGRP; ++g) {
        const int s = GT.s[g], nb = GT.s[g + 1] - s, nc = nb + (g < 2 ? NPM / 2 : 0), mem = nc + (!SELF && g >= PG0 ? PP : 0);
        if (b >= 0 && b < 8 * mem) {
            const int j = b / mem, m = b - j * mem; r.j = j;
            if (m >= nc) r.T = TPUSH + ((g - PG0) * 8 + j) * PP + m - nc;
            else if (g < 2 && m < NPM / 2) r.T = (m >> 1) * 4 + (m & 1) + 2 * g;
            else r.T = NPM + s + m - (g < 2 ? NPM / 2 : 0);
        }
        b -= 8 * mem;
    }
    if (b >= 0) {
        if (SELF) { r.j = b / (NR * NEX); const int m = b - r.j * (NR * NEX); r.dest = m & 7; r.T = NPM + NCOM + r.dest * NEX + (m >> 3); }
        else r.T = TPUSH + (NGRP - PG0) * 8 * PP + b;
    }
    return r;
}
static_assert(GT.s[NGRP] == NCOM, "groups cover every common column tile");
__device__ __forceinline__ unsigned push_off(int u, int kf, int wid, int rank) {
    const int xt = u >> 3;
    return (unsigned)((xt / NEX) * 256 + (u & 7) * 32 + wid * 8 + kf * 4) * (unsigned)N + (unsigned)(NCOM + rank * NEX + xt % NEX) * 256u;
}
template <int SET> __device__ __forceinline__ void slab_load(const Params& p, const Blk& k, int kf, int wid, int lane) {
    const bool isx = k.T < NPM, isp = k.T >= TPUSH;
    const float* src = isx ? p.x + (size_t)(k.T * 256 + wid * 64 + (lane & 15)) * KL + k.j * 64 + kf * 32 + (lane >> 4) * 8
                     : isp ? p.myout + push_off(k.T - TPUSH, kf, wid, p.rank) + lane * 4
                           : p.w + (size_t)(k.j * 64 + kf * 32 + (lane >> 4) * 8) * N + (size_t)(k.T - NPM) * 256 + wid * 64 + (lane & 15) * 4;
    static_for<8>([&](auto R_) { constexpr int r = decltype(R_)::value;
        const size_t o = isx ? (size_t)((r >> 1) * 16 * KL + (r & 1) * 4) : (size_t)(isp ? r & 3 : r) * N;
        const float* q = src + o;
        asm volatile("global_load_dwordx4 a[%c1:%c2], %0, off nt" :: "v"(q), "i"(SET * 32 + r * 4), "i"(SET * 32 + r * 4 + 3) : "memory", ACLOB);
    });
}
template <int SET> __device__ __forceinline__ void slab_fetch(f32x4 (&v)[8]) {
    static_for<8>([&](auto R_) { constexpr int r = decltype(R_)::value; float x0, x1, x2, x3;
        asm volatile("v_accvgpr_read_b32 %0, a[%c4]\n\tv_accvgpr_read_b32 %1, a[%c5]\n\tv_accvgpr_read_b32 %2, a[%c6]\n\tv_accvgpr_read_b32 %3, a[%c7]"
                     : "=v"(x0), "=v"(x1), "=v"(x2), "=v"(x3) : "i"(SET * 32 + r * 4), "i"(SET * 32 + r * 4 + 1), "i"(SET * 32 + r * 4 + 2), "i"(SET * 32 + r * 4 + 3) : "memory", ACLOB);
        v[r] = (f32x4){x0, x1, x2, x3};
    });
}
__device__ __forceinline__ void slab_convert(const Blk& k, const f32x4 (&v)[8], u32x4 (&o)[4]) {
    if (k.T >= TPUSH) {
#pragma unroll
        for (int i = 0; i < 4; ++i) o[i] = __builtin_bit_cast(u32x4, v[i]);
    } else if (k.T < NPM) {
#pragma unroll
        for (int i = 0; i < 4; ++i) { h16x8 h;
#pragma unroll
            for (int e = 0; e < 4; ++e) { h[e] = (_Float16)v[2 * i][e]; h[4 + e] = (_Float16)v[2 * i + 1][e]; }
            o[i] = __builtin_bit_cast(u32x4, h); }
    } else {
#pragma unroll
        for (int i = 0; i < 4; ++i) { h16x8 h;
#pragma unroll
            for (int r = 0; r < 8; ++r) h[r] = (_Float16)v[r][i];
            o[i] = __builtin_bit_cast(u32x4, h); }
    }
}

__device__ __forceinline__ void stream_role(const Params& p, const Ctx& cx, LAS unsigned char* lds, unsigned call, int c) {
    const int tid = threadIdx.x, wid = __builtin_amdgcn_readfirstlane(tid >> 6), lane = tid & 63, rank = cx.rank;
    GAS char* myws = cx.myws;
    const size_t bufoff = CTRL_BYTES + (size_t)(call & 1u) * BUF_BYTES;
    __amdgpu_buffer_rsrc_t rs[NR - 1];
#pragma unroll
    for (int q = 0; q < NR - 1; ++q) rs[q] = __builtin_amdgcn_make_buffer_rsrc((void*)(p.ws[(rank + 1 + (q + c * 4 + wid) % 7) & 7] + bufoff), 0, (int)BUF_BYTES, 0x00020000);
    __amdgpu_buffer_rsrc_t ro[NR - 1];
#pragma unroll
    for (int q = 0; q < NR - 1; ++q) ro[q] = __builtin_amdgcn_make_buffer_rsrc((void*)p.out[(rank + 1 + (q + c * 4 + wid) % 7) & 7], 0, (int)((size_t)M * N * 4), 0x00020000);
    GAS unsigned* flag_base = (GAS unsigned*)(p.ws[lane & 7] + O_FLAG);
    GAS unsigned* tmo = (GAS unsigned*)(myws + O_TMO);
    const int voff = wid * 4096 + lane * 16;
    bool xok = (cx.mode & 4) != 0, bad = false;
    auto publish = [&](const Blk& k) {
        if (wid == 0 && k.T < TPUSH && lane < NR && lane != rank) st_sys(flag_base + k.T * NS + k.j * 8 + rank, call);
    };
    auto store_slab = [&](const Blk& k, int kf, u32x4 (&o)[4]) {
        if (k.T < TPUSH) {
            const unsigned base = (unsigned)k.T * (unsigned)TILE_BYTES + (unsigned)(k.j * 8 + rank) * BLK + (unsigned)kf * SLAB;
#pragma unroll
            for (int i = 0; i < 4; ++i)
#pragma unroll
                for (int q = 0; q < NR - 1; ++q) __builtin_amdgcn_raw_buffer_store_b128(o[i], rs[q], voff + i * 1024, (int)base, AUX_SYS);
        } else {
            const unsigned base = push_off(k.T - TPUSH, kf, wid, rank) * 4u;
            if (bad) {
#pragma unroll
                for (int i = 0; i < 4; ++i) o[i] = (u32x4){0x7fc00000u, 0x7fc00000u, 0x7fc00000u, 0x7fc00000u}; }
#pragma unroll
            for (int i = 0; i < 4; ++i)
#pragma unroll
                for (int q = 0; q < NR - 1; ++q) __builtin_amdgcn_raw_buffer_store_b128(o[i], ro[q], lane * 16, (int)(base + (unsigned)i * (unsigned)(N * 4)), AUX_PUSH);
        }
    };
    u32x4 o[4]; f32x4 v[8];
    const GAS unsigned* bcnt = (const GAS unsigned*)(myws + O_BCNT + (call & 1u) * 128u);
    if (c == 0 && wid == 0 && lane < 2) st_agent((GAS unsigned*)(myws + O_BCNT + (~call & 1u) * 128u) + lane * 16, 0u);
    asm volatile("v_accvgpr_write_b32 a[65], 1" ::: ACLOB);
    int bc = c, bn = c + NCOMM;
    Blk cur = decode<false>(bc), prev = cur;
    WAIT_V(0); slab_fetch<0>(v); slab_convert(cur, v, o);
    for (int i = 0; bc < NITEM; ++i) {
        const Blk nxt = decode<false>(bn < NITEM ? bn : bc);
        if (nxt.T >= TPUSH && !xok) {
            const GAS unsigned* xc = (const GAS unsigned*)(myws + O_XCNT + (lane < NPM * NEX ? lane : 0) * 64);
            const unsigned long long t0 = __builtin_amdgcn_s_memrealtime();
            while (!__all(ld_agent(xc) >= call * 4u)) { if (give_up(tmo, call, t0)) { bad = true; break; } __builtin_amdgcn_s_sleep(8); }
            xok = true;
        }
        if (wid == 0 && lane == 0 && bn < NITEM) asm volatile("global_atomic_add a[64], %0, a[65], off sc0" :: "v"(bcnt) : "memory", ACLOB);
        slab_load<0>(p, nxt, 0, wid, lane);
        store_slab(cur, 0, o);
        WAIT_V(36);
        BAR;
        if (i > 0) publish(prev);
        slab_fetch<1>(v);
        slab_convert(cur, v, o);
        slab_load<1>(p, nxt, 1, wid, lane);
        store_slab(cur, 1, o);
        const bool drain = bc >= G0END - DRAIN && bc < G0END;
        if (drain) WAIT_V(0); else WAIT_V(63);
        if (wid == 0 && lane == 0) { unsigned tk; asm volatile("v_accvgpr_read_b32 %0, a[64]" : "=v"(tk) :: "memory", ACLOB); lds_st(lds, L_TKT + (i & 1) * 4, bn < NITEM && tk < (unsigned)NITEM ? tk + 2u * (unsigned)NCOMM : (unsigned)NITEM); }
        WAIT_L(0); BAR;
        if (drain) { publish(cur); if (wid == 0 && lane == 0) __hip_atomic_fetch_add((GAS unsigned*)bcnt + 16, 1u, __ATOMIC_RELAXED, __HIP_MEMORY_SCOPE_AGENT); }
        slab_fetch<0>(v);
        slab_convert(nxt, v, o);
        prev = cur; cur = nxt; bc = bn; bn = __builtin_amdgcn_readfirstlane((int)lds_ld(lds, L_TKT + (i & 1) * 4));
    }
    WAIT_V(0); BAR;
    publish(prev);
}

__device__ __forceinline__ void done_role(const Params& p, const Ctx& cx, unsigned call) {
    const int tid = threadIdx.x, wid = __builtin_amdgcn_readfirstlane(tid >> 6), lane = tid & 63, c = blockIdx.x, rank = cx.rank;
    GAS char* myws = cx.myws;
    GAS unsigned* tmo = (GAS unsigned*)(myws + O_TMO);
    if (!(p.mode & 2)) {
        if (wid == 0 && lane < NR && lane != rank) st_sys((GAS unsigned*)(p.ws[lane & 7] + O_DONE) + rank * 32 + c, call);
        if (c == 0) {
            const GAS unsigned* dn = (const GAS unsigned*)(myws + O_DONE) + tid;
            const unsigned long long t0 = __builtin_amdgcn_s_memrealtime();
            for (;;) {
                const bool ok = (tid >> 5) == rank || (tid & 31) >= NCOMM || ld_sys(dn) >= call;
                if (__all(ok)) break;
                if (give_up(tmo, call, t0)) { if (lane == 0) cx.myout[0] = __builtin_nanf(""); break; }
                __builtin_amdgcn_s_sleep(8);
            }
            __builtin_amdgcn_fence(__ATOMIC_ACQUIRE, "");
        }
    }
}

struct Unit { int pm, pn; };
__device__ __forceinline__ Unit unit_of(int e, int xcd, int rank) {
    Unit u;
    const int rb = 4 * (xcd >> 1), par = xcd & 1;
    if (e < EX0) {
        const int h = e >= R0 ? 1 : 0, i = e - h * R0;
        if ((R0 & 1) && i == R0 - 1) { u.pm = rb + 2 * h + par; u.pn = R0 - 1; }
        else { u.pm = rb + 2 * h + (i & 1); u.pn = 2 * (i >> 1) + par; }
    } else if (e < ES0) {
        const int i = e - EX0;
        if ((NEX & 1) && i >= 2 * NEX - 2) { u.pm = 2 * xcd + (i & 1); u.pn = NCOM + rank * NEX + NEX - 1; }
        else { u.pm = rb + (i & 3); u.pn = NCOM + rank * NEX + 2 * (i >> 2) + par; }
    } else if (e < EP0) { u.pm = 2 * xcd + ((e - ES0) & 1); u.pn = R0; }
    else { const int r = e - EP0; u.pm = rb + (r & 3); u.pn = C1 + 2 * (r >> 2) + par; }
    return u;
}
typedef float f32x2 __attribute__((ext_vector_type(2)));
__device__ __forceinline__ f32x2 gelu2(f32x2 y) {
    constexpr float k1 = -2.0f * 0.7978845608028654f * 1.4426950408889634f, k3 = k1 * 0.044715f;
    const f32x2 t = y * ((y * y) * k3 + k1);
    f32x2 d; d.x = __builtin_amdgcn_exp2f(t.x); d.y = __builtin_amdgcn_exp2f(t.y);
    d = d + 1.0f;
    f32x2 r; r.x = __builtin_amdgcn_rcpf(d.x); r.y = __builtin_amdgcn_rcpf(d.y);
    return y * r;
}

template <int OFF, bool WT> __device__ __forceinline__ void out_store(const f32x4& v, int voff, const __amdgpu_buffer_rsrc_t& rs, unsigned soff) {
    if constexpr (WT) asm volatile("buffer_store_dwordx4 %0, %1, %2, %3 offen offset:%c4 sc0 sc1\n\ts_nop 1" :: "v"(v), "v"(voff), "s"(rs), "s"(soff), "i"(OFF) : "memory");
    else asm volatile("buffer_store_dwordx4 %0, %1, %2, %3 offen offset:%c4" "\n\ts_nop 1" :: "v"(v), "v"(voff), "s"(rs), "s"(soff), "i"(OFF) : "memory");
}

__device__ __forceinline__ void self_role(const Params& p, const Ctx& cx, unsigned call, int ib) {
    const int tid = threadIdx.x, wid = __builtin_amdgcn_readfirstlane(tid >> 6), lane = tid & 63, rank = cx.rank;
    const size_t bufoff = CTRL_BYTES + (size_t)(call & 1u) * BUF_BYTES;
    GAS unsigned* flag_base = (GAS unsigned*)(p.ws[lane & 7] + O_FLAG);
    const int voff = wid * 4096 + lane * 16;
    Blk cur = decode<true>(ib), prev = cur; prev.dest = NR;
    bool go = false;
    auto publish = [&]() { if (wid == 0 && lane < NR && lane == (prev.dest < 0 ? rank : prev.dest)) st_sys(flag_base + prev.T * NS + prev.j * 8 + rank, call); };
    auto trip = [&](auto PH, int b) {
        constexpr int ph = decltype(PH)::value;
        const Blk nxt = decode<true>(b + NSELF < NBLK ? b + NSELF : b);
        slab_load<2 * (ph ^ 1)>(p, nxt, 0, wid, lane); slab_load<2 * (ph ^ 1) + 1>(p, nxt, 1, wid, lane);
        WAIT_V(16); BAR;
        publish();
        if (cur.dest >= 0 && !go) {
            const GAS unsigned* bcnt = (const GAS unsigned*)(cx.myws + O_BCNT + (call & 1u) * 128u);
            const unsigned long long t0 = __builtin_amdgcn_s_memrealtime();
            while ((int)__builtin_amdgcn_readfirstlane(ld_agent(bcnt + 16)) < DRAIN && __builtin_amdgcn_s_memrealtime() - t0 < 20000ull) __builtin_amdgcn_s_sleep(16);
            go = true;
        }
        const __amdgpu_buffer_rsrc_t rs = __builtin_amdgcn_make_buffer_rsrc((void*)(p.ws[(cur.dest < 0 ? rank : cur.dest) & 7] + bufoff), 0, (int)BUF_BYTES, 0x00020000);
        const unsigned base = (unsigned)cur.T * (unsigned)TILE_BYTES + (unsigned)(cur.j * 8 + rank) * BLK;
        u32x4 o[4], o2[4]; f32x4 v[8];
        slab_fetch<2 * ph>(v); slab_convert(cur, v, o);
        slab_fetch<2 * ph + 1>(v); slab_convert(cur, v, o2);
#pragma unroll
        for (int i = 0; i < 4; ++i) __builtin_amdgcn_raw_buffer_store_b128(o[i], rs, voff + i * 1024, (int)base, AUX_SYS);
#pragma unroll
        for (int i = 0; i < 4; ++i) __builtin_amdgcn_raw_buffer_store_b128(o2[i], rs, voff + i * 1024, (int)(base + SLAB), AUX_SYS);
        asm volatile("s_nop 1" ::: "memory"); SCHED;
        prev = cur; cur = nxt;
    };
    for (int b = ib; b < NBLK; b += 2 * NSELF) {
        trip(std::integral_constant<int, 0>{}, b);
        if (b + NSELF < NBLK) trip(std::integral_constant<int, 1>{}, b + NSELF);
    }
    WAIT_V(0); BAR;
    publish();
}

__device__ __forceinline__ void gemm_role(const Ctx& cx, LAS unsigned char* lds, unsigned call) {
    const int tid = threadIdx.x, wid = __builtin_amdgcn_readfirstlane(tid >> 6), lane = tid & 63, wr = wid >> 1, wc = wid & 1;
    const int rank = cx.rank, xcd = blockIdx.x & 7, w = (blockIdx.x - NCOMM) >> 3;
    GAS char* myws = cx.myws;
    GAS unsigned* tmo = (GAS unsigned*)(myws + O_TMO);
    const GAS unsigned* flags = (const GAS unsigned*)(myws + O_FLAG) + lane;
    const __amdgpu_buffer_rsrc_t rsrc = __builtin_amdgcn_make_buffer_rsrc((void*)(myws + CTRL_BYTES + (size_t)(call & 1u) * BUF_BYTES), 0, (int)BUF_BYTES, 0x00020000);
    const __amdgpu_buffer_rsrc_t orsrc = __builtin_amdgcn_make_buffer_rsrc((void*)cx.myout, 0, (int)((size_t)M * N * 4), 0x00020000);
    const int ovoff = (lane >> 4) * 4 * N * 4 + (lane & 15) * 16;
    const int ntiles = w < R0 ? NGRP : NGRP - 1, total = ntiles * NS;
    const int e1 = w < R0 ? R0 - WPX + w : R0 + w;
    auto tile_of = [&](int i) { const int e = e1 + i * WPX; return unit_of(e < R0 ? w : e, xcd, rank); };
    const unsigned voff = (unsigned)(wid * 4096 + lane * 16), ldsw = (unsigned)wid * 4096u;
    const int aoff = wr * 8192 + lane * 16, boff = SLAB + wc * 8192 + lane * 16;
    const int mode = cx.mode;
    int P = (mode & 1) ? total : 0;
    int pt = 0;
#define DMA(slot, ga, gb, i) __builtin_amdgcn_raw_ptr_buffer_load_lds(rsrc, (LAS void*)(lds + (slot) * 2 * SLAB + ((i) < 4 ? 0 : SLAB) + ldsw), 16, \
        (int)voff, (int)((i) < 4 ? (ga) : (gb)), ((i) & 3) * 1024, AUX_LD)
#define RDA(slot, m) (*(const LAS h16x8*)(lds + (slot) * 2 * SLAB + aoff + (m) * 1024))
#define RDB(slot, n) (*(const LAS h16x8*)(lds + (slot) * 2 * SLAB + boff + (n) * 1024))
#define RDF(s, nA_, nB_, f) do { if constexpr ((f) < 8) nB_[(f)] = RDB(((s) + 1) & 3, (f)); else nA_[(f) - 8] = RDA(((s) + 1) & 3, (f) - 8); } while (0)
#define STEP(s, Z, WC, cA_, cB_, nA_, nB_, ga, gb) do { \
        __builtin_amdgcn_s_waitcnt(WC); BAR; SCHED; \
        static_for<32>([&](auto H_) { constexpr int h = decltype(H_)::value; \
            mfma2<2 * h, Z>(cA_[h >> 2], cB_[(h & 3) * 2], cB_[(h & 3) * 2 + 1]); \
            SCHED; \
            if constexpr ((h & 3) == 2) { DMA(s, ga, gb, h >> 2); } \
            else if constexpr (h <= 20) { RDF(s, nA_, nB_, h - ((h + 1) >> 2)); } \
            SCHED; \
        }); } while (0)
#define DMA8(s, ga, gb) do { DMA(s, ga, gb, 0); DMA(s, ga, gb, 1); DMA(s, ga, gb, 2); DMA(s, ga, gb, 3); DMA(s, ga, gb, 4); DMA(s, ga, gb, 5); DMA(s, ga, gb, 6); DMA(s, ga, gb, 7); } while (0)
#define ITER(Z, WC, gp, ga, gb) do { \
        if (__builtin_expect((gp) >= thr, 0)) arrive(gp, 4); \
        STEP(0, Z, WC, A0, B0, A1, B1, ga, gb); STEP(1, false, WC, A1, B1, A0, B0, (ga) + SLAB, (gb) + SLAB); \
        STEP(2, false, WC, A0, B0, A1, B1, (ga) + 2 * SLAB, (gb) + 2 * SLAB); STEP(3, false, 0x4070, A1, B1, A0, B0, (ga) + 3 * SLAB, (gb) + 3 * SLAB); } while (0)
    int ui = 0, thr = 0;
    bool pend = false;
    auto absorb = [&](unsigned fa, unsigned fb) {
        const unsigned long long mask = __ballot(fa >= call && fb >= call);
        const int cnt = mask == ~0ull ? 64 : __builtin_ctzll(~mask);
        if (pt * NS + cnt > P) P = pt * NS + cnt;
        const bool adv = cnt == 64 && pt + 1 < ntiles && pt <= ui;
        if (adv) ++pt;
        return adv;
    };
    auto arrive = [&](int gpos, int ahead) {
        const int need = gpos + ahead < total ? gpos + ahead : total;
        if (wid == 0) {
            const int P0 = P;
            if (pend) { absorb(lds_ld(lds, L_FLG + lane * 4), lds_ld(lds, L_FLG + 256 + lane * 4)); pend = false; }
            if (P < need) {
                const unsigned long long t0 = __builtin_amdgcn_s_memrealtime();
                for (;;) {
                    const Unit u = tile_of(pt);
                    if (absorb(ld_sys(flags + u.pm * NS), ld_sys(flags + (NPM + u.pn) * NS))) continue;
                    if (P >= need) break;
                    if (give_up(tmo, call, t0)) { lds_st(lds, L_BAD, 1u); P = total; break; }
                    __builtin_amdgcn_s_sleep(4);
                }
            }
            if (P != P0) { lds_st(lds, L_P, (unsigned)P); WAIT_L(0); }
            if (P < total && P < gpos + 28 && ahead == 4) {
                const Unit u = tile_of(pt);
                __builtin_amdgcn_global_load_lds(flags + u.pm * NS, (LAS unsigned*)(lds + L_FLG), 4, 0, AUX_SYS);
                __builtin_amdgcn_global_load_lds(flags + (NPM + u.pn) * NS, (LAS unsigned*)(lds + L_FLG + 256), 4, 0, AUX_SYS);
                pend = true;
            }
            thr = P >= total ? 0x7fffffff : P - 27;
        } else {
            while (P < need) { P = __builtin_amdgcn_readfirstlane((int)lds_ld(lds, L_P)); if (P < need) __builtin_amdgcn_s_sleep(2); }
            thr = P >= total ? 0x7fffffff : P - 3;
        }
    };
    Unit cur = tile_of(0), nxt = cur;
    h16x8 A0[8], B0[8], A1[8], B1[8];
    const int lm = (mode & 8) ? 0 : -1;
    unsigned cA = (unsigned)(cur.pm & lm) * (unsigned)TILE_BYTES, cB = (unsigned)(NPM + (cur.pn & lm)) * (unsigned)TILE_BYTES;
    arrive(0, 2);
    DMA8(0, cA, cB); DMA8(1, cA + SLAB, cB + SLAB); DMA8(2, cA + 2 * SLAB, cB + 2 * SLAB); DMA8(3, cA + 3 * SLAB, cB + 3 * SLAB);
    WAIT_V(0); BAR;
#pragma unroll
    for (int i = 0; i < 8; ++i) { A0[i] = RDA(0, i); B0[i] = RDB(0, i); }
    for (;;) {
        const bool has_next = ui + 1 < ntiles;
        if (has_next) nxt = tile_of(ui + 1);
        const unsigned nA = (unsigned)(nxt.pm & lm) * (unsigned)TILE_BYTES, nB = (unsigned)(NPM + (nxt.pn & lm)) * (unsigned)TILE_BYTES;
        const int g0 = ui * NS;
        ITER(true, 0xc07c, g0, cA + 2 * BLK, cB + 2 * BLK);
        for (int t = 2; t < NS - 2; t += 4) {
            const unsigned a2 = cA + (unsigned)(t + 2) * BLK, b2 = cB + (unsigned)(t + 2) * BLK;
            ITER(false, 0x4070, g0 + t, a2, b2);
            ITER(false, 0x4070, g0 + t + 2, a2 + 2 * BLK, b2 + 2 * BLK);
        }
        ITER(false, 0x4070, g0 + NS - 2, nA, nB);
        {
            const float poison = lds_ld(lds, L_BAD) != 0u ? __builtin_nanf("") : -0.0f;
            const unsigned obase = ((unsigned)(cur.pm * 256 + wr * 128) * (unsigned)N + (unsigned)(cur.pn) * 256u + (unsigned)wc * 128u) * 4u;
            asm volatile("s_nop 15\n\ts_nop 15" ::: "memory");
            auto epi = [&](auto WT_) { static_for<16>([&](auto E_) { constexpr int e = decltype(E_)::value, m = e >> 1, g = e & 1;
                float a[16]; acc_read16<16 * e>(a);
#pragma unroll
                for (int r = 0; r < 4; ++r) {
                    const f32x2 lo = gelu2((f32x2){a[r], a[4 + r]}) + poison, hi = gelu2((f32x2){a[8 + r], a[12 + r]}) + poison;
                    const f32x4 v = (f32x4){lo.x, lo.y, hi.x, hi.y};
                    out_store<g * 256, decltype(WT_)::value>(v, ovoff, orsrc, obase + (unsigned)(m * 16 + r) * (unsigned)(N * 4));
                }
            }); };
            if (cur.pn >= NCOM) epi(std::true_type{}); else epi(std::false_type{});
            if (cur.pn >= NCOM) {
                WAIT_V(0);
                if (lane == 0) __hip_atomic_fetch_add((GAS unsigned*)(myws + O_XCNT + (cur.pm * NEX + cur.pn - NCOM - rank * NEX) * 64), 1u, __ATOMIC_RELAXED, __HIP_MEMORY_SCOPE_AGENT);
            }
        }
        if (!has_next) break;
        cur = nxt; cA = nA; cB = nB; ++ui;
    }
    WAIT_V(0);
}

__global__ __launch_bounds__(256, 1) void fused_kernel(Params p) {
    extern __shared__ __attribute__((aligned(16))) unsigned char smem[];
    LAS unsigned char* lds = (LAS unsigned char*)smem;
    Ctx cx; cx.myws = (GAS char*)p.myws; cx.myout = (GAS float*)p.myout; cx.rank = p.rank; cx.mode = p.mode;
    asm volatile("" : "+s"(cx.myws), "+s"(cx.myout), "+s"(cx.rank), "+s"(cx.mode));
    const int bid = blockIdx.x, idle = bid - NCOMM - R0 * 8;
    const bool streams = bid < NCOMM;
    if (streams || idle >= 0) {
        const Blk k = streams ? decode<false>(bid) : decode<true>(idle);
        const int wid = __builtin_amdgcn_readfirstlane(threadIdx.x >> 6);
        slab_load<0>(p, k, 0, wid, threadIdx.x & 63); slab_load<1>(p, k, 1, wid, threadIdx.x & 63);
    }
    if (threadIdx.x < 32) {
        unsigned v = 0;
        if (threadIdx.x == 0) v = __hip_atomic_fetch_add((GAS unsigned*)(cx.myws + O_TICKET), 1u, __ATOMIC_RELAXED, __HIP_MEMORY_SCOPE_AGENT) / (unsigned)GRID + 1u;
        lds_st(lds, L_CALL + threadIdx.x * 4, v);
    }
    __builtin_amdgcn_fence(__ATOMIC_ACQUIRE, "");
    __syncthreads();
    const unsigned call = __builtin_amdgcn_readfirstlane(lds_ld(lds, L_CALL));
    if (cx.mode & 2) WAIT_V(0);
    else {
        if (streams) stream_role(p, cx, lds, call, bid);
        else if (idle >= 0) self_role(p, cx, call, idle);
    }
    if (bid < NCOMM) done_role(p, cx, call);
    else if (!(cx.mode & 4)) gemm_role(cx, lds, call);
}
}

#define FK_MODE 0

extern "C" void kernel_launch(int rank, int nranks, ncclComm_t comm, hipStream_t stream,
                              void* const* d_in, const int* in_sizes, int n_in,
                              void* d_out, int out_size, const int* in_dtypes, int out_dtype,
                              void* const* d_out_all, void* const* d_ws_all, size_t ws_bytes) {
    using namespace fk;
    if (nranks != fk::NR || ws_bytes < WS_NEED || (unsigned)rank >= (unsigned)fk::NR) { fprintf(stderr, "unexpected configuration\n"); return; }
    static bool attr_set = false;
    if (!attr_set) { HIP_CHECK(hipFuncSetAttribute((const void*)fused_kernel, hipFuncAttributeMaxDynamicSharedMemorySize, LDS_BYTES)); attr_set = true; }
    Params p;
    p.x = (const float*)d_in[0]; p.w = (const float*)d_in[1];
    for (int q = 0; q < fk::NR; ++q) { p.out[q] = (float*)d_out_all[q]; p.ws[q] = (char*)d_ws_all[q]; }
    p.out[rank] = (float*)d_out;
    p.myws = p.ws[rank]; p.myout = (float*)d_out; p.rank = rank; p.mode = FK_MODE;
    fused_kernel<<<GRID, 256, LDS_BYTES, stream>>>(p);
    HIP_CHECK(hipGetLastError());
}
```

```cpp
#include <hip/hip_runtime.h>
#include <rccl/rccl.h>
#include <cstdio>
#include <cstdint>
#include <type_traits>

#define HIP_CHECK(expr) do { hipError_t _e = (expr); if (_e != hipSuccess) { \
    fprintf(stderr, "HIP error %s at %s:%d\n", hipGetErrorString(_e), __FILE__, __LINE__); \
    return; } } while (0)

namespace fk {
typedef _Float16 h16x8 __attribute__((ext_vector_type(8)));
typedef float f32x4 __attribute__((ext_vector_type(4)));
typedef unsigned u32x4 __attribute__((ext_vector_type(4)));
#define LAS __attribute__((address_space(3)))
#define GAS __attribute__((address_space(1)))

constexpr int M = 4096, N = 32768, NR = 8, KL = 512;
constexpr int NPM = 16, NPN = 128, NS = 64;
constexpr int NCOMM = 24, GRID = 256, WPX = (GRID - NCOMM) / 8;
constexpr int NEX = 3;
constexpr int NCOM = NPN - NR * NEX;
constexpr int ENT = 2 * NCOM + 2 * NEX;
constexpr int NGRP = (ENT + WPX - 1) / WPX;
constexpr int R0 = ENT - (NGRP - 1) * WPX;
constexpr int EX0 = 2 * R0, ES0 = EX0 + 2 * NEX, EP0 = ES0 + 2 * (R0 & 1), C1 = R0 + (R0 & 1);
constexpr int PP = 8, PG0 = 4, NPUSH = NPM * NEX * 8;
constexpr int NSELF = (WPX - R0) * 8;
static_assert(NPM * NEX * 64 <= 3072 && NCOMM <= 32, "control block layout");
static_assert(NSELF > 0 && EP0 <= R0 + WPX && 8 * PP * (NGRP - PG0) <= NPUSH, "schedule shape");
constexpr unsigned BLK = 32768, SLAB = 16384;
constexpr unsigned TILE_BYTES = 8 * BLK + 4096, LEVEL_BYTES = (NPM + NPN) * TILE_BYTES;
constexpr size_t BUF_BYTES = (size_t)8 * LEVEL_BYTES;
constexpr size_t CTRL_BYTES = 1u << 20;
constexpr size_t WS_NEED = CTRL_BYTES + 2 * BUF_BYTES;
constexpr unsigned O_TICKET = 0, O_TMO = 128, O_BCNT = 256, O_XCNT = 1024  , O_DONE = 4096  ,
                   O_FLAG = 8192  , O_DBG = 65536;
constexpr unsigned long long DEADLINE = 150000000ull;
constexpr int STAGE_BYTES = 4 * 2 * SLAB;
constexpr int L_CALL = STAGE_BYTES, L_P = STAGE_BYTES + 4, L_BAD = STAGE_BYTES + 8, L_TKT = STAGE_BYTES + 16  , L_FLG = STAGE_BYTES + 256  ;
constexpr int EPD = 3;
constexpr int LDS_BYTES = STAGE_BYTES + 1024;
constexpr int AUX_SYS = 17, AUX_PUSH = 19, AUX_LD = 16;

struct Params { const float* x; const float* w; float* out[NR]; char* ws[NR]; char* myws; float* myout; int rank; int mode; };
struct Ctx { GAS char* myws; GAS float* myout; int rank, mode; };

__device__ __forceinline__ unsigned ld_sys(const GAS unsigned* p) { return __hip_atomic_load(p, __ATOMIC_RELAXED, __HIP_MEMORY_SCOPE_SYSTEM); }
__device__ __forceinline__ unsigned ld_agent(const GAS unsigned* p) { return __hip_atomic_load(p, __ATOMIC_RELAXED, __HIP_MEMORY_SCOPE_AGENT); }
__device__ __forceinline__ void st_sys(GAS unsigned* p, unsigned v) { __hip_atomic_store(p, v, __ATOMIC_RELAXED, __HIP_MEMORY_SCOPE_SYSTEM); }
__device__ __forceinline__ void st_agent(GAS unsigned* p, unsigned v) { __hip_atomic_store(p, v, __ATOMIC_RELAXED, __HIP_MEMORY_SCOPE_AGENT); }
__device__ __forceinline__ unsigned lds_ld(LAS unsigned char* lds, int off) { return *(volatile LAS unsigned*)(lds + off); }
__device__ __forceinline__ void lds_st(LAS unsigned char* lds, int off, unsigned v) { *(volatile LAS unsigned*)(lds + off) = v; }
#define WAIT_V_(n) asm volatile("s_nop 1\n\ts_waitcnt vmcnt(" #n ")" ::: "memory")
#define WAIT_V(n) WAIT_V_(n)
#define WAIT_L(n) asm volatile("s_waitcnt lgkmcnt(" #n ")" ::: "memory")
#define BAR __builtin_amdgcn_s_barrier()
#define SCHED __builtin_amdgcn_sched_barrier(0)
#define ACLOB "a0","a1","a2","a3","a4","a5","a6","a7","a8","a9","a10","a11","a12","a13","a14","a15","a16","a17","a18","a19","a20","a21","a22","a23","a24","a25","a26","a27","a28","a29","a30","a31","a32","a33","a34","a35","a36","a37","a38","a39","a40","a41","a42","a43","a44","a45","a46","a47","a48","a49","a50","a51","a52","a53","a54","a55","a56","a57","a58","a59","a60","a61","a62","a63","a64","a65","a66","a67","a68","a69","a70","a71","a72","a73","a74","a75","a76","a77","a78","a79","a80","a81","a82","a83","a84","a85","a86","a87","a88","a89","a90","a91","a92","a93","a94","a95","a96","a97","a98","a99","a100","a101","a102","a103","a104","a105","a106","a107","a108","a109","a110","a111","a112","a113","a114","a115","a116","a117","a118","a119","a120","a121","a122","a123","a124","a125","a126","a127","a128","a129","a130","a131","a132","a133","a134","a135","a136","a137","a138","a139","a140","a141","a142","a143","a144","a145","a146","a147","a148","a149","a150","a151","a152","a153","a154","a155","a156","a157","a158","a159","a160","a161","a162","a163","a164","a165","a166","a167","a168","a169","a170","a171","a172","a173","a174","a175","a176","a177","a178","a179","a180","a181","a182","a183","a184","a185","a186","a187","a188","a189","a190","a191","a192","a193","a194","a195","a196","a197","a198","a199","a200","a201","a202","a203","a204","a205","a206","a207","a208","a209","a210","a211","a212","a213","a214","a215","a216","a217","a218","a219","a220","a221","a222","a223","a224","a225","a226","a227","a228","a229","a230","a231","a232","a233","a234","a235","a236","a237","a238","a239","a240","a241","a242","a243","a244","a245","a246","a247","a248","a249","a250","a251","a252","a253","a254","a255"
template <int I, bool ZERO> __device__ __forceinline__ void mfma2(const h16x8& a, const h16x8& b0, const h16x8& b1) {
    if constexpr (ZERO)
    asm volatile("v_mfma_f32_16x16x32_f16 a[%c3:%c4], %0, %1, 0\n\tv_mfma_f32_16x16x32_f16 a[%c5:%c6], %0, %2, 0"
                 :: "v"(a), "v"(b0), "v"(b1), "i"(4 * I), "i"(4 * I + 3), "i"(4 * I + 4), "i"(4 * I + 7) : ACLOB);
    else
    asm volatile("v_mfma_f32_16x16x32_f16 a[%c3:%c4], %0, %1, a[%c3:%c4]\n\tv_mfma_f32_16x16x32_f16 a[%c5:%c6], %0, %2, a[%c5:%c6]"
                 :: "v"(a), "v"(b0), "v"(b1), "i"(4 * I), "i"(4 * I + 3), "i"(4 * I + 4), "i"(4 * I + 7) : ACLOB);
}
template <int R> __device__ __forceinline__ void acc_read4(float (&v)[4]) {
    asm volatile("v_accvgpr_read_b32 %0, a[%c4]\n\tv_accvgpr_read_b32 %1, a[%c5]\n\tv_accvgpr_read_b32 %2, a[%c6]\n\tv_accvgpr_read_b32 %3, a[%c7]"
                 : "=v"(v[0]), "=v"(v[1]), "=v"(v[2]), "=v"(v[3]) : "i"(R), "i"(R + 4), "i"(R + 8), "i"(R + 12) : ACLOB);
}
template <int... Is> struct Seq {};
template <int N_, int... Is> struct MkSeq : MkSeq<N_ - 1, N_ - 1, Is...> {};
template <int... Is> struct MkSeq<0, Is...> { typedef Seq<Is...> type; };
template <class F, int... Is> __device__ __forceinline__ void static_for_impl(F&& f, Seq<Is...>) { (f(std::integral_constant<int, Is>{}), ...); }
template <int N_, class F> __device__ __forceinline__ void static_for(F&& f) { static_for_impl(f, typename MkSeq<N_>::type{}); }

__device__ __forceinline__ bool give_up(GAS unsigned* tmo, unsigned call, unsigned long long t0) {
    const bool g = (unsigned)__builtin_amdgcn_readfirstlane(ld_agent(tmo)) == call || __builtin_amdgcn_s_memrealtime() - t0 > DEADLINE;
    if (g) st_agent(tmo, call);
    return g;
}

constexpr int grp_start_c(int g) {
    if (g <= 0) return 0;
    if (g == 1) return R0;
    const int r = R0 + (g - 1) * WPX - 1 - EP0;
    return r < 0 ? C1 : C1 + 2 * (r / 4 + 1);
}
struct GrpTab { int s[NGRP + 1]; constexpr GrpTab() : s{} { for (int g = 0; g <= NGRP; ++g) s[g] = grp_start_c(g); } };
constexpr GrpTab GT{};
constexpr int NBLK = 8 * (NPM + NPN);
constexpr int NITEM = 8 * (NPM + NCOM) + NPUSH;
constexpr int TPUSH = NPM + NPN, G0END = 8 * (NPM / 2 + GT.s[1]), DRAIN = 24;
struct Blk { int T, j, dest; };
template <bool SELF> __device__ __forceinline__ Blk decode(int b) {
    Blk r; r.T = 0; r.j = 0; r.dest = -1;
#pragma unroll
    for (int g = 0; g < NGRP; ++g) {
        const int s = GT.s[g], nb = GT.s[g + 1] - s, nc = nb + (g < 2 ? NPM / 2 : 0), mem = nc + (!SELF && g >= PG0 ? PP : 0);
        if (b >= 0 && b < 8 * mem) {
            const int j = b / mem, m = b - j * mem; r.j = j;
            if (m >= nc) r.T = TPUSH + ((g - PG0) * 8 + j) * PP + m - nc;
            else if (g < 2 && m < NPM / 2) r.T = (m >> 1) * 4 + (m & 1) + 2 * g;
            else r.T = NPM + s + m - (g < 2 ? NPM / 2 : 0);
        }
        b -= 8 * mem;
    }
    if (b >= 0) {
        if (SELF) { r.j = b / (NR * NEX); const int m = b - r.j * (NR * NEX); r.dest = m & 7; r.T = NPM + NCOM + r.dest * NEX + (m >> 3); }
        else r.T = TPUSH + (NGRP - PG0) * 8 * PP + b;
    }
    return r;
}
static_assert(GT.s[NGRP] == NCOM, "groups cover every common column tile");
__device__ __forceinline__ unsigned push_off(int u, int kf, int wid, int rank, int i) {
    const unsigned sg = (unsigned)(u * 32 + wid * 8 + kf * 4 + i);
    return sg / NEX * (unsigned)N + ((unsigned)(NCOM + rank * NEX) + sg % NEX) * 256u;
}
template <int SET> __device__ __forceinline__ void slab_load(const Params& p, const Blk& k, int kf, int wid, int lane) {
    const bool isx = k.T < NPM, isp = k.T >= TPUSH;
    const float* src = isx ? p.x + (size_t)(k.T * 256 + wid * 64 + (lane & 15)) * KL + k.j * 64 + kf * 32 + (lane >> 4) * 8
                     : isp ? p.myout + lane * 4
                           : p.w + (size_t)(k.j * 64 + kf * 32 + (lane >> 4) * 8) * N + (size_t)(k.T - NPM) * 256 + wid * 64 + (lane & 15) * 4;
    static_for<8>([&](auto R_) { constexpr int r = decltype(R_)::value;
        const size_t o = isx ? (size_t)((r >> 1) * 16 * KL + (r & 1) * 4) : isp ? (size_t)push_off(k.T - TPUSH, kf, wid, p.rank, r & 3) : (size_t)r * N;
        const float* q = src + o;
        asm volatile("global_load_dwordx4 a[%c1:%c2], %0, off nt" :: "v"(q), "i"(SET * 32 + r * 4), "i"(SET * 32 + r * 4 + 3) : "memory", ACLOB);
    });
}
template <int SET> __device__ __forceinline__ void slab_fetch(f32x4 (&v)[8]) {
    static_for<8>([&](auto R_) { constexpr int r = decltype(R_)::value; float x0, x1, x2, x3;
        asm volatile("v_accvgpr_read_b32 %0, a[%c4]\n\tv_accvgpr_read_b32 %1, a[%c5]\n\tv_accvgpr_read_b32 %2, a[%c6]\n\tv_accvgpr_read_b32 %3, a[%c7]"
                     : "=v"(x0), "=v"(x1), "=v"(x2), "=v"(x3) : "i"(SET * 32 + r * 4), "i"(SET * 32 + r * 4 + 1), "i"(SET * 32 + r * 4 + 2), "i"(SET * 32 + r * 4 + 3) : "memory", ACLOB);
        v[r] = (f32x4){x0, x1, x2, x3};
    });
}
__device__ __forceinline__ void slab_convert(const Blk& k, const f32x4 (&v)[8], u32x4 (&o)[4]) {
    if (k.T >= TPUSH) {
#pragma unroll
        for (int i = 0; i < 4; ++i) o[i] = __builtin_bit_cast(u32x4, v[i]);
    } else if (k.T < NPM) {
#pragma unroll
        for (int i = 0; i < 4; ++i) { h16x8 h;
#pragma unroll
            for (int e = 0; e < 4; ++e) { h[e] = (_Float16)v[2 * i][e]; h[4 + e] = (_Float16)v[2 * i + 1][e]; }
            o[i] = __builtin_bit_cast(u32x4, h); }
    } else {
#pragma unroll
        for (int i = 0; i < 4; ++i) { h16x8 h;
#pragma unroll
            for (int r = 0; r < 8; ++r) h[r] = (_Float16)v[r][i];
            o[i] = __builtin_bit_cast(u32x4, h); }
    }
}

__device__ __forceinline__ void stream_role(const Params& p, const Ctx& cx, LAS unsigned char* lds, unsigned call, int c) {
    const int tid = threadIdx.x, wid = __builtin_amdgcn_readfirstlane(tid >> 6), lane = tid & 63, rank = cx.rank;
    GAS char* myws = cx.myws;
    const size_t bufoff = CTRL_BYTES + (size_t)(call & 1u) * BUF_BYTES;
    __amdgpu_buffer_rsrc_t rs[NR - 1];
#pragma unroll
    for (int q = 0; q < NR - 1; ++q) rs[q] = __builtin_amdgcn_make_buffer_rsrc((void*)(p.ws[(rank + 1 + (q + c * 4 + wid) % 7) & 7] + bufoff), 0, (int)BUF_BYTES, 0x00020000);
    __amdgpu_buffer_rsrc_t ro[NR - 1];
#pragma unroll
    for (int q = 0; q < NR - 1; ++q) ro[q] = __builtin_amdgcn_make_buffer_rsrc((void*)p.out[(rank + 1 + (q + c * 4 + wid) % 7) & 7], 0, (int)((size_t)M * N * 4), 0x00020000);
    GAS unsigned* flag_base = (GAS unsigned*)(p.ws[lane & 7] + O_FLAG);
    GAS unsigned* tmo = (GAS unsigned*)(myws + O_TMO);
    const int voff = wid * 4096 + lane * 16;
    bool xok = (cx.mode & 4) != 0, bad = false;
    auto publish = [&](const Blk& k) {
        if (wid == 0 && k.T < TPUSH && lane < NR && lane != rank) st_sys(flag_base + k.T * NS + k.j * 8 + rank, call);
    };
    auto store_slab = [&](const Blk& k, int kf, u32x4 (&o)[4]) {
        if (k.T < TPUSH) {
            const unsigned base = (unsigned)k.T * (unsigned)TILE_BYTES + (unsigned)k.j * LEVEL_BYTES + (unsigned)rank * BLK + (unsigned)kf * SLAB;
#pragma unroll
            for (int i = 0; i < 4; ++i)
#pragma unroll
                for (int q = 0; q < NR - 1; ++q) __builtin_amdgcn_raw_buffer_store_b128(o[i], rs[q], voff + i * 1024, (int)base, AUX_SYS);
        } else {
            if (bad) {
#pragma unroll
                for (int i = 0; i < 4; ++i) o[i] = (u32x4){0x7fc00000u, 0x7fc00000u, 0x7fc00000u, 0x7fc00000u}; }
#pragma unroll
            for (int i = 0; i < 4; ++i)
#pragma unroll
                for (int q = 0; q < NR - 1; ++q) __builtin_amdgcn_raw_buffer_store_b128(o[i], ro[q], lane * 16, (int)(push_off(k.T - TPUSH, kf, wid, rank, i) * 4u), AUX_PUSH);
        }
    };
    u32x4 o[4]; f32x4 v[8];
    const GAS unsigned* bcnt = (const GAS unsigned*)(myws + O_BCNT + (call & 1u) * 128u);
    if (c == 0 && wid == 0 && lane < 2) st_agent((GAS unsigned*)(myws + O_BCNT + (~call & 1u) * 128u) + lane * 16, 0u);
    asm volatile("v_accvgpr_write_b32 a[65], 1" ::: ACLOB);
    int bc = c, bn = c + NCOMM;
    Blk cur = decode<false>(bc), prev = cur;
    WAIT_V(0); slab_fetch<0>(v); slab_convert(cur, v, o);
    for (int i = 0; bc < NITEM; ++i) {
        const Blk nxt = decode<false>(bn < NITEM ? bn : bc);
        if (nxt.T >= TPUSH && !xok) {
            const GAS unsigned* xc = (const GAS unsigned*)(myws + O_XCNT + (lane < NPM * NEX ? lane : 0) * 64);
            const unsigned long long t0 = __builtin_amdgcn_s_memrealtime();
            while (!__all(ld_agent(xc) >= call * 4u)) { if (give_up(tmo, call, t0)) { bad = true; break; } __builtin_amdgcn_s_sleep(8); }
            xok = true;
        }
        if (wid == 0 && lane == 0 && bn < NITEM) asm volatile("global_atomic_add a[64], %0, a[65], off sc0" :: "v"(bcnt) : "memory", ACLOB);
        slab_load<0>(p, nxt, 0, wid, lane);
        store_slab(cur, 0, o);
        WAIT_V(36);
        BAR;
        if (i > 0) publish(prev);
        slab_fetch<1>(v);
        slab_convert(cur, v, o);
        slab_load<1>(p, nxt, 1, wid, lane);
        store_slab(cur, 1, o);
        const bool drain = bc >= G0END - DRAIN && bc < G0END;
        if (drain) WAIT_V(0); else WAIT_V(63);
        if (wid == 0 && lane == 0) { unsigned tk; asm volatile("v_accvgpr_read_b32 %0, a[64]" : "=v"(tk) :: "memory", ACLOB); lds_st(lds, L_TKT + (i & 1) * 4, bn < NITEM && tk < (unsigned)NITEM ? tk + 2u * (unsigned)NCOMM : (unsigned)NITEM); }
        WAIT_L(0); BAR;
        if (drain) { publish(cur); if (wid == 0 && lane == 0) __hip_atomic_fetch_add((GAS unsigned*)bcnt + 16, 1u, __ATOMIC_RELAXED, __HIP_MEMORY_SCOPE_AGENT); }
        slab_fetch<0>(v);
        slab_convert(nxt, v, o);
        prev = cur; cur = nxt; bc = bn; bn = __builtin_amdgcn_readfirstlane((int)lds_ld(lds, L_TKT + (i & 1) * 4));
    }
    WAIT_V(0); BAR;
    publish(prev);
}

__device__ __forceinline__ void done_role(const Params& p, const Ctx& cx, unsigned call) {
    const int tid = threadIdx.x, wid = __builtin_amdgcn_readfirstlane(tid >> 6), lane = tid & 63, c = blockIdx.x, rank = cx.rank;
    GAS char* myws = cx.myws;
    GAS unsigned* tmo = (GAS unsigned*)(myws + O_TMO);
    if (!(p.mode & 2)) {
        if (wid == 0 && lane < NR && lane != rank) st_sys((GAS unsigned*)(p.ws[lane & 7] + O_DONE) + rank * 32 + c, call);
        if (c == 0) {
            const GAS unsigned* dn = (const GAS unsigned*)(myws + O_DONE) + tid;
            const unsigned long long t0 = __builtin_amdgcn_s_memrealtime();
            for (;;) {
                const bool ok = (tid >> 5) == rank || (tid & 31) >= NCOMM || ld_sys(dn) >= call;
                if (__all(ok)) break;
                if (give_up(tmo, call, t0)) { if (lane == 0) cx.myout[0] = __builtin_nanf(""); break; }
                __builtin_amdgcn_s_sleep(8);
            }
            __builtin_amdgcn_fence(__ATOMIC_ACQUIRE, "");
        }
    }
}

struct Unit { int pm, pn; };
__device__ __forceinline__ Unit unit_of(int e, int xcd, int rank) {
    Unit u;
    const int rb = 4 * (xcd >> 1), par = xcd & 1;
    if (e < EX0) {
        const int h = e >= R0 ? 1 : 0, i = e - h * R0;
        if ((R0 & 1) && i == R0 - 1) { u.pm = rb + 2 * h + par; u.pn = R0 - 1; }
        else { u.pm = rb + 2 * h + (i & 1); u.pn = 2 * (i >> 1) + par; }
    } else if (e < ES0) {
        const int i = e - EX0;
        if ((NEX & 1) && i >= 2 * NEX - 2) { u.pm = 2 * xcd + (i & 1); u.pn = NCOM + rank * NEX + NEX - 1; }
        else { u.pm = rb + (i & 3); u.pn = NCOM + rank * NEX + 2 * (i >> 2) + par; }
    } else if (e < EP0) { u.pm = 2 * xcd + ((e - ES0) & 1); u.pn = R0; }
    else { const int r = e - EP0; u.pm = rb + (r & 3); u.pn = C1 + 2 * (r >> 2) + par; }
    return u;
}
typedef float f32x2 __attribute__((ext_vector_type(2)));
__device__ __forceinline__ f32x2 gelu2(f32x2 y) {
    constexpr float k1 = -2.0f * 0.7978845608028654f * 1.4426950408889634f, k3 = k1 * 0.044715f;
    const f32x2 t = y * ((y * y) * k3 + k1);
    f32x2 d; d.x = __builtin_amdgcn_exp2f(t.x); d.y = __builtin_amdgcn_exp2f(t.y);
    d = d + 1.0f;
    f32x2 r; r.x = __builtin_amdgcn_rcpf(d.x); r.y = __builtin_amdgcn_rcpf(d.y);
    return y * r;
}

template <int OFF, bool WT> __device__ __forceinline__ void out_store(const f32x4& v, int voff, const __amdgpu_buffer_rsrc_t& rs, unsigned soff) {
    if constexpr (WT) asm volatile("buffer_store_dwordx4 %0, %1, %2, %3 offen offset:%c4 sc0 sc1\n\ts_nop 1" :: "v"(v), "v"(voff), "s"(rs), "s"(soff), "i"(OFF) : "memory");
    else asm volatile("buffer_store_dwordx4 %0, %1, %2, %3 offen offset:%c4" "\n\ts_nop 1" :: "v"(v), "v"(voff), "s"(rs), "s"(soff), "i"(OFF) : "memory");
}

__device__ __forceinline__ void self_role(const Params& p, const Ctx& cx, unsigned call, int ib) {
    const int tid = threadIdx.x, wid = __builtin_amdgcn_readfirstlane(tid >> 6), lane = tid & 63, rank = cx.rank;
    const size_t bufoff = CTRL_BYTES + (size_t)(call & 1u) * BUF_BYTES;
    GAS unsigned* flag_base = (GAS unsigned*)(p.ws[lane & 7] + O_FLAG);
    const int voff = wid * 4096 + lane * 16;
    Blk cur = decode<true>(ib), prev = cur; prev.dest = NR;
    bool go = false;
    auto publish = [&]() { if (wid == 0 && lane < NR && lane == (prev.dest < 0 ? rank : prev.dest)) st_sys(flag_base + prev.T * NS + prev.j * 8 + rank, call); };
    auto trip = [&](auto PH, int b) {
        constexpr int ph = decltype(PH)::value;
        const Blk nxt = decode<true>(b + NSELF < NBLK ? b + NSELF : b);
        slab_load<2 * (ph ^ 1)>(p, nxt, 0, wid, lane); slab_load<2 * (ph ^ 1) + 1>(p, nxt, 1, wid, lane);
        WAIT_V(16); BAR;
        publish();
        if (cur.dest >= 0 && !go) {
            const GAS unsigned* bcnt = (const GAS unsigned*)(cx.myws + O_BCNT + (call & 1u) * 128u);
            const unsigned long long t0 = __builtin_amdgcn_s_memrealtime();
            while ((int)__builtin_amdgcn_readfirstlane(ld_agent(bcnt + 16)) < DRAIN && __builtin_amdgcn_s_memrealtime() - t0 < 20000ull) __builtin_amdgcn_s_sleep(16);
            go = true;
        }
        const __amdgpu_buffer_rsrc_t rs = __builtin_amdgcn_make_buffer_rsrc((void*)(p.ws[(cur.dest < 0 ? rank : cur.dest) & 7] + bufoff), 0, (int)BUF_BYTES, 0x00020000);
        const unsigned base = (unsigned)cur.T * (unsigned)TILE_BYTES + (unsigned)cur.j * LEVEL_BYTES + (unsigned)rank * BLK;
        u32x4 o[4], o2[4]; f32x4 v[8];
        slab_fetch<2 * ph>(v); slab_convert(cur, v, o);
        slab_fetch<2 * ph + 1>(v); slab_convert(cur, v, o2);
#pragma unroll
        for (int i = 0; i < 4; ++i) __builtin_amdgcn_raw_buffer_store_b128(o[i], rs, voff + i * 1024, (int)base, AUX_SYS);
#pragma unroll
        for (int i = 0; i < 4; ++i) __builtin_amdgcn_raw_buffer_store_b128(o2[i], rs, voff + i * 1024, (int)(base + SLAB), AUX_SYS);
        asm volatile("s_nop 1" ::: "memory"); SCHED;
        prev = cur; cur = nxt;
    };
    for (int b = ib; b < NBLK; b += 2 * NSELF) {
        trip(std::integral_constant<int, 0>{}, b);
        if (b + NSELF < NBLK) trip(std::integral_constant<int, 1>{}, b + NSELF);
    }
    WAIT_V(0); BAR;
    publish();
}

__device__ __forceinline__ void gemm_role(const Ctx& cx, LAS unsigned char* lds, unsigned call) {
    const int tid = threadIdx.x, wid = __builtin_amdgcn_readfirstlane(tid >> 6), lane = tid & 63, wr = wid >> 1, wc = wid & 1;
    const int rank = cx.rank, xcd = blockIdx.x & 7, w = (blockIdx.x - NCOMM) >> 3;
    GAS char* myws = cx.myws;
    GAS unsigned* tmo = (GAS unsigned*)(myws + O_TMO);
    const GAS unsigned* flags = (const GAS unsigned*)(myws + O_FLAG) + lane;
    const __amdgpu_buffer_rsrc_t rsrc = __builtin_amdgcn_make_buffer_rsrc((void*)(myws + CTRL_BYTES + (size_t)(call & 1u) * BUF_BYTES), 0, (int)BUF_BYTES, 0x00020000);
    const __amdgpu_buffer_rsrc_t orsrc = __builtin_amdgcn_make_buffer_rsrc((void*)cx.myout, 0, (int)((size_t)M * N * 4), 0x00020000);
    const int ovoff = (lane >> 4) * 4 * N * 4 + (lane & 15) * 16;
    const int ntiles = w < R0 ? NGRP : NGRP - 1, total = ntiles * NS;
    const int e1 = w < R0 ? R0 - WPX + w : R0 + w;
    auto tile_of = [&](int i) { const int e = e1 + i * WPX; return unit_of(e < R0 ? w : e, xcd, rank); };
    const unsigned voff = (unsigned)(wid * 4096 + lane * 16), ldsw = (unsigned)wid * 4096u;
    const int aoff = wr * 8192 + lane * 16, boff = SLAB + wc * 8192 + lane * 16;
    const int mode = cx.mode;
    int P = (mode & 1) ? total : 0;
    int pt = 0;
#define DMA(slot, ga, gb, i) __builtin_amdgcn_raw_ptr_buffer_load_lds(rsrc, (LAS void*)(lds + (slot) * 2 * SLAB + ((i) < 4 ? 0 : SLAB) + ldsw), 16, \
        (int)voff, (int)((i) < 4 ? (ga) : (gb)), ((i) & 3) * 1024, AUX_LD)
#define RDA(slot, m) (*(const LAS h16x8*)(lds + (slot) * 2 * SLAB + aoff + (m) * 1024))
#define RDB(slot, n) (*(const LAS h16x8*)(lds + (slot) * 2 * SLAB + boff + (n) * 1024))
#define RDF(s, nA_, nB_, f) do { if constexpr ((f) < 8) nB_[(f)] = RDB(((s) + 1) & 3, (f)); else nA_[(f) - 8] = RDA(((s) + 1) & 3, (f) - 8); } while (0)
#define STEP(s, Z, WC, cA_, cB_, nA_, nB_, ga, gb) do { \
        __builtin_amdgcn_s_waitcnt(WC); BAR; SCHED; \
        static_for<32>([&](auto H_) { constexpr int h = decltype(H_)::value; \
            mfma2<2 * h, Z>(cA_[h >> 2], cB_[(h & 3) * 2], cB_[(h & 3) * 2 + 1]); \
            SCHED; \
            if constexpr ((h & 3) == 2) { DMA(s, ga, gb, h >> 2); } \
            else if constexpr (h <= 20) { RDF(s, nA_, nB_, h - ((h + 1) >> 2)); } \
            SCHED; \
        }); } while (0)
#define DMA8(s, ga, gb) do { DMA(s, ga, gb, 0); DMA(s, ga, gb, 1); DMA(s, ga, gb, 2); DMA(s, ga, gb, 3); DMA(s, ga, gb, 4); DMA(s, ga, gb, 5); DMA(s, ga, gb, 6); DMA(s, ga, gb, 7); } while (0)
#define ITER(Z, WC, gp, ga, gb) do { \
        if (__builtin_expect((gp) >= thr, 0)) arrive(gp, 4); \
        STEP(0, Z, WC, A0, B0, A1, B1, ga, gb); STEP(1, false, WC, A1, B1, A0, B0, (ga) + SLAB, (gb) + SLAB); \
        STEP(2, false, WC, A0, B0, A1, B1, (ga) + 2 * SLAB, (gb) + 2 * SLAB); STEP(3, false, 0x4070, A1, B1, A0, B0, (ga) + 3 * SLAB, (gb) + 3 * SLAB); } while (0)
    int ui = 0, thr = 0;
    bool pend = false;
    auto absorb = [&](unsigned fa, unsigned fb) {
        const unsigned long long mask = __ballot(fa >= call && fb >= call);
        const int cnt = mask == ~0ull ? 64 : __builtin_ctzll(~mask);
        if (pt * NS + cnt > P) P = pt * NS + cnt;
        const bool adv = cnt == 64 && pt + 1 < ntiles && pt <= ui;
        if (adv) ++pt;
        return adv;
    };
    auto arrive = [&](int gpos, int ahead) {
        const int need = gpos + ahead < total ? gpos + ahead : total;
        if (wid == 0) {
            const int P0 = P;
            if (pend) { absorb(lds_ld(lds, L_FLG + lane * 4), lds_ld(lds, L_FLG + 256 + lane * 4)); pend = false; }
            if (P < need) {
                const unsigned long long t0 = __builtin_amdgcn_s_memrealtime();
                for (;;) {
                    const Unit u = tile_of(pt);
                    if (absorb(ld_sys(flags + u.pm * NS), ld_sys(flags + (NPM + u.pn) * NS))) continue;
                    if (P >= need) break;
                    if (give_up(tmo, call, t0)) { lds_st(lds, L_BAD, 1u); P = total; break; }
                    __builtin_amdgcn_s_sleep(4);
                }
            }
            if (P != P0) { lds_st(lds, L_P, (unsigned)P); WAIT_L(0); }
            if (P < total && P < gpos + 28 && ahead == 4) {
                const Unit u = tile_of(pt);
                __builtin_amdgcn_global_load_lds(flags + u.pm * NS, (LAS unsigned*)(lds + L_FLG), 4, 0, AUX_SYS);
                __builtin_amdgcn_global_load_lds(flags + (NPM + u.pn) * NS, (LAS unsigned*)(lds + L_FLG + 256), 4, 0, AUX_SYS);
                pend = true;
            }
            thr = P >= total ? 0x7fffffff : P - 27;
        } else {
            while (P < need) { P = __builtin_amdgcn_readfirstlane((int)lds_ld(lds, L_P)); if (P < need) __builtin_amdgcn_s_sleep(2); }
            thr = P >= total ? 0x7fffffff : P - 3;
        }
    };
    Unit cur = tile_of(0), nxt = cur;
    h16x8 A0[8], B0[8], A1[8], B1[8];
    const int lm = (mode & 8) ? 0 : -1;
    unsigned cA = (unsigned)(cur.pm & lm) * (unsigned)TILE_BYTES, cB = (unsigned)(NPM + (cur.pn & lm)) * (unsigned)TILE_BYTES;
    arrive(0, 2);
    DMA8(0, cA, cB); DMA8(1, cA + SLAB, cB + SLAB); DMA8(2, cA + 2 * SLAB, cB + 2 * SLAB); DMA8(3, cA + 3 * SLAB, cB + 3 * SLAB);
    WAIT_V(0); BAR;
#pragma unroll
    for (int i = 0; i < 8; ++i) { A0[i] = RDA(0, i); B0[i] = RDB(0, i); }
    for (;;) {
        const bool has_next = ui + 1 < ntiles;
        if (has_next) nxt = tile_of(ui + 1);
        const unsigned nA = (unsigned)(nxt.pm & lm) * (unsigned)TILE_BYTES, nB = (unsigned)(NPM + (nxt.pn & lm)) * (unsigned)TILE_BYTES;
        const int g0 = ui * NS;
        ITER(true, 0xc07c, g0, cA + 2 * BLK, cB + 2 * BLK);
        for (int t = 2; t < NS - 2; t += 4) {
            const unsigned so = (unsigned)((t + 2) >> 3) * LEVEL_BYTES + (unsigned)((t + 2) & 7) * BLK, a2 = cA + so, b2 = cB + so;
            ITER(false, 0x4070, g0 + t, a2, b2);
            ITER(false, 0x4070, g0 + t + 2, a2 + 2 * BLK, b2 + 2 * BLK);
        }
        ITER(false, 0x4070, g0 + NS - 2, nA, nB);
        {
            const float poison = lds_ld(lds, L_BAD) != 0u ? __builtin_nanf("") : -0.0f;
            const unsigned obase = ((unsigned)(cur.pm * 256 + wr * 128) * (unsigned)N + (unsigned)(cur.pn) * 256u + (unsigned)wc * 128u) * 4u;
            asm volatile("s_nop 15\n\ts_nop 15" ::: "memory");
            auto epi = [&](auto WT_) {
                float a[EPD + 1][4];
                static_for<EPD>([&](auto Z_) { constexpr int z = decltype(Z_)::value; acc_read4<16 * (z >> 2) + (z & 3)>(a[z]); });
                static_for<64>([&](auto Z_) { constexpr int z = decltype(Z_)::value, e = z >> 2, r = z & 3, m = e >> 1, g = e & 1, y = z + EPD;
                    if constexpr (y < 64) acc_read4<16 * (y >> 2) + (y & 3)>(a[y % (EPD + 1)]);
                    const float (&c)[4] = a[z % (EPD + 1)];
                    const f32x2 lo = gelu2((f32x2){c[0], c[1]}) + poison, hi = gelu2((f32x2){c[2], c[3]}) + poison;
                    const f32x4 v = (f32x4){lo.x, lo.y, hi.x, hi.y};
                    out_store<g * 256, decltype(WT_)::value>(v, ovoff, orsrc, obase + (unsigned)(m * 16 + r) * (unsigned)(N * 4));
                }); };
            if (cur.pn >= NCOM) epi(std::true_type{}); else epi(std::false_type{});
            if (cur.pn >= NCOM) {
                WAIT_V(0);
                if (lane == 0) __hip_atomic_fetch_add((GAS unsigned*)(myws + O_XCNT + (cur.pm * NEX + cur.pn - NCOM - rank * NEX) * 64), 1u, __ATOMIC_RELAXED, __HIP_MEMORY_SCOPE_AGENT);
            }
        }
        if (!has_next) break;
        cur = nxt; cA = nA; cB = nB; ++ui;
    }
    WAIT_V(0);
}

__global__ __launch_bounds__(256, 1) void fused_kernel(Params p) {
    extern __shared__ __attribute__((aligned(16))) unsigned char smem[];
    LAS unsigned char* lds = (LAS unsigned char*)smem;
    Ctx cx; cx.myws = (GAS char*)p.myws; cx.myout = (GAS float*)p.myout; cx.rank = p.rank; cx.mode = p.mode;
    asm volatile("" : "+s"(cx.myws), "+s"(cx.myout), "+s"(cx.rank), "+s"(cx.mode));
    const int bid = blockIdx.x, idle = bid - NCOMM - R0 * 8;
    const bool streams = bid < NCOMM;
    if (streams || idle >= 0) {
        const Blk k = streams ? decode<false>(bid) : decode<true>(idle);
        const int wid = __builtin_amdgcn_readfirstlane(threadIdx.x >> 6);
        slab_load<0>(p, k, 0, wid, threadIdx.x & 63); slab_load<1>(p, k, 1, wid, threadIdx.x & 63);
    }
    if (threadIdx.x < 32) {
        unsigned v = 0;
        if (threadIdx.x == 0) v = __hip_atomic_fetch_add((GAS unsigned*)(cx.myws + O_TICKET), 1u, __ATOMIC_RELAXED, __HIP_MEMORY_SCOPE_AGENT) / (unsigned)GRID + 1u;
        lds_st(lds, L_CALL + threadIdx.x * 4, v);
    }
    __builtin_amdgcn_fence(__ATOMIC_ACQUIRE, "");
    __syncthreads();
    const unsigned call = __builtin_amdgcn_readfirstlane(lds_ld(lds, L_CALL));
    if (cx.mode & 2) WAIT_V(0);
    else {
        if (streams) stream_role(p, cx, lds, call, bid);
        else if (idle >= 0) self_role(p, cx, call, idle);
    }
    if (bid < NCOMM) done_role(p, cx, call);
    else if (!(cx.mode & 4)) gemm_role(cx, lds, call);
}
}

#define FK_MODE 0

extern "C" void kernel_launch(int rank, int nranks, ncclComm_t comm, hipStream_t stream,
                              void* const* d_in, const int* in_sizes, int n_in,
                              void* d_out, int out_size, const int* in_dtypes, int out_dtype,
                              void* const* d_out_all, void* const* d_ws_all, size_t ws_bytes) {
    using namespace fk;
    if (nranks != fk::NR || ws_bytes < WS_NEED || (unsigned)rank >= (unsigned)fk::NR) { fprintf(stderr, "unexpected configuration\n"); return; }
    static bool attr_set = false;
    if (!attr_set) { HIP_CHECK(hipFuncSetAttribute((const void*)fused_kernel, hipFuncAttributeMaxDynamicSharedMemorySize, LDS_BYTES)); attr_set = true; }
    Params p;
    p.x = (const float*)d_in[0]; p.w = (const float*)d_in[1];
    for (int q = 0; q < fk::NR; ++q) { p.out[q] = (float*)d_out_all[q]; p.ws[q] = (char*)d_ws_all[q]; }
    p.out[rank] = (float*)d_out;
    p.myws = p.ws[rank]; p.myout = (float*)d_out; p.rank = rank; p.mode = FK_MODE;
    fused_kernel<<<GRID, 256, LDS_BYTES, stream>>>(p);
    HIP_CHECK(hipGetLastError());
}
```
